```python
import numpy as np

kernel_name = "gpu_gemm_rs_m4096_k4096_n4096_w8a8_dqrelu"


ABSMAX_RTOL = 0.001

COMPUTE_DTYPE = "int8"
EPILOGUE = "dequant_relu"
IN_QUANT = "int8"
OUT_QUANT = None
IN_DTYPES = {"x": "int8", "w_mat": "int8",
             "scale_x": "float32", "scale_w": "float32"}
WORLD_SIZE = 8

def setup_inputs():

    x = np.random.randint(-127, 128, size=(4096, 4096)).astype(np.int8)
    w_mat = np.random.randint(-127, 128, size=(4096, 4096)).astype(np.int8)
    scale_x = (0.02 * (1.0 + np.random.rand(1))).astype(np.float32)
    scale_w = (0.02 * (1.0 + np.random.rand(1))).astype(np.float32)
    return {"x": x, "w_mat": w_mat, "scale_x": scale_x, "scale_w": scale_w}

def reference(x, w_mat, scale_x, scale_w):
    acc = x.astype(np.float64) @ w_mat.astype(np.float64)
    y = (acc.astype(np.float32) * (scale_x * scale_w)).astype(np.float32)
    return np.maximum(y, 0.0).astype(np.float32)
```

```cpp
#include <hip/hip_runtime.h>
#include <rccl/rccl.h>
#include <cstdio>
#include <cstdint>

#define HIP_CHECK(expr) do { hipError_t _e = (expr); if (_e != hipSuccess) { \
    fprintf(stderr, "HIP error %s at %s:%d\n", hipGetErrorString(_e), __FILE__, __LINE__); \
    return; } } while (0)

namespace {

typedef int i32x4 __attribute__((ext_vector_type(4)));
typedef unsigned u32x4 __attribute__((ext_vector_type(4)));
typedef unsigned u32x2 __attribute__((ext_vector_type(2)));
typedef int i32x16 __attribute__((ext_vector_type(16)));
typedef float f32x4 __attribute__((ext_vector_type(4)));
typedef __amdgpu_buffer_rsrc_t rsrc_t;

constexpr int NR = 8;
constexpr int MS = 512, KS = 512;
constexpr int NN = 4096;
constexpr int NWG = 256;

constexpr unsigned OFF_SEQ   = 0;
constexpr unsigned OFF_WCNT  = 32768;
constexpr unsigned OFF_TMO   = 36864;
constexpr unsigned OFF_XFLAG = 65536;
constexpr unsigned OFF_WFLAG = 98304;
constexpr unsigned CTRL_BYTES = 131072;
constexpr unsigned OFF_XBUF  = 2u << 20;
constexpr unsigned OFF_WBUF  = 6u << 20;
constexpr unsigned BUF_BYTES = 2u << 20;
constexpr unsigned OFF_DBG   = 16u << 20;
constexpr size_t   WS_NEED   = 18u << 20;

#define PULL_D 4
#define RING 8
#define XWIN 8
constexpr unsigned long long DEADLINE_TICKS = 100000000ull;

constexpr int AUX_SYS = 17;

struct Params {
    const int8_t* x;
    const int8_t* w;
    const float* sx;
    const float* sw;
    float* out;
    char* ws[NR];
    int rank;
};

#define VMCNT(n) asm volatile("s_waitcnt vmcnt(%0)" :: "i"(n) : "memory")
#define LGKM0() asm volatile("s_waitcnt lgkmcnt(0)" ::: "memory")

__device__ __forceinline__ rsrc_t mk_rsrc(const void* p, unsigned bytes) {
    return __builtin_amdgcn_make_buffer_rsrc(const_cast<void*>(p), 0, bytes, 0x00020000);
}
__device__ __forceinline__ unsigned rfl(unsigned v) { return __builtin_amdgcn_readfirstlane(v); }

#define STAMP(i) do {} while (0)

__device__ __forceinline__ bool wait_flag(rsrc_t ctrl, unsigned off, bool two, unsigned call,
                                          unsigned long long t0) {
    unsigned spins = 0;
    for (;;) {
        u32x2 v = __builtin_amdgcn_raw_buffer_load_b64(ctrl, 0, off, AUX_SYS);
        unsigned a = rfl(v.x), b = rfl(v.y);
        if (a >= call && (!two || b >= call)) return true;
        if (++spins > 256) {
            unsigned tmo = rfl(__builtin_amdgcn_raw_buffer_load_b32(ctrl, 0, OFF_TMO, AUX_SYS));
            if (tmo == call || __builtin_amdgcn_s_memrealtime() - t0 > DEADLINE_TICKS) {
                __builtin_amdgcn_raw_buffer_store_b32(call, ctrl, 0, OFF_TMO, AUX_SYS);
                return false;
            }
            __builtin_amdgcn_s_sleep(8);
        }
        __builtin_amdgcn_s_sleep(1);
    }
}

constexpr int LDS_ACC = 0;
constexpr int LDS_CNT = 32768;
constexpr int LDS_BYTES = 84 * 1024;

__global__ void __launch_bounds__(512, 2) gemm_rs_fused(Params P) {
    __shared__ __attribute__((aligned(16))) unsigned lds[LDS_BYTES / 4];
    const int tid = threadIdx.x, lane = tid & 63;
    const unsigned wave = rfl(tid >> 6);
    const unsigned b = blockIdx.x;
    const unsigned idx = b >> 3;
    const unsigned g = (b & 7) * 2 + (idx >> 4);
    const unsigned rb = idx & 15;
    const unsigned rank = (unsigned)P.rank & 7u;
    const unsigned src = (rank + wave) & 7u;
    const unsigned long long t0 = __builtin_amdgcn_s_memrealtime();
    char* const myws = P.ws[rank];
    const rsrc_t ctrl = mk_rsrc(myws, CTRL_BYTES);
    STAMP(0);

    const bool pusher = (wave == 0) && (rb >= 12);
    const bool stager = (wave == 1) || (wave == 2);

    unsigned seqv = __builtin_amdgcn_raw_buffer_load_b32(ctrl, 0, OFF_SEQ + b * 128, AUX_SYS);
    unsigned call;

    if (pusher) {
        const unsigned id = g * 4 + (rb - 12);
        const unsigned dst = (rank + (id & 7)) & 7u, q = id >> 3;
        const rsrc_t rx = mk_rsrc(P.x + (size_t)dst * MS * KS, MS * KS);
        const unsigned row = lane & 7, c = lane >> 3;
        u32x4 v[32];
#pragma unroll
        for (int r = 0; r < 4; ++r)
#pragma unroll
            for (int e = 0; e < 8; ++e) {
                const unsigned unit = 4 * r + (q >> 1);
                const unsigned soff = unit * 16384 + ((q & 1) * 2 + (e >> 2)) * 128 + (e & 3) * 4096;
                v[r * 8 + e] = __builtin_amdgcn_raw_buffer_load_b128(rx, row * 512 + c * 16, soff, 0);
            }
        *(u32x4*)&lds[tid * 16] = u32x4{0, 0, 0, 0}; *(u32x4*)&lds[tid * 16 + 4] = u32x4{0, 0, 0, 0};
        *(u32x4*)&lds[tid * 16 + 8] = u32x4{0, 0, 0, 0}; *(u32x4*)&lds[tid * 16 + 12] = u32x4{0, 0, 0, 0};
        if (lane < 16) lds[LDS_CNT / 4 + lane] = 0;
        LGKM0();
        __builtin_amdgcn_s_barrier();
        call = rfl(seqv) + 1;
        const rsrc_t rd = mk_rsrc(P.ws[dst] + OFF_XBUF + (call & 1) * BUF_BYTES + rank * (MS * KS), MS * KS);
        const rsrc_t rf = mk_rsrc(P.ws[dst] + OFF_XFLAG + rank * 2048, 2048);
#pragma unroll
        for (int s = 0; s < 32 + XWIN; ++s) {
            if (s >= XWIN) {
                const int done = s - XWIN;
                int younger = (s < 32 ? s : 32) - 1 - done;
                for (int r = 0; r < 4; ++r) { const int it = 8 * r + 7 + XWIN; if (it < s && it > done) ++younger; }
                VMCNT(younger);
                if ((done & 7) == 7) {
                    const unsigned unit = 4 * (done >> 3) + (q >> 1);
                    if (lane == 0) __builtin_amdgcn_raw_buffer_store_b32(call, rf, 0, unit * 128 + (q & 1) * 4, AUX_SYS);
                }
            }
            if (s < 32) {
                const int r = s >> 3, e = s & 7;
                const unsigned unit = 4 * r + (q >> 1);
                const unsigned soff = unit * 16384 + ((q & 1) * 2 + (e >> 2)) * 4096 + (e & 3) * 128;
                __builtin_amdgcn_raw_buffer_store_b128(v[s], rd, c * 512 + row * 16, soff, AUX_SYS);
            }
        }
        STAMP(2);
    } else if (stager) {
        const unsigned h = wave - 1;
        const rsrc_t rw = mk_rsrc(P.w, KS * NN);
        unsigned d[16];
#pragma unroll
        for (int r = 0; r < 16; ++r)
            d[r] = __builtin_amdgcn_raw_buffer_load_b32(rw, lane * 4, (rb * 32 + h * 16 + r) * NN + g * 256, 0);
        *(u32x4*)&lds[tid * 16] = u32x4{0, 0, 0, 0}; *(u32x4*)&lds[tid * 16 + 4] = u32x4{0, 0, 0, 0};
        *(u32x4*)&lds[tid * 16 + 8] = u32x4{0, 0, 0, 0}; *(u32x4*)&lds[tid * 16 + 12] = u32x4{0, 0, 0, 0};
        LGKM0();
        __builtin_amdgcn_s_barrier();
        call = rfl(seqv) + 1;
        u32x4 o[4];
#pragma unroll
        for (int a = 0; a < 4; ++a) {
            const unsigned t0_ = __builtin_amdgcn_perm(d[4 * a + 1], d[4 * a], 0x05010400u);
            const unsigned t1_ = __builtin_amdgcn_perm(d[4 * a + 1], d[4 * a], 0x07030602u);
            const unsigned t2_ = __builtin_amdgcn_perm(d[4 * a + 3], d[4 * a + 2], 0x05010400u);
            const unsigned t3_ = __builtin_amdgcn_perm(d[4 * a + 3], d[4 * a + 2], 0x07030602u);
            o[0][a] = __builtin_amdgcn_perm(t2_, t0_, 0x05040100u);
            o[1][a] = __builtin_amdgcn_perm(t2_, t0_, 0x07060302u);
            o[2][a] = __builtin_amdgcn_perm(t3_, t1_, 0x05040100u);
            o[3][a] = __builtin_amdgcn_perm(t3_, t1_, 0x07060302u);
        }
        STAMP(1);
        const rsrc_t rd = mk_rsrc(myws + OFF_WBUF + (call & 1) * BUF_BYTES + g * 131072, 131072);
#pragma unroll
        for (int j = 0; j < 4; ++j)
            __builtin_amdgcn_raw_buffer_store_b128(o[j], rd, (lane >> 3) * 16384 + (lane & 7) * 16,
                                                   rb * 1024 + h * 512 + j * 128, AUX_SYS);
        VMCNT(0);
        STAMP(2);
        unsigned tk = 0;
        if (lane == 0)
            tk = __hip_atomic_fetch_add((unsigned*)(myws + OFF_WCNT + g * 128), 1u, __ATOMIC_RELAXED, __HIP_MEMORY_SCOPE_AGENT);
        tk = rfl(tk);
        if (tk + 1 == call * 32u) {
            if (lane < NR)
                __hip_atomic_store((unsigned*)(P.ws[lane] + OFF_WFLAG + (rank * 16 + g) * 128), call,
                                   __ATOMIC_RELAXED, __HIP_MEMORY_SCOPE_SYSTEM);
        }
        STAMP(3);
    } else {
        *(u32x4*)&lds[tid * 16] = u32x4{0, 0, 0, 0}; *(u32x4*)&lds[tid * 16 + 4] = u32x4{0, 0, 0, 0};
        *(u32x4*)&lds[tid * 16 + 8] = u32x4{0, 0, 0, 0}; *(u32x4*)&lds[tid * 16 + 12] = u32x4{0, 0, 0, 0};
        if (wave == 0 && lane < 16) lds[LDS_CNT / 4 + lane] = 0;
        LGKM0();
        __builtin_amdgcn_s_barrier();
        call = rfl(seqv) + 1;
        if (wave == 3) {
            __builtin_amdgcn_fence(__ATOMIC_ACQUIRE, "");
            VMCNT(0);
            STAMP(1);
            if (lane == 0) __hip_atomic_store(&lds[LDS_CNT / 4 + 8], 1u, __ATOMIC_RELAXED, __HIP_MEMORY_SCOPE_WORKGROUP);
        }
    }

    bool ok = wait_flag(ctrl, OFF_WFLAG + (src * 16 + g) * 128, false, call, t0);
    STAMP(4);
    while (rfl(__hip_atomic_load(&lds[LDS_CNT / 4 + 8], __ATOMIC_RELAXED, __HIP_MEMORY_SCOPE_WORKGROUP)) == 0)
        __builtin_amdgcn_s_sleep(1);
    asm volatile("" ::: "memory");
    const rsrc_t rwb = mk_rsrc(P.ws[src] + OFF_WBUF + (call & 1) * BUF_BYTES + g * 131072, 131072);
    const unsigned voff = lane * 16;
    u32x4 ring[RING];
#pragma unroll
    for (int f = 0; f < RING; ++f) {
        if (f >= PULL_D) VMCNT(PULL_D - 1);
        ring[f] = __builtin_amdgcn_raw_buffer_load_b128(rwb, voff, f * 1024, 0);
    }
    STAMP(5);
    ok &= wait_flag(ctrl, OFF_XFLAG + (src * 16 + rb) * 128, true, call, t0);
    STAMP(6);
    asm volatile("" ::: "memory");
    if (!ok && lane == 0) __hip_atomic_store(&lds[LDS_CNT / 4 + 9], 1u, __ATOMIC_RELAXED, __HIP_MEMORY_SCOPE_WORKGROUP);
    const rsrc_t rxb = mk_rsrc(myws + OFF_XBUF + (call & 1) * BUF_BYTES + src * (MS * KS) + rb * 16384, 16384);
    u32x4 xf[16];
#pragma unroll
    for (int kb = 0; kb < 16; ++kb) xf[kb] = __builtin_amdgcn_raw_buffer_load_b128(rxb, voff, kb * 1024, 0);

    const unsigned lcol = 4 * (lane & 7) + ((lane >> 3) & 3);
    unsigned* const lacc = &lds[LDS_ACC / 4 + (lane >> 5) * 128 + lcol];

    auto tile = [&](unsigned t, bool last) __attribute__((always_inline)) {
        i32x16 acc = {0, 0, 0, 0, 0, 0, 0, 0, 0, 0, 0, 0, 0, 0, 0, 0};
#pragma unroll
        for (int kb = 0; kb < 16; ++kb) {
            const bool more = !last || (kb + RING < 16);
            if (more) VMCNT(PULL_D - 1);
            __builtin_amdgcn_sched_barrier(0);
            acc = __builtin_amdgcn_mfma_i32_32x32x32_i8((i32x4)xf[kb], (i32x4)ring[kb % RING], acc, 0, 0, 0);
            __builtin_amdgcn_sched_barrier(0);
            if (more) ring[kb % RING] = __builtin_amdgcn_raw_buffer_load_b128(rwb, voff, t * 16384 + (kb + RING) * 1024, 0);
            __builtin_amdgcn_sched_barrier(0);
        }
#pragma unroll
        for (int r = 0; r < 16; ++r)
            __hip_atomic_fetch_add(lacc + t * 1024 + ((r & 3) + 8 * (r >> 2)) * 32, (unsigned)acc[r],
                                   __ATOMIC_RELAXED, __HIP_MEMORY_SCOPE_WORKGROUP);
        LGKM0();
        if (lane == 0) __hip_atomic_fetch_add(&lds[LDS_CNT / 4 + t], 1u, __ATOMIC_RELAXED, __HIP_MEMORY_SCOPE_WORKGROUP);
    };
#pragma unroll 1
    for (unsigned t = 0; t < 7; ++t) {
        tile(t, false);
        if (t == 0) STAMP(7);
        if (t == 3) STAMP(8);
    }
    tile(7, true);
    STAMP(9);

    if (wave == 0) {
        const float s = P.sx[0] * P.sw[0];
        const rsrc_t ro = mk_rsrc(P.out + (size_t)rb * 32 * NN + g * 256, (31 * NN + 256) * 4);
        const unsigned ooff = (lane >> 3) * (NN * 4) + (lane & 7) * 16;
#pragma unroll 1
        for (unsigned t = 0; t < 8; ++t) {
            while (rfl(__hip_atomic_load(&lds[LDS_CNT / 4 + t], __ATOMIC_RELAXED, __HIP_MEMORY_SCOPE_WORKGROUP)) < 8u) {
                if (t < 7) __builtin_amdgcn_s_sleep(4);
            }
            asm volatile("" ::: "memory");
            if (t == 7) STAMP(10);
            const bool bad = __hip_atomic_load(&lds[LDS_CNT / 4 + 9], __ATOMIC_RELAXED, __HIP_MEMORY_SCOPE_WORKGROUP) != 0;
            i32x4 a[4];
            f32x4 y[4];
#pragma unroll
            for (int q = 0; q < 4; ++q) a[q] = *(const i32x4*)&lds[LDS_ACC / 4 + t * 1024 + q * 256 + lane * 4];
#pragma unroll
            for (int q = 0; q < 4; ++q)
#pragma unroll
                for (int e = 0; e < 4; ++e) y[q][e] = bad ? __builtin_nanf("") : fmaxf((float)a[q][e] * s, 0.0f);
            __builtin_amdgcn_sched_barrier(0);
#pragma unroll
            for (int q = 0; q < 4; ++q)
                __builtin_amdgcn_raw_buffer_store_b128((u32x4)y[q], ro, ooff, q * 8 * (NN * 4) + t * 128, AUX_SYS);
            __builtin_amdgcn_sched_barrier(0);
            asm volatile("s_nop 1");
        }
        if (lane == 0) __builtin_amdgcn_raw_buffer_store_b32(call, ctrl, 0, OFF_SEQ + b * 128, AUX_SYS);
        STAMP(11);
    }
}

}

extern "C" void kernel_launch(int rank, int nranks, ncclComm_t comm,
                              hipStream_t stream,
                              void* const* d_in, const int* in_sizes, int n_in,
                              void* d_out, int out_size,
                              const int* in_dtypes, int out_dtype,
                              void* const* d_out_all,
                              void* const* d_ws_all, size_t ws_bytes) {
    if (nranks != NR || (unsigned)rank >= (unsigned)NR || n_in < 4 || ws_bytes < WS_NEED ||
        out_size != MS * NN || in_sizes[0] != 4096 * KS || in_sizes[1] != KS * NN) {
        fprintf(stderr, "unexpected problem shape\n");
        return;
    }
    Params P;
    P.x = (const int8_t*)d_in[0];
    P.w = (const int8_t*)d_in[1];
    P.sx = (const float*)d_in[2];
    P.sw = (const float*)d_in[3];
    P.out = (float*)d_out;
    for (int p = 0; p < NR; ++p) P.ws[p] = (char*)d_ws_all[p];
    P.rank = rank;
    gemm_rs_fused<<<NWG, 512, 0, stream>>>(P);
    HIP_CHECK(hipGetLastError());
}
```

```cpp
#include <hip/hip_runtime.h>
#include <rccl/rccl.h>
#include <cstdio>
#include <cstdint>

#define HIP_CHECK(expr) do { hipError_t _e = (expr); if (_e != hipSuccess) { \
    fprintf(stderr, "HIP error %s at %s:%d\n", hipGetErrorString(_e), __FILE__, __LINE__); \
    return; } } while (0)

namespace {

typedef int i32x4 __attribute__((ext_vector_type(4)));
typedef unsigned u32x4 __attribute__((ext_vector_type(4)));
typedef unsigned u32x2 __attribute__((ext_vector_type(2)));
typedef int i32x16 __attribute__((ext_vector_type(16)));
typedef float f32x4 __attribute__((ext_vector_type(4)));
typedef __amdgpu_buffer_rsrc_t rsrc_t;

constexpr int NR = 8;
constexpr int MS = 512, KS = 512;
constexpr int NN = 4096;
constexpr int NWG = 256;

constexpr unsigned OFF_SEQ   = 0;
constexpr unsigned OFF_WCNT  = 32768;
constexpr unsigned OFF_TMO   = 36864;
constexpr unsigned OFF_INV   = 40960;
constexpr unsigned OFF_XFLAG = 65536;
constexpr unsigned OFF_WFLAG = 98304;
constexpr unsigned CTRL_BYTES = 131072;
constexpr unsigned OFF_XBUF  = 2u << 20;
constexpr unsigned OFF_WBUF  = 6u << 20;
constexpr unsigned BUF_BYTES = 2u << 20;
constexpr unsigned OFF_DBG   = 16u << 20;
constexpr size_t   WS_NEED   = 18u << 20;

#define PULL_D 4
#define RING 8
#define LAST_D PULL_D
#define LAST_N 8
#define XWIN 8
constexpr unsigned long long DEADLINE_TICKS = 100000000ull;

constexpr int AUX_SYS = 17;

struct Params {
    const int8_t* x;
    const int8_t* w;
    const float* sx;
    const float* sw;
    float* out;
    char* ws[NR];
    int rank;
};

#define VMCNT(n) asm volatile("s_waitcnt vmcnt(%0)" :: "i"(n) : "memory")
#define LGKM0() asm volatile("s_waitcnt lgkmcnt(0)" ::: "memory")

__device__ __forceinline__ rsrc_t mk_rsrc(const void* p, unsigned bytes) {
    return __builtin_amdgcn_make_buffer_rsrc(const_cast<void*>(p), 0, bytes, 0x00020000);
}
__device__ __forceinline__ unsigned rfl(unsigned v) { return __builtin_amdgcn_readfirstlane(v); }

#define STAMP(i) do {} while (0)

__device__ __forceinline__ unsigned wait_flags(rsrc_t ctrl, unsigned offw, unsigned offx, unsigned need,
                                               unsigned call, unsigned long long t0) {
    unsigned spins = 0;
    for (;;) {
        asm volatile("" ::: "memory");
        const unsigned wv = __builtin_amdgcn_raw_buffer_load_b32(ctrl, 0, offw, AUX_SYS);
        const u32x2 xv = __builtin_amdgcn_raw_buffer_load_b64(ctrl, 0, offx, AUX_SYS);
        const unsigned a = rfl(xv.x), c = rfl(xv.y);
        const unsigned ready = (rfl(wv) >= call ? 1u : 0u) | ((a >= call && c >= call) ? 2u : 0u);
        if (ready & need) return ready;
        if (++spins > 256) {
            const unsigned tmo = rfl(__builtin_amdgcn_raw_buffer_load_b32(ctrl, 0, OFF_TMO, AUX_SYS));
            if (tmo == call || __builtin_amdgcn_s_memrealtime() - t0 > DEADLINE_TICKS) {
                __builtin_amdgcn_raw_buffer_store_b32(call, ctrl, 0, OFF_TMO, AUX_SYS);
                return 7u;
            }
            __builtin_amdgcn_s_sleep(8);
        }
        __builtin_amdgcn_s_sleep(1);
    }
}

constexpr int LDS_ACC = 0;
constexpr int LDS_CNT = 32768;
constexpr int LDS_BYTES = 84 * 1024;

__global__ void __launch_bounds__(512, 2) gemm_rs_fused(Params P) {
    __shared__ __attribute__((aligned(16))) unsigned lds[LDS_BYTES / 4];
    const int tid = threadIdx.x, lane = tid & 63;
    const unsigned wave = rfl(tid >> 6);
    const unsigned b = blockIdx.x;
    const unsigned idx = b >> 3;
    const unsigned g = (b & 7) * 2 + (idx >> 4);
    const unsigned rb = idx & 15;
    const unsigned rank = (unsigned)P.rank & 7u;
    const unsigned src = (rank + wave) & 7u;
    const unsigned long long t0 = __builtin_amdgcn_s_memrealtime();
    char* const myws = P.ws[rank];
    const rsrc_t ctrl = mk_rsrc(myws, CTRL_BYTES);
    STAMP(0);

    const bool pusher = (wave == 0) && (rb >= 12);
    const bool stager = (wave >= 1) && (wave <= 4) && (rb < 8);

    unsigned seqv = __builtin_amdgcn_raw_buffer_load_b32(ctrl, 0, OFF_SEQ + b * 128, AUX_SYS);
    unsigned call;

    if (pusher) {
        const unsigned id = g * 4 + (rb - 12);
        const unsigned dst = (rank + (id & 7)) & 7u, q = id >> 3;
        const rsrc_t rx = mk_rsrc(P.x + (size_t)dst * MS * KS, MS * KS);
        const unsigned row = lane & 7, c = lane >> 3;
        u32x4 v[32];
#pragma unroll
        for (int r = 0; r < 4; ++r)
#pragma unroll
            for (int e = 0; e < 8; ++e) {
                const unsigned unit = 4 * r + (q >> 1);
                const unsigned soff = unit * 16384 + ((q & 1) * 2 + (e >> 2)) * 128 + (e & 3) * 4096;
                v[r * 8 + e] = __builtin_amdgcn_raw_buffer_load_b128(rx, row * 512 + c * 16, soff, 0);
            }
        *(u32x4*)&lds[tid * 16] = u32x4{0, 0, 0, 0}; *(u32x4*)&lds[tid * 16 + 4] = u32x4{0, 0, 0, 0};
        *(u32x4*)&lds[tid * 16 + 8] = u32x4{0, 0, 0, 0}; *(u32x4*)&lds[tid * 16 + 12] = u32x4{0, 0, 0, 0};
        if (lane < 16) lds[LDS_CNT / 4 + lane] = 0;
        LGKM0();
        __builtin_amdgcn_s_barrier();
        call = rfl(seqv) + 1;
        const rsrc_t rd = mk_rsrc(P.ws[dst] + OFF_XBUF + (call & 1) * BUF_BYTES + rank * (MS * KS), MS * KS);
        const rsrc_t rf = mk_rsrc(P.ws[dst] + OFF_XFLAG + rank * 2048, 2048);
#pragma unroll
        for (int s = 0; s < 32 + XWIN; ++s) {
            if (s >= XWIN) {
                const int done = s - XWIN;
                int younger = (s < 32 ? s : 32) - 1 - done;
                for (int r = 0; r < 4; ++r) { const int it = 8 * r + 7 + XWIN; if (it < s && it > done) ++younger; }
                VMCNT(younger);
                if ((done & 7) == 7) {
                    const unsigned unit = 4 * (done >> 3) + (q >> 1);
                    if (lane == 0) __builtin_amdgcn_raw_buffer_store_b32(call, rf, 0, unit * 128 + (q & 1) * 4, AUX_SYS);
                }
            }
            if (s < 32) {
                const int r = s >> 3, e = s & 7;
                const unsigned unit = 4 * r + (q >> 1);
                const unsigned soff = unit * 16384 + ((q & 1) * 2 + (e >> 2)) * 4096 + (e & 3) * 128;
                __builtin_amdgcn_raw_buffer_store_b128(v[s], rd, c * 512 + row * 16, soff, AUX_SYS);
            }
        }
        STAMP(2);
    } else if (stager) {
        const unsigned h = (wave - 1) & 1, skb = rb * 2 + ((wave - 1) >> 1);
        const rsrc_t rw = mk_rsrc(P.w, KS * NN);
        unsigned d[16];
#pragma unroll
        for (int r = 0; r < 16; ++r)
            d[r] = __builtin_amdgcn_raw_buffer_load_b32(rw, lane * 4, (skb * 32 + h * 16 + r) * NN + g * 256, 0);
        *(u32x4*)&lds[tid * 16] = u32x4{0, 0, 0, 0}; *(u32x4*)&lds[tid * 16 + 4] = u32x4{0, 0, 0, 0};
        *(u32x4*)&lds[tid * 16 + 8] = u32x4{0, 0, 0, 0}; *(u32x4*)&lds[tid * 16 + 12] = u32x4{0, 0, 0, 0};
        LGKM0();
        __builtin_amdgcn_s_barrier();
        call = rfl(seqv) + 1;
        u32x4 o[4];
#pragma unroll
        for (int a = 0; a < 4; ++a) {
            const unsigned t0_ = __builtin_amdgcn_perm(d[4 * a + 1], d[4 * a], 0x05010400u);
            const unsigned t1_ = __builtin_amdgcn_perm(d[4 * a + 1], d[4 * a], 0x07030602u);
            const unsigned t2_ = __builtin_amdgcn_perm(d[4 * a + 3], d[4 * a + 2], 0x05010400u);
            const unsigned t3_ = __builtin_amdgcn_perm(d[4 * a + 3], d[4 * a + 2], 0x07030602u);
            o[0][a] = __builtin_amdgcn_perm(t2_, t0_, 0x05040100u);
            o[1][a] = __builtin_amdgcn_perm(t2_, t0_, 0x07060302u);
            o[2][a] = __builtin_amdgcn_perm(t3_, t1_, 0x05040100u);
            o[3][a] = __builtin_amdgcn_perm(t3_, t1_, 0x07060302u);
        }
        STAMP(1);
        const rsrc_t rd = mk_rsrc(myws + OFF_WBUF + (call & 1) * BUF_BYTES + g * 131072, 131072);
#pragma unroll
        for (int j = 0; j < 4; ++j)
            __builtin_amdgcn_raw_buffer_store_b128(o[j], rd, (lane >> 3) * 16384 + (lane & 7) * 16,
                                                   skb * 1024 + h * 512 + j * 128, AUX_SYS);
        VMCNT(0);
        STAMP(2);
        unsigned tk = 0;
        if (lane == 0)
            tk = __hip_atomic_fetch_add((unsigned*)(myws + OFF_WCNT + g * 128), 1u, __ATOMIC_RELAXED, __HIP_MEMORY_SCOPE_AGENT);
        tk = rfl(tk);
        if (tk + 1 == call * 32u) {
            if (lane < NR)
                __hip_atomic_store((unsigned*)(P.ws[lane] + OFF_WFLAG + (rank * 16 + g) * 128), call,
                                   __ATOMIC_RELAXED, __HIP_MEMORY_SCOPE_SYSTEM);
        }
        STAMP(3);
    } else {
        *(u32x4*)&lds[tid * 16] = u32x4{0, 0, 0, 0}; *(u32x4*)&lds[tid * 16 + 4] = u32x4{0, 0, 0, 0};
        *(u32x4*)&lds[tid * 16 + 8] = u32x4{0, 0, 0, 0}; *(u32x4*)&lds[tid * 16 + 12] = u32x4{0, 0, 0, 0};
        if (wave == 0 && lane < 16) lds[LDS_CNT / 4 + lane] = 0;
        LGKM0();
        __builtin_amdgcn_s_barrier();
        call = rfl(seqv) + 1;
        if (wave == 5) {
            const unsigned xcc = __builtin_amdgcn_s_getreg(20 | (3 << 11)) & 15u;
            bool mine = idx == 0;
            while (!mine) {
                asm volatile("" ::: "memory");
                if (rfl(__builtin_amdgcn_raw_buffer_load_b32(ctrl, 0, OFF_INV + xcc * 128, AUX_SYS)) >= call) break;
                if (__builtin_amdgcn_s_memrealtime() - t0 > 1000ull) mine = true;
            }
            if (mine) {
                __builtin_amdgcn_fence(__ATOMIC_ACQUIRE, "");
                VMCNT(0);
                if (lane == 0) __builtin_amdgcn_raw_buffer_store_b32(call, ctrl, 0, OFF_INV + xcc * 128, AUX_SYS);
            }
            STAMP(1);
            if (lane == 0) __hip_atomic_store(&lds[LDS_CNT / 4 + 8], 1u, __ATOMIC_RELAXED, __HIP_MEMORY_SCOPE_WORKGROUP);
        }
    }

    const unsigned offw = OFF_WFLAG + (src * 16 + g) * 128, offx = OFF_XFLAG + (src * 16 + rb) * 128;
    unsigned st = wait_flags(ctrl, offw, offx, 3u, call, t0);
    STAMP(4);
    while (rfl(__hip_atomic_load(&lds[LDS_CNT / 4 + 8], __ATOMIC_RELAXED, __HIP_MEMORY_SCOPE_WORKGROUP)) == 0)
        __builtin_amdgcn_s_sleep(1);
    asm volatile("" ::: "memory");
    const rsrc_t rwb = mk_rsrc(P.ws[src] + OFF_WBUF + (call & 1) * BUF_BYTES + g * 131072, 131072);
    const rsrc_t rxb = mk_rsrc(myws + OFF_XBUF + (call & 1) * BUF_BYTES + src * (MS * KS) + rb * 16384, 16384);
    const unsigned voff = lane * 16;
    const bool remote = wave != 0;
    u32x4 ring[RING];
    u32x4 xf[16];
    if (st & 2u) {
        if (st & 1u) {
#pragma unroll
            for (int f = 0; f < PULL_D; ++f) ring[f] = __builtin_amdgcn_raw_buffer_load_b128(rwb, voff, f * 1024, 0);
        }
        asm volatile("" ::: "memory");
#pragma unroll
        for (int kb = 0; kb < 16; ++kb) xf[kb] = __builtin_amdgcn_raw_buffer_load_b128(rxb, voff, kb * 1024, 0);
        if (!(st & 1u)) {
            st = wait_flags(ctrl, offw, offx, 1u, call, t0);
            asm volatile("" ::: "memory");
#pragma unroll
            for (int f = 0; f < PULL_D; ++f) ring[f] = __builtin_amdgcn_raw_buffer_load_b128(rwb, voff, f * 1024, 0);
        }
        STAMP(5);
#pragma unroll
        for (int f = PULL_D; f < RING; ++f) {
            VMCNT(PULL_D - 1);
            ring[f] = __builtin_amdgcn_raw_buffer_load_b128(rwb, voff, f * 1024, 0);
        }
    } else {
#pragma unroll
        for (int f = 0; f < RING; ++f) {
            if (f >= PULL_D) VMCNT(PULL_D - 1);
            ring[f] = __builtin_amdgcn_raw_buffer_load_b128(rwb, voff, f * 1024, 0);
        }
        st = wait_flags(ctrl, offw, offx, 2u, call, t0);
        STAMP(5);
        asm volatile("" ::: "memory");
#pragma unroll
        for (int kb = 0; kb < 16; ++kb) xf[kb] = __builtin_amdgcn_raw_buffer_load_b128(rxb, voff, kb * 1024, 0);
    }
    STAMP(6);
    if ((st & 4u) && lane == 0) __hip_atomic_store(&lds[LDS_CNT / 4 + 9], 1u, __ATOMIC_RELAXED, __HIP_MEMORY_SCOPE_WORKGROUP);

    const unsigned lcol = 4 * (lane & 7) + ((lane >> 3) & 3);
    unsigned* const lacc = &lds[LDS_ACC / 4 + (lane >> 5) * 128 + lcol];

    auto tile = [&](unsigned t, int mode) __attribute__((always_inline)) {
        i32x16 acc = {0, 0, 0, 0, 0, 0, 0, 0, 0, 0, 0, 0, 0, 0, 0, 0};
#pragma unroll
        for (int kb = 0; kb < 16; ++kb) {
            const int fl = (mode == 0 ? 0 : mode == 1 ? 96 : 112) + kb + RING;
            const bool more = fl < 128;
            if (more && remote) { if (fl >= 128 - LAST_N) VMCNT(LAST_D - 1); else VMCNT(PULL_D - 1); }
            __builtin_amdgcn_sched_barrier(0);
            acc = __builtin_amdgcn_mfma_i32_32x32x32_i8((i32x4)xf[kb], (i32x4)ring[kb % RING], acc, 0, 0, 0);
            __builtin_amdgcn_sched_barrier(0);
            if (more) ring[kb % RING] = __builtin_amdgcn_raw_buffer_load_b128(rwb, voff, t * 16384 + (kb + RING) * 1024, 0);
            __builtin_amdgcn_sched_barrier(0);
        }
#pragma unroll
        for (int r = 0; r < 16; ++r)
            __hip_atomic_fetch_add(lacc + t * 1024 + ((r & 3) + 8 * (r >> 2)) * 32, (unsigned)acc[r],
                                   __ATOMIC_RELAXED, __HIP_MEMORY_SCOPE_WORKGROUP);
        LGKM0();
        if (lane == 0) __hip_atomic_fetch_add(&lds[LDS_CNT / 4 + t], 1u, __ATOMIC_RELAXED, __HIP_MEMORY_SCOPE_WORKGROUP);
    };
#pragma unroll 1
    for (unsigned t = 0; t < 6; ++t) {
        tile(t, 0);
        if (t == 0) STAMP(7);
        if (t == 3) STAMP(8);
    }
    tile(6, 1);
    tile(7, 2);
    STAMP(9);

    if (wave == 0) {
        const float s = P.sx[0] * P.sw[0];
        const rsrc_t ro = mk_rsrc(P.out + (size_t)rb * 32 * NN + g * 256, (31 * NN + 256) * 4);
        const unsigned ooff = (lane >> 3) * (NN * 4) + (lane & 7) * 16;
#pragma unroll 1
        for (unsigned t = 0; t < 8; ++t) {
            while (rfl(__hip_atomic_load(&lds[LDS_CNT / 4 + t], __ATOMIC_RELAXED, __HIP_MEMORY_SCOPE_WORKGROUP)) < 8u) {
                if (t < 7) __builtin_amdgcn_s_sleep(4);
            }
            asm volatile("" ::: "memory");
            if (t == 7) STAMP(10);
            const bool bad = __hip_atomic_load(&lds[LDS_CNT / 4 + 9], __ATOMIC_RELAXED, __HIP_MEMORY_SCOPE_WORKGROUP) != 0;
            i32x4 a[4];
            f32x4 y[4];
#pragma unroll
            for (int q = 0; q < 4; ++q) a[q] = *(const i32x4*)&lds[LDS_ACC / 4 + t * 1024 + q * 256 + lane * 4];
#pragma unroll
            for (int q = 0; q < 4; ++q)
#pragma unroll
                for (int e = 0; e < 4; ++e) y[q][e] = bad ? __builtin_nanf("") : fmaxf((float)a[q][e] * s, 0.0f);
            __builtin_amdgcn_sched_barrier(0);
#pragma unroll
            for (int q = 0; q < 4; ++q)
                __builtin_amdgcn_raw_buffer_store_b128((u32x4)y[q], ro, ooff, q * 8 * (NN * 4) + t * 128, AUX_SYS);
            __builtin_amdgcn_sched_barrier(0);
            asm volatile("s_nop 1");
        }
        if (lane == 0) __builtin_amdgcn_raw_buffer_store_b32(call, ctrl, 0, OFF_SEQ + b * 128, AUX_SYS);
        STAMP(11);
    }
}

}

extern "C" void kernel_launch(int rank, int nranks, ncclComm_t comm,
                              hipStream_t stream,
                              void* const* d_in, const int* in_sizes, int n_in,
                              void* d_out, int out_size,
                              const int* in_dtypes, int out_dtype,
                              void* const* d_out_all,
                              void* const* d_ws_all, size_t ws_bytes) {
    if (nranks != NR || (unsigned)rank >= (unsigned)NR || n_in < 4 || ws_bytes < WS_NEED ||
        out_size != MS * NN || in_sizes[0] != 4096 * KS || in_sizes[1] != KS * NN) {
        fprintf(stderr, "unexpected problem shape\n");
        return;
    }
    Params P;
    P.x = (const int8_t*)d_in[0];
    P.w = (const int8_t*)d_in[1];
    P.sx = (const float*)d_in[2];
    P.sw = (const float*)d_in[3];
    P.out = (float*)d_out;
    for (int p = 0; p < NR; ++p) P.ws[p] = (char*)d_ws_all[p];
    P.rank = rank;
    gemm_rs_fused<<<NWG, 512, 0, stream>>>(P);
    HIP_CHECK(hipGetLastError());
}
```

```cpp
#include <hip/hip_runtime.h>
#include <rccl/rccl.h>
#include <cstdio>
#include <cstdint>

#define HIP_CHECK(expr) do { hipError_t _e = (expr); if (_e != hipSuccess) { \
    fprintf(stderr, "HIP error %s at %s:%d\n", hipGetErrorString(_e), __FILE__, __LINE__); \
    return; } } while (0)

namespace {

typedef int i32x4 __attribute__((ext_vector_type(4)));
typedef unsigned u32x4 __attribute__((ext_vector_type(4)));
typedef unsigned u32x2 __attribute__((ext_vector_type(2)));
typedef int i32x16 __attribute__((ext_vector_type(16)));
typedef float f32x4 __attribute__((ext_vector_type(4)));
typedef __amdgpu_buffer_rsrc_t rsrc_t;
typedef __attribute__((address_space(1))) unsigned gu32;
typedef __attribute__((address_space(1))) unsigned long long gu64;

constexpr int NR = 8;
constexpr int MS = 512, KS = 512;
constexpr int NN = 4096;
constexpr int NWG = 256;

constexpr unsigned REG_HDR = 8192, REG_DATA = 262144, REG_BYTES = REG_HDR + 2 * REG_DATA;
constexpr unsigned HDR_XFLAG = 0, HDR_WFLAG = 2048, HDR_WARM = 4096;
constexpr unsigned OFF_XREG = 0, OFF_WREG = 6u << 20;
__host__ __device__ constexpr unsigned reg_off(unsigned i) {
    return ((i * 11u) >> 5) * (2u << 20) + (i - ((i * 11u) >> 5) * 3u) * REG_BYTES;
}
constexpr unsigned OFF_CTRL  = 0x190000;
constexpr unsigned OFF_SEQ   = 0;
constexpr unsigned OFF_WCNT  = 32768;
constexpr unsigned OFF_TMO   = 36864;
constexpr unsigned OFF_INV   = 40960;
constexpr unsigned CTRL_BYTES = 65536;
constexpr unsigned OFF_DBG   = 16u << 20;
constexpr size_t   WS_NEED   = 18u << 20;
static_assert(3 * REG_BYTES <= OFF_CTRL && OFF_CTRL + CTRL_BYTES <= (2u << 20), "layout");

#define PULL_D 4
#define RING 8
#define LAG_P 2
#define LAG_T 2
#define REMAP 1
#define LAST_AUX AUX_SYS
#define POLL_SLEEP 1
#define POLL_START 0
#define BAR_AT XWIN
#define WARM_DELAY 2
#define XWIN 10
constexpr unsigned long long DEADLINE_TICKS = 100000000ull;

constexpr int AUX_SYS = 17;

struct Params {
    const int8_t* x;
    const int8_t* w;
    const float* sx;
    const float* sw;
    float* out;
    char* ws[NR];
    int rank;
};

#define VMCNT(n) asm volatile("s_waitcnt vmcnt(%0)" :: "i"(n) : "memory")
#define LGKM0() asm volatile("s_waitcnt lgkmcnt(0)" ::: "memory")

__device__ __forceinline__ rsrc_t mk_rsrc(const void* p, unsigned bytes) {
    return __builtin_amdgcn_make_buffer_rsrc(const_cast<void*>(p), 0, bytes, 0x00020000);
}
__device__ __forceinline__ unsigned rfl(unsigned v) { return __builtin_amdgcn_readfirstlane(v); }

#define STAMP(i) do {} while (0)

__device__ __forceinline__ unsigned wait_flags(rsrc_t ctrl, rsrc_t hdr, unsigned offw, unsigned offx, unsigned need,
                                               unsigned call, unsigned long long t0) {
    unsigned spins = 0;
    for (;;) {
        asm volatile("" ::: "memory");
        const unsigned wv = __builtin_amdgcn_raw_buffer_load_b32(hdr, 0, offw, AUX_SYS);
        const u32x2 xv = __builtin_amdgcn_raw_buffer_load_b64(hdr, 0, offx, AUX_SYS);
        const unsigned a = rfl(xv.x), c = rfl(xv.y);
        const unsigned ready = (rfl(wv) >= call ? 1u : 0u) | ((a >= call && c >= call) ? 2u : 0u);
        if (ready & need) return ready;
        if (++spins > 256) {
            const unsigned tmo = rfl(__builtin_amdgcn_raw_buffer_load_b32(ctrl, 0, OFF_TMO, AUX_SYS));
            if (tmo == call || __builtin_amdgcn_s_memrealtime() - t0 > DEADLINE_TICKS) {
                __builtin_amdgcn_raw_buffer_store_b32(call, ctrl, 0, OFF_TMO, AUX_SYS);
                return 7u;
            }
            __builtin_amdgcn_s_sleep(8);
        }
        __builtin_amdgcn_s_sleep(POLL_SLEEP);
    }
}

constexpr int LDS_ACC = 0;
constexpr int LDS_CNT = 32768;
constexpr int LDS_BYTES = 84 * 1024;

#define LDS_INIT() do { \
    *(u32x4*)&lds[tid * 16] = u32x4{0, 0, 0, 0}; *(u32x4*)&lds[tid * 16 + 4] = u32x4{0, 0, 0, 0}; \
    *(u32x4*)&lds[tid * 16 + 8] = u32x4{0, 0, 0, 0}; *(u32x4*)&lds[tid * 16 + 12] = u32x4{0, 0, 0, 0}; } while (0)

__global__ void __launch_bounds__(512, 2) gemm_rs_fused(Params P) {
    __shared__ __attribute__((aligned(16))) unsigned lds[LDS_BYTES / 4];
    const int tid = threadIdx.x, lane = tid & 63;
    const unsigned wave = rfl(tid >> 6);
    const unsigned b = blockIdx.x;
    const unsigned idx = b >> 3;
    const unsigned gh = idx & 1;
    const unsigned rb = 15 - (idx >> 1);
    const bool invwg = idx == 8;
    const unsigned g = (b & 7) * 2 + gh;
    if (wave == 5 && invwg) { __builtin_amdgcn_fence(__ATOMIC_ACQUIRE, ""); }
    const unsigned pj = gh * 4 + (rb & 3);
    unsigned long long t0 = __builtin_amdgcn_s_memrealtime();
    char* myws = P.ws[0];
    char* wsv = P.ws[wave];
    char* wsj = P.ws[pj];
    const int8_t* px = P.x;
    const int8_t* pw = P.w;
    int prank = P.rank;
    asm volatile("" : "+s"(myws), "+s"(wsv), "+s"(wsj), "+s"(px), "+s"(pw), "+s"(prank), "+s"(t0));
    const rsrc_t ctrl = mk_rsrc(myws + OFF_CTRL, CTRL_BYTES);

    const bool pusher = (wave == 0) && (rb >= 12);
    const bool stager = (wave >= 1) && (wave <= 4) && (rb < 8);
    const bool warmer = rb >= 8 && rb < 12 && wave < 2;
    char* const xr = wsj + OFF_XREG + reg_off((8 - pj) & 7);
    unsigned call;

    if (__builtin_expect(pusher, 1)) {
        const unsigned q = b & 7;
        const unsigned dst = ((unsigned)prank + pj) & 7u;
        const rsrc_t rx = mk_rsrc(px + (size_t)dst * MS * KS, MS * KS);
        const unsigned row = lane & 7, c = lane >> 3;
        u32x4 v[32];
        unsigned seqv = 0;
#pragma unroll
        for (int r = 0; r < 4; ++r) {
#pragma unroll
            for (int e = 0; e < 8; ++e) {
                const unsigned unit = 4 * r + (q >> 1);
                const unsigned soff = unit * 16384 + ((q & 1) * 2 + (e >> 2)) * 128 + (e & 3) * 4096;
                v[r * 8 + e] = __builtin_amdgcn_raw_buffer_load_b128(rx, row * 512 + c * 16, soff, 0);
                if (r == 0 && e == 0) {
                    asm volatile("" ::: "memory");
                    seqv = __builtin_amdgcn_raw_buffer_load_b32(ctrl, 0, OFF_SEQ + b * 128, AUX_SYS);
                    asm volatile("" ::: "memory");
                }
            }
        }
        LDS_INIT();
        call = rfl(seqv) + 1;
        const rsrc_t rd = mk_rsrc(xr + REG_HDR + (call & 1) * REG_DATA, REG_DATA);
        const rsrc_t rf = mk_rsrc(xr + HDR_XFLAG, 2048);
#pragma unroll
        for (int s = 0; s < 32 + XWIN; ++s) {
            if (s == BAR_AT) {
                if (lane < 16) lds[LDS_CNT / 4 + lane] = lane == 10 ? call : 0;
                LGKM0();
                __builtin_amdgcn_s_barrier();
            }
            if (s >= XWIN) {
                const int done = s - XWIN;
                int younger = (s < 32 ? s : 32) - 1 - done;
                for (int r = 0; r < 4; ++r) { const int it = 8 * r + 7 + XWIN; if (it < s && it > done) ++younger; }
                VMCNT(younger);
                if ((done & 7) == 7) {
                    const unsigned unit = 4 * (done >> 3) + (q >> 1);
                    if (lane == 0) __builtin_amdgcn_raw_buffer_store_b32(call, rf, 0, unit * 128 + (q & 1) * 4, AUX_SYS);
                }
            }
            if (s < 32) {
                const int r = s >> 3, e = s & 7;
                const unsigned unit = 4 * r + (q >> 1);
                const unsigned soff = unit * 16384 + ((q & 1) * 2 + (e >> 2)) * 4096 + (e & 3) * 128;
                __builtin_amdgcn_raw_buffer_store_b128(v[s], rd, c * 512 + row * 16, soff, AUX_SYS);
            }
        }
        STAMP(2);
    } else if (stager) {
        const unsigned h = (wave - 1) & 1, skb = rb * 2 + ((wave - 1) >> 1);
        const rsrc_t rw = mk_rsrc(pw, KS * NN);
        unsigned d[16];
#pragma unroll
        for (int r = 0; r < 16; ++r)
            d[r] = __builtin_amdgcn_raw_buffer_load_b32(rw, lane * 4, (skb * 32 + h * 16 + r) * NN + g * 256, 0);
        LDS_INIT();
        LGKM0();
        __builtin_amdgcn_s_barrier();
        asm volatile("" ::: "memory");
        call = rfl(lds[LDS_CNT / 4 + 10]);
        u32x4 o[4];
#pragma unroll
        for (int a = 0; a < 4; ++a) {
            const unsigned t0_ = __builtin_amdgcn_perm(d[4 * a + 1], d[4 * a], 0x05010400u);
            const unsigned t1_ = __builtin_amdgcn_perm(d[4 * a + 1], d[4 * a], 0x07030602u);
            const unsigned t2_ = __builtin_amdgcn_perm(d[4 * a + 3], d[4 * a + 2], 0x05010400u);
            const unsigned t3_ = __builtin_amdgcn_perm(d[4 * a + 3], d[4 * a + 2], 0x07030602u);
            o[0][a] = __builtin_amdgcn_perm(t2_, t0_, 0x05040100u);
            o[1][a] = __builtin_amdgcn_perm(t2_, t0_, 0x07060302u);
            o[2][a] = __builtin_amdgcn_perm(t3_, t1_, 0x05040100u);
            o[3][a] = __builtin_amdgcn_perm(t3_, t1_, 0x07060302u);
        }
        STAMP(1);
        const rsrc_t rd = mk_rsrc(myws + OFF_WREG + reg_off(b & 7) + REG_HDR + (call & 1) * REG_DATA + gh * 131072, 131072);
#pragma unroll
        for (int j = 0; j < 4; ++j)
            __builtin_amdgcn_raw_buffer_store_b128(o[j], rd, (lane >> 3) * 16384 + (lane & 7) * 16,
                                                   skb * 1024 + h * 512 + j * 128, AUX_SYS);
        VMCNT(0);
        STAMP(2);
        unsigned tk = 0;
        if (lane == 0)
            tk = __hip_atomic_fetch_add((gu32*)(myws + OFF_CTRL + OFF_WCNT + g * 128), 1u, __ATOMIC_RELAXED, __HIP_MEMORY_SCOPE_AGENT);
        tk = rfl(tk);
        if (tk + 1 == call * 32u) {
            if (lane == 0) {
#pragma unroll
                for (int jj = 1; jj <= NR; ++jj)
                    __builtin_amdgcn_raw_buffer_store_b32(call, mk_rsrc(P.ws[jj & 7] + OFF_XREG + reg_off((8 - jj) & 7) + HDR_WFLAG, 2048),
                                                          0, g * 128, AUX_SYS);
            }
        }
        STAMP(3);
    } else {
        unsigned warm = 0;
        if (warmer) {
            const char* wp = (wave & 1) == 0 ? xr + HDR_WARM : wsj + OFF_WREG + reg_off(b & 7) + HDR_WARM;
            if (wave != 0) __builtin_amdgcn_s_sleep(WARM_DELAY);
            warm = __builtin_amdgcn_raw_buffer_load_b32(mk_rsrc(wp, 4), 0, 0, AUX_SYS);
        }
        unsigned seqv = 0;
        if (wave == 0) seqv = __builtin_amdgcn_raw_buffer_load_b32(ctrl, 0, OFF_SEQ + b * 128, AUX_SYS);
        LDS_INIT();
        if (wave == 0 && lane < 16) lds[LDS_CNT / 4 + lane] = lane == 10 ? seqv + 1 : 0;
        LGKM0();
        __builtin_amdgcn_s_barrier();
        asm volatile("" ::: "memory");
        call = rfl(lds[LDS_CNT / 4 + 10]);
        asm volatile("" :: "v"(warm));
        if (POLL_START && !(wave == 5 && invwg)) __builtin_amdgcn_s_sleep(POLL_START);
        if (wave == 5) {
            const unsigned xcc = __builtin_amdgcn_s_getreg(20 | (3 << 11)) & 15u;
            bool mine = invwg;
            while (!mine) {
                asm volatile("" ::: "memory");
                if (rfl(__builtin_amdgcn_raw_buffer_load_b32(ctrl, 0, OFF_INV + xcc * 128, AUX_SYS)) >= call) break;
                if (__builtin_amdgcn_s_memrealtime() - t0 > 1000ull) mine = true;
                __builtin_amdgcn_s_sleep(POLL_SLEEP);
            }
            if (mine) {
                if (!invwg) __builtin_amdgcn_fence(__ATOMIC_ACQUIRE, "");
                VMCNT(0);
                if (lane == 0) __builtin_amdgcn_raw_buffer_store_b32(call, ctrl, 0, OFF_INV + xcc * 128, AUX_SYS);
            }
            STAMP(1);
            if (lane == 0) __hip_atomic_store(&lds[LDS_CNT / 4 + 8], 1u, __ATOMIC_RELAXED, __HIP_MEMORY_SCOPE_WORKGROUP);
        }
    }

    char* const myx = myws + OFF_XREG + reg_off(wave);
    const rsrc_t hdr = mk_rsrc(myx, REG_HDR);
    const unsigned offw = HDR_WFLAG + g * 128, offx = HDR_XFLAG + rb * 128;
    unsigned st = wait_flags(ctrl, hdr, offw, offx, 3u, call, t0);
    STAMP(4);
    while (rfl(__hip_atomic_load(&lds[LDS_CNT / 4 + 8], __ATOMIC_RELAXED, __HIP_MEMORY_SCOPE_WORKGROUP)) == 0)
        __builtin_amdgcn_s_sleep(1);
    asm volatile("" ::: "memory");
    const rsrc_t rwb = mk_rsrc(wsv + OFF_WREG + reg_off(b & 7) + REG_HDR + (call & 1) * REG_DATA + gh * 131072, 131072);
    const rsrc_t rxb = mk_rsrc(myx + REG_HDR + (call & 1) * REG_DATA + rb * 16384, 16384);
    const unsigned voff = lane * 16;
    const bool remote = wave != 0;
    u32x4 ring[RING];
    u32x4 xf[16];
    if (st & 2u) {
        if (st & 1u) {
            STAMP(13);
#pragma unroll
            for (int f = 0; f < PULL_D; ++f) ring[f] = __builtin_amdgcn_raw_buffer_load_b128(rwb, voff, f * 1024, 0);
        }
        asm volatile("" ::: "memory");
#pragma unroll
        for (int kb = 0; kb < 16; ++kb) xf[kb] = __builtin_amdgcn_raw_buffer_load_b128(rxb, voff, kb * 1024, 0);
        if (!(st & 1u)) {
            st = wait_flags(ctrl, hdr, offw, offx, 1u, call, t0);
            asm volatile("" ::: "memory");
            STAMP(13);
#pragma unroll
            for (int f = 0; f < PULL_D; ++f) ring[f] = __builtin_amdgcn_raw_buffer_load_b128(rwb, voff, f * 1024, 0);
        }
        STAMP(5);
#pragma unroll
        for (int f = PULL_D; f < RING; ++f) {
            VMCNT(PULL_D - 1);
            ring[f] = __builtin_amdgcn_raw_buffer_load_b128(rwb, voff, f * 1024, 0);
        }
    } else {
        STAMP(13);
#pragma unroll
        for (int f = 0; f < RING; ++f) {
            if (f >= PULL_D) VMCNT(PULL_D - 1);
            ring[f] = __builtin_amdgcn_raw_buffer_load_b128(rwb, voff, f * 1024, 0);
        }
        st = wait_flags(ctrl, hdr, offw, offx, 2u, call, t0);
        STAMP(5);
        asm volatile("" ::: "memory");
#pragma unroll
        for (int kb = 0; kb < 16; ++kb) xf[kb] = __builtin_amdgcn_raw_buffer_load_b128(rxb, voff, kb * 1024, 0);
    }
    STAMP(6);
    if ((st & 4u) && lane == 0) __hip_atomic_store(&lds[LDS_CNT / 4 + 9], 1u, __ATOMIC_RELAXED, __HIP_MEMORY_SCOPE_WORKGROUP);

    const unsigned lcol = 4 * (lane & 7) + ((lane >> 3) & 3);
    unsigned* const lacc = &lds[LDS_ACC / 4 + (lane >> 5) * 128 + lcol];

    bool throttle = remote;
    auto tile = [&](unsigned t, int mode) __attribute__((always_inline)) {
        i32x16 acc = {0, 0, 0, 0, 0, 0, 0, 0, 0, 0, 0, 0, 0, 0, 0, 0};
#pragma unroll
        for (int kb = 0; kb < 16; ++kb) {
            const int fl = (mode == 0 ? 0 : mode == 1 ? 96 : 112) + kb + RING;
            const bool more = fl < 128;
            if (more && remote && (kb & (LAG_P - 1)) == 0) {
                const unsigned pos = t * 16 + kb;
                unsigned front = 0;
                if (lane == 0) front = __hip_atomic_fetch_max(&lds[LDS_CNT / 4 + 11], pos, __ATOMIC_RELAXED, __HIP_MEMORY_SCOPE_WORKGROUP);
                throttle = rfl(front) < pos + LAG_T;
            }
            if (more && throttle) VMCNT(PULL_D - 1);
            __builtin_amdgcn_sched_barrier(0);
            acc = __builtin_amdgcn_mfma_i32_32x32x32_i8((i32x4)xf[kb], (i32x4)ring[kb % RING], acc, 0, 0, 0);
            __builtin_amdgcn_sched_barrier(0);
            if (more) ring[kb % RING] = __builtin_amdgcn_raw_buffer_load_b128(rwb, voff, t * 16384 + (kb + RING) * 1024, 0);
            __builtin_amdgcn_sched_barrier(0);
        }
#pragma unroll
        for (int r = 0; r < 16; ++r)
            __hip_atomic_fetch_add(lacc + t * 1024 + ((r & 3) + 8 * (r >> 2)) * 32, (unsigned)acc[r],
                                   __ATOMIC_RELAXED, __HIP_MEMORY_SCOPE_WORKGROUP);
        LGKM0();
        if (lane == 0) __hip_atomic_fetch_add(&lds[LDS_CNT / 4 + t], 1u, __ATOMIC_RELAXED, __HIP_MEMORY_SCOPE_WORKGROUP);
    };
#pragma unroll 1
    for (unsigned t = 0; t < 6; ++t) {
        tile(t, 0);
        if (t == 0) STAMP(7);
        if (t == 3) STAMP(8);
    }
    tile(6, 1);
    tile(7, 2);
    STAMP(9);

    if (wave == 0) {
        if (lane == 0) __builtin_amdgcn_raw_buffer_store_b32(call, ctrl, 0, OFF_SEQ + b * 128, AUX_SYS);
        const float s = P.sx[0] * P.sw[0];
        const rsrc_t ro = mk_rsrc(P.out + (size_t)rb * 32 * NN + g * 256, (31 * NN + 256) * 4);
        const unsigned ooff = (lane >> 3) * (NN * 4) + (lane & 7) * 16;
#pragma unroll 1
        for (unsigned t = 0; t < 8; ++t) {
            while (rfl(__hip_atomic_load(&lds[LDS_CNT / 4 + t], __ATOMIC_RELAXED, __HIP_MEMORY_SCOPE_WORKGROUP)) < 8u) {
                if (t < 7) __builtin_amdgcn_s_sleep(4);
            }
            asm volatile("" ::: "memory");
            if (t == 7) STAMP(10);
            const bool bad = __hip_atomic_load(&lds[LDS_CNT / 4 + 9], __ATOMIC_RELAXED, __HIP_MEMORY_SCOPE_WORKGROUP) != 0;
            i32x4 a[4];
            f32x4 y[4];
#pragma unroll
            for (int q = 0; q < 4; ++q) a[q] = *(const i32x4*)&lds[LDS_ACC / 4 + t * 1024 + q * 256 + lane * 4];
#pragma unroll
            for (int q = 0; q < 4; ++q)
#pragma unroll
                for (int e = 0; e < 4; ++e) y[q][e] = bad ? __builtin_nanf("") : fmaxf((float)a[q][e] * s, 0.0f);
            __builtin_amdgcn_sched_barrier(0);
#pragma unroll
            for (int q = 0; q < 4; ++q) {
                if (LAST_AUX != AUX_SYS && t == 7) __builtin_amdgcn_raw_buffer_store_b128((u32x4)y[q], ro, ooff, q * 8 * (NN * 4) + t * 128, LAST_AUX);
                else __builtin_amdgcn_raw_buffer_store_b128((u32x4)y[q], ro, ooff, q * 8 * (NN * 4) + t * 128, AUX_SYS);
            }
            __builtin_amdgcn_sched_barrier(0);
            asm volatile("s_nop 1");
        }
        STAMP(11);
    }
}

}

extern "C" void kernel_launch(int rank, int nranks, ncclComm_t comm,
                              hipStream_t stream,
                              void* const* d_in, const int* in_sizes, int n_in,
                              void* d_out, int out_size,
                              const int* in_dtypes, int out_dtype,
                              void* const* d_out_all,
                              void* const* d_ws_all, size_t ws_bytes) {
    if (nranks != NR || (unsigned)rank >= (unsigned)NR || n_in < 4 || ws_bytes < WS_NEED ||
        out_size != MS * NN || in_sizes[0] != 4096 * KS || in_sizes[1] != KS * NN) {
        fprintf(stderr, "unexpected problem shape\n");
        return;
    }
    Params P;
    P.x = (const int8_t*)d_in[0];
    P.w = (const int8_t*)d_in[1];
    P.sx = (const float*)d_in[2];
    P.sw = (const float*)d_in[3];
    P.out = (float*)d_out;
    for (int j = 0; j < NR; ++j) P.ws[j] = (char*)d_ws_all[(rank + j) % NR];
    P.rank = rank;
    gemm_rs_fused<<<NWG, 512, 0, stream>>>(P);
    HIP_CHECK(hipGetLastError());
}
```

```cpp
#include <hip/hip_runtime.h>
#include <rccl/rccl.h>
#include <cstdio>
#include <cstdint>

#define HIP_CHECK(expr) do { hipError_t _e = (expr); if (_e != hipSuccess) { \
    fprintf(stderr, "HIP error %s at %s:%d\n", hipGetErrorString(_e), __FILE__, __LINE__); \
    return; } } while (0)

namespace {

typedef int i32x4 __attribute__((ext_vector_type(4)));
typedef unsigned u32x4 __attribute__((ext_vector_type(4)));
typedef unsigned u32x2 __attribute__((ext_vector_type(2)));
typedef int i32x16 __attribute__((ext_vector_type(16)));
typedef float f32x4 __attribute__((ext_vector_type(4)));
typedef __amdgpu_buffer_rsrc_t rsrc_t;
typedef __attribute__((address_space(1))) unsigned gu32;
typedef __attribute__((address_space(1))) unsigned long long gu64;

constexpr int NR = 8;
constexpr int MS = 512, KS = 512;
constexpr int NN = 4096;
constexpr int NWG = 256;

constexpr unsigned REG_HDR = 8192, REG_DATA = 262144, REG_BYTES = REG_HDR + 2 * REG_DATA;
constexpr unsigned HDR_XFLAG = 0, HDR_WFLAG = 2048, HDR_WARM = 4096;
constexpr unsigned OFF_XREG = 0, OFF_WREG = 6u << 20;
__host__ __device__ constexpr unsigned reg_off(unsigned i) {
    return ((i * 11u) >> 5) * (2u << 20) + (i - ((i * 11u) >> 5) * 3u) * REG_BYTES;
}
constexpr unsigned OFF_CTRL  = 0x190000;
constexpr unsigned OFF_SEQ   = 0;
constexpr unsigned OFF_WCNT  = 32768;
constexpr unsigned OFF_TMO   = 36864;
constexpr unsigned OFF_INV   = 40960;
constexpr unsigned CTRL_BYTES = 65536;
constexpr unsigned OFF_DBG   = 16u << 20;
constexpr size_t   WS_NEED   = 18u << 20;
static_assert(3 * REG_BYTES <= OFF_CTRL && OFF_CTRL + CTRL_BYTES <= (2u << 20), "layout");

#define PULL_D 4
#define RING 8
#define LAG_P 2
#define LAG_T 2
#define GATE_X 200
#define GATE_W 400
#define GATE_P 0
#define REMAP 1
#define LAST_AUX AUX_SYS
#define POLL_SLEEP 1
#define POLL_START 0
#define BAR_AT XWIN
#define WARM_DELAY 2
#define XWIN 10
constexpr unsigned long long DEADLINE_TICKS = 100000000ull;

constexpr int AUX_SYS = 17;

struct Params {
    const int8_t* x;
    const int8_t* w;
    const float* sx;
    const float* sw;
    float* out;
    char* ws[NR];
    int rank;
};

#define VMCNT(n) asm volatile("s_waitcnt vmcnt(%0)" :: "i"(n) : "memory")
#define LGKM0() asm volatile("s_waitcnt lgkmcnt(0)" ::: "memory")

__device__ __forceinline__ rsrc_t mk_rsrc(const void* p, unsigned bytes) {
    return __builtin_amdgcn_make_buffer_rsrc(const_cast<void*>(p), 0, bytes, 0x00020000);
}
__device__ __forceinline__ unsigned rfl(unsigned v) { return __builtin_amdgcn_readfirstlane(v); }

#define STAMP(i) do {} while (0)

__device__ __forceinline__ unsigned wait_flags(rsrc_t ctrl, rsrc_t hdr, unsigned offw, unsigned offx, unsigned need,
                                               unsigned call, unsigned long long t0, unsigned wgate) {
    unsigned spins = 0;
    for (;;) {
        asm volatile("" ::: "memory");
        const unsigned wv = __builtin_amdgcn_raw_buffer_load_b32(hdr, 0, offw, AUX_SYS);
        const u32x2 xv = __builtin_amdgcn_raw_buffer_load_b64(hdr, 0, offx, AUX_SYS);
        const unsigned a = rfl(xv.x), c = rfl(xv.y);
        const bool open = GATE_W == 0 || (int)((unsigned)__builtin_amdgcn_s_memrealtime() - wgate) >= 0;
        const unsigned ready = ((rfl(wv) >= call && open) ? 1u : 0u) | ((a >= call && c >= call) ? 2u : 0u);
        if (ready & need) return ready;
        if (++spins > 256) {
            const unsigned tmo = rfl(__builtin_amdgcn_raw_buffer_load_b32(ctrl, 0, OFF_TMO, AUX_SYS));
            if (tmo == call || __builtin_amdgcn_s_memrealtime() - t0 > DEADLINE_TICKS) {
                __builtin_amdgcn_raw_buffer_store_b32(call, ctrl, 0, OFF_TMO, AUX_SYS);
                return 7u;
            }
            __builtin_amdgcn_s_sleep(8);
        }
        __builtin_amdgcn_s_sleep(POLL_SLEEP);
    }
}

constexpr int LDS_ACC = 0;
constexpr int LDS_CNT = 32768;
constexpr int LDS_BYTES = 84 * 1024;

#define LDS_INIT() do { \
    *(u32x4*)&lds[tid * 16] = u32x4{0, 0, 0, 0}; *(u32x4*)&lds[tid * 16 + 4] = u32x4{0, 0, 0, 0}; \
    *(u32x4*)&lds[tid * 16 + 8] = u32x4{0, 0, 0, 0}; *(u32x4*)&lds[tid * 16 + 12] = u32x4{0, 0, 0, 0}; } while (0)

__global__ void __launch_bounds__(512, 2) gemm_rs_fused(Params P) {
    __shared__ __attribute__((aligned(16))) unsigned lds[LDS_BYTES / 4];
    const int tid = threadIdx.x, lane = tid & 63;
    const unsigned wave = rfl(tid >> 6);
    const unsigned b = blockIdx.x;
    const unsigned idx = b >> 3;
    const unsigned gh = idx & 1;
    const unsigned rb = 15 - (idx >> 1);
    const bool invwg = idx == 8;
    const unsigned g = (b & 7) * 2 + gh;
    if (wave == 5 && invwg) { __builtin_amdgcn_fence(__ATOMIC_ACQUIRE, ""); }
    const unsigned pj = gh * 4 + (rb & 3);
    unsigned long long t0 = __builtin_amdgcn_s_memrealtime();
    char* myws = P.ws[0];
    char* wsv = P.ws[wave];
    char* wsj = P.ws[pj];
    const int8_t* px = P.x;
    const int8_t* pw = P.w;
    int prank = P.rank;
    asm volatile("" : "+s"(myws), "+s"(wsv), "+s"(wsj), "+s"(px), "+s"(pw), "+s"(prank), "+s"(t0));
    const rsrc_t ctrl = mk_rsrc(myws + OFF_CTRL, CTRL_BYTES);

    const bool pusher = (wave == 0) && (rb >= 12);
    const bool stager = (wave >= 1) && (wave <= 4) && (rb < 8);
    const bool warmer = rb >= 8 && rb < 12 && wave < 2;
    char* const xr = wsj + OFF_XREG + reg_off((8 - pj) & 7);
    unsigned call, tbar;

    if (__builtin_expect(pusher, 1)) {
        const unsigned q = b & 7;
        const unsigned dst = ((unsigned)prank + pj) & 7u;
        const rsrc_t rx = mk_rsrc(px + (size_t)dst * MS * KS, MS * KS);
        const unsigned row = lane & 7, c = lane >> 3;
        u32x4 v[32];
        unsigned seqv = 0;
#pragma unroll
        for (int r = 0; r < 4; ++r) {
#pragma unroll
            for (int e = 0; e < 8; ++e) {
                const unsigned unit = 4 * r + (q >> 1);
                const unsigned soff = unit * 16384 + ((q & 1) * 2 + (e >> 2)) * 128 + (e & 3) * 4096;
                v[r * 8 + e] = __builtin_amdgcn_raw_buffer_load_b128(rx, row * 512 + c * 16, soff, 0);
                if (r == 0 && e == 0) {
                    asm volatile("" ::: "memory");
                    seqv = __builtin_amdgcn_raw_buffer_load_b32(ctrl, 0, OFF_SEQ + b * 128, AUX_SYS);
                    asm volatile("" ::: "memory");
                }
            }
        }
        LDS_INIT();
        call = rfl(seqv) + 1;
        tbar = (unsigned)t0;
        const rsrc_t rd = mk_rsrc(xr + REG_HDR + (call & 1) * REG_DATA, REG_DATA);
        const rsrc_t rf = mk_rsrc(xr + HDR_XFLAG, 2048);
#pragma unroll
        for (int s = 0; s < 32 + XWIN; ++s) {
            if (s == BAR_AT) {
                if (lane < 16) lds[LDS_CNT / 4 + lane] = lane == 10 ? call : 0;
                LGKM0();
                __builtin_amdgcn_s_barrier();
            }
            if (s >= XWIN) {
                const int done = s - XWIN;
                int younger = (s < 32 ? s : 32) - 1 - done;
                for (int r = 0; r < 4; ++r) { const int it = 8 * r + 7 + XWIN; if (it < s && it > done) ++younger; }
                VMCNT(younger);
                if ((done & 7) == 7) {
                    const unsigned unit = 4 * (done >> 3) + (q >> 1);
                    if (lane == 0) __builtin_amdgcn_raw_buffer_store_b32(call, rf, 0, unit * 128 + (q & 1) * 4, AUX_SYS);
                }
            }
            if (s < 32) {
                const int r = s >> 3, e = s & 7;
                const unsigned unit = 4 * r + (q >> 1);
                const unsigned soff = unit * 16384 + ((q & 1) * 2 + (e >> 2)) * 4096 + (e & 3) * 128;
                __builtin_amdgcn_raw_buffer_store_b128(v[s], rd, c * 512 + row * 16, soff, AUX_SYS);
            }
        }
        STAMP(2);
    } else if (stager) {
        const unsigned h = (wave - 1) & 1, skb = rb * 2 + ((wave - 1) >> 1);
        const rsrc_t rw = mk_rsrc(pw, KS * NN);
        unsigned d[16];
#pragma unroll
        for (int r = 0; r < 16; ++r)
            d[r] = __builtin_amdgcn_raw_buffer_load_b32(rw, lane * 4, (skb * 32 + h * 16 + r) * NN + g * 256, 0);
        LDS_INIT();
        LGKM0();
        __builtin_amdgcn_s_barrier();
        asm volatile("" ::: "memory");
        call = rfl(lds[LDS_CNT / 4 + 10]);
        tbar = (unsigned)__builtin_amdgcn_s_memrealtime();
        u32x4 o[4];
#pragma unroll
        for (int a = 0; a < 4; ++a) {
            const unsigned t0_ = __builtin_amdgcn_perm(d[4 * a + 1], d[4 * a], 0x05010400u);
            const unsigned t1_ = __builtin_amdgcn_perm(d[4 * a + 1], d[4 * a], 0x07030602u);
            const unsigned t2_ = __builtin_amdgcn_perm(d[4 * a + 3], d[4 * a + 2], 0x05010400u);
            const unsigned t3_ = __builtin_amdgcn_perm(d[4 * a + 3], d[4 * a + 2], 0x07030602u);
            o[0][a] = __builtin_amdgcn_perm(t2_, t0_, 0x05040100u);
            o[1][a] = __builtin_amdgcn_perm(t2_, t0_, 0x07060302u);
            o[2][a] = __builtin_amdgcn_perm(t3_, t1_, 0x05040100u);
            o[3][a] = __builtin_amdgcn_perm(t3_, t1_, 0x07060302u);
        }
        STAMP(1);
        const rsrc_t rd = mk_rsrc(myws + OFF_WREG + reg_off(b & 7) + REG_HDR + (call & 1) * REG_DATA + gh * 131072, 131072);
#pragma unroll
        for (int j = 0; j < 4; ++j)
            __builtin_amdgcn_raw_buffer_store_b128(o[j], rd, (lane >> 3) * 16384 + (lane & 7) * 16,
                                                   skb * 1024 + h * 512 + j * 128, AUX_SYS);
        VMCNT(0);
        STAMP(2);
        unsigned tk = 0;
        if (lane == 0)
            tk = __hip_atomic_fetch_add((gu32*)(myws + OFF_CTRL + OFF_WCNT + g * 128), 1u, __ATOMIC_RELAXED, __HIP_MEMORY_SCOPE_AGENT);
        tk = rfl(tk);
        if (tk + 1 == call * 32u) {
            if (lane == 0) {
#pragma unroll
                for (int jj = 1; jj <= NR; ++jj)
                    __builtin_amdgcn_raw_buffer_store_b32(call, mk_rsrc(P.ws[jj & 7] + OFF_XREG + reg_off((8 - jj) & 7) + HDR_WFLAG, 2048),
                                                          0, g * 128, AUX_SYS);
            }
        }
        STAMP(3);
    } else {
        unsigned warm = 0;
        if (warmer) {
            const char* wp = (wave & 1) == 0 ? xr + HDR_WARM : wsj + OFF_WREG + reg_off(b & 7) + HDR_WARM;
            if (wave != 0) __builtin_amdgcn_s_sleep(WARM_DELAY);
            warm = __builtin_amdgcn_raw_buffer_load_b32(mk_rsrc(wp, 4), 0, 0, AUX_SYS);
        }
        unsigned seqv = 0;
        if (wave == 0) seqv = __builtin_amdgcn_raw_buffer_load_b32(ctrl, 0, OFF_SEQ + b * 128, AUX_SYS);
        LDS_INIT();
        if (wave == 0 && lane < 16) lds[LDS_CNT / 4 + lane] = lane == 10 ? seqv + 1 : 0;
        LGKM0();
        __builtin_amdgcn_s_barrier();
        asm volatile("" ::: "memory");
        call = rfl(lds[LDS_CNT / 4 + 10]);
        tbar = (unsigned)__builtin_amdgcn_s_memrealtime();
        asm volatile("" :: "v"(warm));
        if (POLL_START && !(wave == 5 && invwg)) __builtin_amdgcn_s_sleep(POLL_START);
        if (wave == 5) {
            const unsigned xcc = __builtin_amdgcn_s_getreg(20 | (3 << 11)) & 15u;
            bool mine = invwg;
            while (!mine) {
                asm volatile("" ::: "memory");
                if (rfl(__builtin_amdgcn_raw_buffer_load_b32(ctrl, 0, OFF_INV + xcc * 128, AUX_SYS)) >= call) break;
                if (__builtin_amdgcn_s_memrealtime() - t0 > 1000ull) mine = true;
                __builtin_amdgcn_s_sleep(POLL_SLEEP);
            }
            if (mine) {
                if (!invwg) __builtin_amdgcn_fence(__ATOMIC_ACQUIRE, "");
                VMCNT(0);
                if (lane == 0) __builtin_amdgcn_raw_buffer_store_b32(call, ctrl, 0, OFF_INV + xcc * 128, AUX_SYS);
            }
            STAMP(1);
            if (lane == 0) __hip_atomic_store(&lds[LDS_CNT / 4 + 8], 1u, __ATOMIC_RELAXED, __HIP_MEMORY_SCOPE_WORKGROUP);
        }
    }

    char* const myx = myws + OFF_XREG + reg_off(wave);
    const rsrc_t hdr = mk_rsrc(myx, REG_HDR);
    const unsigned offw = HDR_WFLAG + g * 128, offx = HDR_XFLAG + rb * 128;
    while ((unsigned)__builtin_amdgcn_s_memrealtime() - tbar < (unsigned)GATE_X) __builtin_amdgcn_s_sleep(2);
    unsigned st = wait_flags(ctrl, hdr, offw, offx, 3u, call, t0, tbar + (wave ? GATE_W : 0));
    STAMP(4);
    while (rfl(__hip_atomic_load(&lds[LDS_CNT / 4 + 8], __ATOMIC_RELAXED, __HIP_MEMORY_SCOPE_WORKGROUP)) == 0)
        __builtin_amdgcn_s_sleep(1);
    asm volatile("" ::: "memory");
    const rsrc_t rwb = mk_rsrc(wsv + OFF_WREG + reg_off(b & 7) + REG_HDR + (call & 1) * REG_DATA + gh * 131072, 131072);
    const rsrc_t rxb = mk_rsrc(myx + REG_HDR + (call & 1) * REG_DATA + rb * 16384, 16384);
    const unsigned voff = lane * 16;
    const bool remote = wave != 0;
    u32x4 ring[RING];
    u32x4 xf[16];
    if (st & 2u) {
        if (st & 1u) {
            STAMP(13);
#pragma unroll
            for (int f = 0; f < PULL_D; ++f) ring[f] = __builtin_amdgcn_raw_buffer_load_b128(rwb, voff, f * 1024, 0);
        }
        asm volatile("" ::: "memory");
#pragma unroll
        for (int kb = 0; kb < 16; ++kb) xf[kb] = __builtin_amdgcn_raw_buffer_load_b128(rxb, voff, kb * 1024, 0);
        if (!(st & 1u)) {
            st = wait_flags(ctrl, hdr, offw, offx, 1u, call, t0, tbar + (wave ? GATE_W : 0));
            asm volatile("" ::: "memory");
            STAMP(13);
#pragma unroll
            for (int f = 0; f < PULL_D; ++f) ring[f] = __builtin_amdgcn_raw_buffer_load_b128(rwb, voff, f * 1024, 0);
        }
        STAMP(5);
#pragma unroll
        for (int f = PULL_D; f < RING; ++f) {
            VMCNT(PULL_D - 1);
            ring[f] = __builtin_amdgcn_raw_buffer_load_b128(rwb, voff, f * 1024, 0);
        }
    } else {
        STAMP(13);
#pragma unroll
        for (int f = 0; f < RING; ++f) {
            if (f >= PULL_D) VMCNT(PULL_D - 1);
            ring[f] = __builtin_amdgcn_raw_buffer_load_b128(rwb, voff, f * 1024, 0);
        }
        st = wait_flags(ctrl, hdr, offw, offx, 2u, call, t0, tbar);
        STAMP(5);
        asm volatile("" ::: "memory");
#pragma unroll
        for (int kb = 0; kb < 16; ++kb) xf[kb] = __builtin_amdgcn_raw_buffer_load_b128(rxb, voff, kb * 1024, 0);
    }
    STAMP(6);
    if ((st & 4u) && lane == 0) __hip_atomic_store(&lds[LDS_CNT / 4 + 9], 1u, __ATOMIC_RELAXED, __HIP_MEMORY_SCOPE_WORKGROUP);

    const unsigned lcol = 4 * (lane & 7) + ((lane >> 3) & 3);
    unsigned* const lacc = &lds[LDS_ACC / 4 + (lane >> 5) * 128 + lcol];

    bool throttle = remote;
    auto tile = [&](unsigned t, int mode) __attribute__((always_inline)) {
        i32x16 acc = {0, 0, 0, 0, 0, 0, 0, 0, 0, 0, 0, 0, 0, 0, 0, 0};
#pragma unroll
        for (int kb = 0; kb < 16; ++kb) {
            const int fl = (mode == 0 ? 0 : mode == 1 ? 96 : 112) + kb + RING;
            const bool more = fl < 128;
            if (more && remote && (kb & (LAG_P - 1)) == 0) {
                const unsigned pos = t * 16 + kb;
                unsigned front = 0;
                if (lane == 0) front = __hip_atomic_fetch_max(&lds[LDS_CNT / 4 + 11], pos, __ATOMIC_RELAXED, __HIP_MEMORY_SCOPE_WORKGROUP);
                throttle = rfl(front) < pos + LAG_T;
            }
            if (more && throttle) VMCNT(PULL_D - 1);
            __builtin_amdgcn_sched_barrier(0);
            acc = __builtin_amdgcn_mfma_i32_32x32x32_i8((i32x4)xf[kb], (i32x4)ring[kb % RING], acc, 0, 0, 0);
            __builtin_amdgcn_sched_barrier(0);
            if (more) ring[kb % RING] = __builtin_amdgcn_raw_buffer_load_b128(rwb, voff, t * 16384 + (kb + RING) * 1024, 0);
            __builtin_amdgcn_sched_barrier(0);
        }
#pragma unroll
        for (int r = 0; r < 16; ++r)
            __hip_atomic_fetch_add(lacc + t * 1024 + ((r & 3) + 8 * (r >> 2)) * 32, (unsigned)acc[r],
                                   __ATOMIC_RELAXED, __HIP_MEMORY_SCOPE_WORKGROUP);
        LGKM0();
        if (lane == 0) __hip_atomic_fetch_add(&lds[LDS_CNT / 4 + t], 1u, __ATOMIC_RELAXED, __HIP_MEMORY_SCOPE_WORKGROUP);
    };
#pragma unroll 1
    for (unsigned t = 0; t < 6; ++t) {
        tile(t, 0);
        if (t == 0) STAMP(7);
        if (t == 3) STAMP(8);
    }
    tile(6, 1);
    tile(7, 2);
    STAMP(9);

    if (wave == 0) {
        if (lane == 0) __builtin_amdgcn_raw_buffer_store_b32(call, ctrl, 0, OFF_SEQ + b * 128, AUX_SYS);
        const float s = P.sx[0] * P.sw[0];
        const rsrc_t ro = mk_rsrc(P.out + (size_t)rb * 32 * NN + g * 256, (31 * NN + 256) * 4);
        const unsigned ooff = (lane >> 3) * (NN * 4) + (lane & 7) * 16;
#pragma unroll 1
        for (unsigned t = 0; t < 8; ++t) {
            while (rfl(__hip_atomic_load(&lds[LDS_CNT / 4 + t], __ATOMIC_RELAXED, __HIP_MEMORY_SCOPE_WORKGROUP)) < 8u) {
                if (t < 7) __builtin_amdgcn_s_sleep(4);
            }
            asm volatile("" ::: "memory");
            if (t == 7) STAMP(10);
            const bool bad = __hip_atomic_load(&lds[LDS_CNT / 4 + 9], __ATOMIC_RELAXED, __HIP_MEMORY_SCOPE_WORKGROUP) != 0;
            i32x4 a[4];
            f32x4 y[4];
#pragma unroll
            for (int q = 0; q < 4; ++q) a[q] = *(const i32x4*)&lds[LDS_ACC / 4 + t * 1024 + q * 256 + lane * 4];
#pragma unroll
            for (int q = 0; q < 4; ++q)
#pragma unroll
                for (int e = 0; e < 4; ++e) y[q][e] = bad ? __builtin_nanf("") : fmaxf((float)a[q][e] * s, 0.0f);
            __builtin_amdgcn_sched_barrier(0);
#pragma unroll
            for (int q = 0; q < 4; ++q) {
                if (LAST_AUX != AUX_SYS && t == 7) __builtin_amdgcn_raw_buffer_store_b128((u32x4)y[q], ro, ooff, q * 8 * (NN * 4) + t * 128, LAST_AUX);
                else __builtin_amdgcn_raw_buffer_store_b128((u32x4)y[q], ro, ooff, q * 8 * (NN * 4) + t * 128, AUX_SYS);
            }
            __builtin_amdgcn_sched_barrier(0);
            asm volatile("s_nop 1");
        }
        STAMP(11);
    }
}

}

extern "C" void kernel_launch(int rank, int nranks, ncclComm_t comm,
                              hipStream_t stream,
                              void* const* d_in, const int* in_sizes, int n_in,
                              void* d_out, int out_size,
                              const int* in_dtypes, int out_dtype,
                              void* const* d_out_all,
                              void* const* d_ws_all, size_t ws_bytes) {
    if (nranks != NR || (unsigned)rank >= (unsigned)NR || n_in < 4 || ws_bytes < WS_NEED ||
        out_size != MS * NN || in_sizes[0] != 4096 * KS || in_sizes[1] != KS * NN) {
        fprintf(stderr, "unexpected problem shape\n");
        return;
    }
    Params P;
    P.x = (const int8_t*)d_in[0];
    P.w = (const int8_t*)d_in[1];
    P.sx = (const float*)d_in[2];
    P.sw = (const float*)d_in[3];
    P.out = (float*)d_out;
    for (int j = 0; j < NR; ++j) P.ws[j] = (char*)d_ws_all[(rank + j) % NR];
    P.rank = rank;
    gemm_rs_fused<<<NWG, 512, 0, stream>>>(P);
    HIP_CHECK(hipGetLastError());
}
```

```cpp
#include <hip/hip_runtime.h>
#include <rccl/rccl.h>
#include <cstdio>
#include <cstdint>

#define HIP_CHECK(expr) do { hipError_t _e = (expr); if (_e != hipSuccess) { \
    fprintf(stderr, "HIP error %s at %s:%d\n", hipGetErrorString(_e), __FILE__, __LINE__); \
    return; } } while (0)

namespace {

typedef int i32x4 __attribute__((ext_vector_type(4)));
typedef unsigned u32x4 __attribute__((ext_vector_type(4)));
typedef unsigned u32x2 __attribute__((ext_vector_type(2)));
typedef int i32x16 __attribute__((ext_vector_type(16)));
typedef float f32x4 __attribute__((ext_vector_type(4)));
typedef __amdgpu_buffer_rsrc_t rsrc_t;
typedef __attribute__((address_space(1))) unsigned gu32;
typedef __attribute__((address_space(1))) unsigned long long gu64;

constexpr int NR = 8;
constexpr int MS = 512, KS = 512;
constexpr int NN = 4096;
constexpr int NWG = 256;

constexpr unsigned REG_HDR = 8192, REG_DATA = 262144, REG_BYTES = REG_HDR + 2 * REG_DATA;
constexpr unsigned HDR_XFLAG = 0, HDR_WFLAG = 2048, HDR_WARM = 4096;
constexpr unsigned OFF_XREG = 0, OFF_WREG = 6u << 20;
__host__ __device__ constexpr unsigned reg_off(unsigned i) {
    return ((i * 11u) >> 5) * (2u << 20) + (i - ((i * 11u) >> 5) * 3u) * REG_BYTES;
}
constexpr unsigned OFF_CTRL  = 0x190000;
constexpr unsigned OFF_SEQ   = 0;
constexpr unsigned OFF_WCNT  = 32768;
constexpr unsigned OFF_TMO   = 36864;
constexpr unsigned OFF_INV   = 40960;
constexpr unsigned OFF_PROG  = 45056;
constexpr unsigned CTRL_BYTES = 65536;
constexpr unsigned OFF_DBG   = 16u << 20;
constexpr size_t   WS_NEED   = 18u << 20;
static_assert(3 * REG_BYTES <= OFF_CTRL && OFF_CTRL + CTRL_BYTES <= (2u << 20), "layout");

#define PULL_D 4
#define RING 8
#define LAG_P 2
#define LAG_T 2
#define BAL_T 30
#define BAL_POS 40
#define BAL_TAU 438
#define BOOST_D 5
#define SLOW_D 3
#define STAGE_SEQ 0
#define GATE_XR 100
#define GATE_XM 30
#define INV_SLEEP 1
#define GATE_X 200
#define GATE_W 400
#define GATE_P 0
#define REMAP 1
#define POLL_SLEEP 1
#define POLL_START 0
#define BAR_AT XWIN
#define WARM_DELAY 2
#define XWIN 10
constexpr unsigned long long DEADLINE_TICKS = 100000000ull;

constexpr int AUX_SYS = 17;

struct Params {
    const int8_t* x;
    const int8_t* w;
    const float* sx;
    const float* sw;
    float* out;
    char* ws[NR];
    int rank;
};

#define VMCNT(n) asm volatile("s_waitcnt vmcnt(%0)" :: "i"(n) : "memory")
#define LGKM0() asm volatile("s_waitcnt lgkmcnt(0)" ::: "memory")

__device__ __forceinline__ rsrc_t mk_rsrc(const void* p, unsigned bytes) {
    return __builtin_amdgcn_make_buffer_rsrc(const_cast<void*>(p), 0, bytes, 0x00020000);
}
__device__ __forceinline__ unsigned rfl(unsigned v) { return __builtin_amdgcn_readfirstlane(v); }

#define STAMP(i) do {} while (0)

__device__ __forceinline__ unsigned wait_flags(rsrc_t ctrl, rsrc_t hdr, unsigned offw, unsigned offx, unsigned need,
                                               unsigned call, unsigned long long t0, unsigned wgate) {
    unsigned spins = 0;
    for (;;) {
        asm volatile("" ::: "memory");
        const unsigned wv = __builtin_amdgcn_raw_buffer_load_b32(hdr, 0, offw, AUX_SYS);
        const u32x2 xv = __builtin_amdgcn_raw_buffer_load_b64(hdr, 0, offx, AUX_SYS);
        const unsigned a = rfl(xv.x), c = rfl(xv.y);
        const bool open = GATE_W == 0 || (int)((unsigned)__builtin_amdgcn_s_memrealtime() - wgate) >= 0;
        const unsigned ready = ((rfl(wv) >= call && open) ? 1u : 0u) | ((a >= call && c >= call) ? 2u : 0u);
        if (ready & need) return ready;
        if (++spins > 256) {
            const unsigned tmo = rfl(__builtin_amdgcn_raw_buffer_load_b32(ctrl, 0, OFF_TMO, AUX_SYS));
            if (tmo == call || __builtin_amdgcn_s_memrealtime() - t0 > DEADLINE_TICKS) {
                __builtin_amdgcn_raw_buffer_store_b32(call, ctrl, 0, OFF_TMO, AUX_SYS);
                return 7u;
            }
            __builtin_amdgcn_s_sleep(8);
        }
        __builtin_amdgcn_s_sleep(POLL_SLEEP);
    }
}

constexpr int LDS_ACC = 0;
constexpr int LDS_CNT = 32768;
constexpr int LDS_BYTES = 84 * 1024;

#define LDS_INIT() do { \
    *(u32x4*)&lds[tid * 16] = u32x4{0, 0, 0, 0}; *(u32x4*)&lds[tid * 16 + 4] = u32x4{0, 0, 0, 0}; \
    *(u32x4*)&lds[tid * 16 + 8] = u32x4{0, 0, 0, 0}; *(u32x4*)&lds[tid * 16 + 12] = u32x4{0, 0, 0, 0}; } while (0)

__global__ void __launch_bounds__(512, 2) gemm_rs_fused(Params P) {
    __shared__ __attribute__((aligned(16))) unsigned lds[LDS_BYTES / 4];
    const int tid = threadIdx.x, lane = tid & 63;
    const unsigned wave = rfl(tid >> 6);
    const unsigned b = blockIdx.x;
    const unsigned idx = b >> 3;
    const unsigned gh = idx & 1;
    const unsigned rb = 15 - (idx >> 1);
    const bool invwg = idx == 8;
    const unsigned g = (b & 7) * 2 + gh;
    if (wave == 5 && invwg) { __builtin_amdgcn_fence(__ATOMIC_ACQUIRE, ""); }
    const unsigned pj = gh * 4 + (rb & 3);
    unsigned long long t0 = __builtin_amdgcn_s_memrealtime();
    char* myws = P.ws[0];
    char* wsv = P.ws[wave];
    char* wsj = P.ws[pj];
    const int8_t* px = P.x;
    const int8_t* pw = P.w;
    int prank = P.rank;
    asm volatile("" : "+s"(myws), "+s"(wsv), "+s"(wsj), "+s"(px), "+s"(pw), "+s"(prank), "+s"(t0));
    const rsrc_t ctrl = mk_rsrc(myws + OFF_CTRL, CTRL_BYTES);

    const bool pusher = (wave == 0) && (rb >= 12);
    const bool stager = (wave >= 1) && (wave <= 4) && (rb < 8);
    const bool warmer = rb >= 8 && rb < 12 && wave < 2;
    char* const xr = wsj + OFF_XREG + reg_off((8 - pj) & 7);
    unsigned call, tbar;

    if (__builtin_expect(pusher, 1)) {
        const unsigned q = b & 7;
        const unsigned dst = ((unsigned)prank + pj) & 7u;
        const rsrc_t rx = mk_rsrc(px + (size_t)dst * MS * KS, MS * KS);
        const unsigned row = lane & 7, c = lane >> 3;
        u32x4 v[32];
        unsigned seqv = 0;
#pragma unroll
        for (int r = 0; r < 4; ++r) {
#pragma unroll
            for (int e = 0; e < 8; ++e) {
                const unsigned unit = 4 * r + (q >> 1);
                const unsigned soff = unit * 16384 + ((q & 1) * 2 + (e >> 2)) * 128 + (e & 3) * 4096;
                v[r * 8 + e] = __builtin_amdgcn_raw_buffer_load_b128(rx, row * 512 + c * 16, soff, 0);
                if (r == 0 && e == 0) {
                    asm volatile("" ::: "memory");
                    seqv = __builtin_amdgcn_raw_buffer_load_b32(ctrl, 0, OFF_SEQ + b * 128, AUX_SYS);
                    asm volatile("" ::: "memory");
                }
            }
        }
        LDS_INIT();
        call = rfl(seqv) + 1;
        tbar = (unsigned)t0;
        const rsrc_t rd = mk_rsrc(xr + REG_HDR + (call & 1) * REG_DATA, REG_DATA);
        const rsrc_t rf = mk_rsrc(xr + HDR_XFLAG, 2048);
#pragma unroll
        for (int s = 0; s < 32 + XWIN; ++s) {
            if (s == BAR_AT) {
                if (lane < 16) lds[LDS_CNT / 4 + lane] = lane == 10 ? call : 0;
                LGKM0();
                __builtin_amdgcn_s_barrier();
            }
            if (s >= XWIN) {
                const int done = s - XWIN;
                int younger = (s < 32 ? s : 32) - 1 - done;
                for (int r = 0; r < 4; ++r) { const int it = 8 * r + 7 + XWIN; if (it < s && it > done) ++younger; }
                VMCNT(younger);
                if ((done & 7) == 7) {
                    const unsigned unit = 4 * (done >> 3) + (q >> 1);
                    if (lane == 0) __builtin_amdgcn_raw_buffer_store_b32(call, rf, 0, unit * 128 + (q & 1) * 4, AUX_SYS);
                }
            }
            if (s < 32) {
                const int r = s >> 3, e = s & 7;
                const unsigned unit = 4 * r + (q >> 1);
                const unsigned soff = unit * 16384 + ((q & 1) * 2 + (e >> 2)) * 4096 + (e & 3) * 128;
                __builtin_amdgcn_raw_buffer_store_b128(v[s], rd, c * 512 + row * 16, soff, AUX_SYS);
            }
        }
        STAMP(2);
    } else if (stager) {
        const unsigned h = (wave - 1) & 1, skb = rb * 2 + ((wave - 1) >> 1);
        const rsrc_t rw = mk_rsrc(pw, KS * NN);
        unsigned d[16];
#pragma unroll
        for (int r = 0; r < 16; ++r)
            d[r] = __builtin_amdgcn_raw_buffer_load_b32(rw, lane * 4, (skb * 32 + h * 16 + r) * NN + g * 256, 0);
        LDS_INIT();
        LGKM0();
        __builtin_amdgcn_s_barrier();
        asm volatile("" ::: "memory");
        call = rfl(lds[LDS_CNT / 4 + 10]);
        tbar = (unsigned)__builtin_amdgcn_s_memrealtime();
        u32x4 o[4];
#pragma unroll
        for (int a = 0; a < 4; ++a) {
            const unsigned t0_ = __builtin_amdgcn_perm(d[4 * a + 1], d[4 * a], 0x05010400u);
            const unsigned t1_ = __builtin_amdgcn_perm(d[4 * a + 1], d[4 * a], 0x07030602u);
            const unsigned t2_ = __builtin_amdgcn_perm(d[4 * a + 3], d[4 * a + 2], 0x05010400u);
            const unsigned t3_ = __builtin_amdgcn_perm(d[4 * a + 3], d[4 * a + 2], 0x07030602u);
            o[0][a] = __builtin_amdgcn_perm(t2_, t0_, 0x05040100u);
            o[1][a] = __builtin_amdgcn_perm(t2_, t0_, 0x07060302u);
            o[2][a] = __builtin_amdgcn_perm(t3_, t1_, 0x05040100u);
            o[3][a] = __builtin_amdgcn_perm(t3_, t1_, 0x07060302u);
        }
        STAMP(1);
        const rsrc_t rd = mk_rsrc(myws + OFF_WREG + reg_off(b & 7) + REG_HDR + (call & 1) * REG_DATA + gh * 131072, 131072);
#pragma unroll
        for (int j = 0; j < 4; ++j)
            __builtin_amdgcn_raw_buffer_store_b128(o[j], rd, (lane >> 3) * 16384 + (lane & 7) * 16,
                                                   skb * 1024 + h * 512 + j * 128, AUX_SYS);
        VMCNT(0);
        STAMP(2);
        unsigned tk = 0;
        if (lane == 0)
            tk = __hip_atomic_fetch_add((gu32*)(myws + OFF_CTRL + OFF_WCNT + g * 128), 1u, __ATOMIC_RELAXED, __HIP_MEMORY_SCOPE_AGENT);
        tk = rfl(tk);
        if (tk + 1 == call * 32u) {
            if (lane == 0) {
#pragma unroll
                for (int jj = 1; jj <= NR; ++jj)
                    __builtin_amdgcn_raw_buffer_store_b32(call, mk_rsrc(P.ws[jj & 7] + OFF_XREG + reg_off((8 - jj) & 7) + HDR_WFLAG, 2048),
                                                          0, g * 128, AUX_SYS);
            }
        }
        STAMP(3);
    } else {
        unsigned warm = 0;
        if (warmer) {
            const char* wp = (wave & 1) == 0 ? xr + HDR_WARM : wsj + OFF_WREG + reg_off(b & 7) + HDR_WARM;
            if (wave != 0) __builtin_amdgcn_s_sleep(WARM_DELAY);
            warm = __builtin_amdgcn_raw_buffer_load_b32(mk_rsrc(wp, 4), 0, 0, AUX_SYS);
        }
        unsigned seqv = 0;
        if (wave == 0) seqv = __builtin_amdgcn_raw_buffer_load_b32(ctrl, 0, OFF_SEQ + b * 128, AUX_SYS);
        LDS_INIT();
        if (wave == 0 && lane < 16) lds[LDS_CNT / 4 + lane] = lane == 10 ? seqv + 1 : 0;
        LGKM0();
        __builtin_amdgcn_s_barrier();
        asm volatile("" ::: "memory");
        call = rfl(lds[LDS_CNT / 4 + 10]);
        tbar = (unsigned)__builtin_amdgcn_s_memrealtime();
        asm volatile("" :: "v"(warm));
        if (POLL_START && !(wave == 5 && invwg)) __builtin_amdgcn_s_sleep(POLL_START);
        if (wave == 5) {
            const unsigned xcc = __builtin_amdgcn_s_getreg(20 | (3 << 11)) & 15u;
            bool mine = invwg;
            while (!mine) {
                asm volatile("" ::: "memory");
                if (rfl(__builtin_amdgcn_raw_buffer_load_b32(ctrl, 0, OFF_INV + xcc * 128, AUX_SYS)) >= call) break;
                if (__builtin_amdgcn_s_memrealtime() - t0 > 1000ull) mine = true;
                __builtin_amdgcn_s_sleep(INV_SLEEP);
            }
            if (mine) {
                if (!invwg) __builtin_amdgcn_fence(__ATOMIC_ACQUIRE, "");
                VMCNT(0);
                if (lane == 0) __builtin_amdgcn_raw_buffer_store_b32(call, ctrl, 0, OFF_INV + xcc * 128, AUX_SYS);
            }
            STAMP(1);
            if (lane == 0) __hip_atomic_store(&lds[LDS_CNT / 4 + 8], 1u, __ATOMIC_RELAXED, __HIP_MEMORY_SCOPE_WORKGROUP);
        }
    }

    char* const myx = myws + OFF_XREG + reg_off(wave);
    const rsrc_t hdr = mk_rsrc(myx, REG_HDR);
    const unsigned offw = HDR_WFLAG + g * 128, offx = HDR_XFLAG + rb * 128;
    {
        unsigned gx = GATE_X + (wave ? (rb >> 2) * GATE_XR : 0);
        if (gx > GATE_W - GATE_XM) gx = GATE_W - GATE_XM;
        while ((unsigned)__builtin_amdgcn_s_memrealtime() - tbar < gx) __builtin_amdgcn_s_sleep(2);
    }
    unsigned st = wait_flags(ctrl, hdr, offw, offx, 3u, call, t0, tbar + (wave ? GATE_W : 0));
    STAMP(4);
    while (rfl(__hip_atomic_load(&lds[LDS_CNT / 4 + 8], __ATOMIC_RELAXED, __HIP_MEMORY_SCOPE_WORKGROUP)) == 0)
        __builtin_amdgcn_s_sleep(1);
    asm volatile("" ::: "memory");
    const rsrc_t rwb = mk_rsrc(wsv + OFF_WREG + reg_off(b & 7) + REG_HDR + (call & 1) * REG_DATA + gh * 131072, 131072);
    const rsrc_t rxb = mk_rsrc(myx + REG_HDR + (call & 1) * REG_DATA + rb * 16384, 16384);
    const unsigned voff = lane * 16;
    const bool remote = wave != 0;
    u32x4 ring[RING];
    u32x4 xf[16];
    if (st & 2u) {
        if (st & 1u) {
            STAMP(13);
#pragma unroll
            for (int f = 0; f < PULL_D; ++f) ring[f] = __builtin_amdgcn_raw_buffer_load_b128(rwb, voff, f * 1024, 0);
        }
        asm volatile("" ::: "memory");
#pragma unroll
        for (int kb = 0; kb < 16; ++kb) xf[kb] = __builtin_amdgcn_raw_buffer_load_b128(rxb, voff, kb * 1024, 0);
        if (!(st & 1u)) {
            st = wait_flags(ctrl, hdr, offw, offx, 1u, call, t0, tbar + (wave ? GATE_W : 0));
            asm volatile("" ::: "memory");
            STAMP(13);
#pragma unroll
            for (int f = 0; f < PULL_D; ++f) ring[f] = __builtin_amdgcn_raw_buffer_load_b128(rwb, voff, f * 1024, 0);
        }
        STAMP(5);
#pragma unroll
        for (int f = PULL_D; f < RING; ++f) {
            VMCNT(PULL_D - 1);
            ring[f] = __builtin_amdgcn_raw_buffer_load_b128(rwb, voff, f * 1024, 0);
        }
    } else {
        STAMP(13);
#pragma unroll
        for (int f = 0; f < RING; ++f) {
            if (f >= PULL_D) VMCNT(PULL_D - 1);
            ring[f] = __builtin_amdgcn_raw_buffer_load_b128(rwb, voff, f * 1024, 0);
        }
        st = wait_flags(ctrl, hdr, offw, offx, 2u, call, t0, tbar);
        STAMP(5);
        asm volatile("" ::: "memory");
#pragma unroll
        for (int kb = 0; kb < 16; ++kb) xf[kb] = __builtin_amdgcn_raw_buffer_load_b128(rxb, voff, kb * 1024, 0);
    }
    STAMP(6);
    if ((st & 4u) && lane == 0) __hip_atomic_store(&lds[LDS_CNT / 4 + 9], 1u, __ATOMIC_RELAXED, __HIP_MEMORY_SCOPE_WORKGROUP);

    const unsigned lcol = 4 * (lane & 7) + ((lane >> 3) & 3);
    unsigned* const lacc = &lds[LDS_ACC / 4 + (lane >> 5) * 128 + lcol];

    unsigned lvl = remote ? 1u : 3u;
    auto tile = [&](unsigned t, bool last) __attribute__((always_inline)) {
        i32x16 acc = {0, 0, 0, 0, 0, 0, 0, 0, 0, 0, 0, 0, 0, 0, 0, 0};
#pragma unroll
        for (int kb = 0; kb < 16; ++kb) {
            const bool more = !last || kb + RING < 16;
            if (more && remote && (kb & (LAG_P - 1)) == 0) {
                const unsigned pos = t * 16 + kb;
                unsigned front = 0;
                if (lane == 0) front = __hip_atomic_fetch_max(&lds[LDS_CNT / 4 + 11], pos, __ATOMIC_RELAXED, __HIP_MEMORY_SCOPE_WORKGROUP);
                const unsigned tref = rfl(__hip_atomic_load(&lds[LDS_CNT / 4 + 12], __ATOMIC_RELAXED, __HIP_MEMORY_SCOPE_WORKGROUP));
                const unsigned now = (unsigned)__builtin_amdgcn_s_memrealtime();
                front = rfl(front);
                if (kb == (BAL_POS & 15) && t == (BAL_POS >> 4) && rb == 0 && front < pos && lane == 0)
                    __builtin_amdgcn_raw_buffer_store_b32(now, ctrl, 0, OFF_PROG + g * 4, AUX_SYS);
                const int behind = (int)(now - tref) - (((int)pos - BAL_POS) * BAL_TAU >> 4);
                const unsigned paced = tref == 0 ? 1u : behind >= BAL_T ? 2u : behind <= -BAL_T ? 0u : 1u;
                lvl = front < pos + LAG_T ? paced : 3u;
            }
            if (more) {
                if (lvl <= 2) VMCNT(BOOST_D - 1);
                if (lvl <= 1) VMCNT(PULL_D - 1);
                if (lvl == 0) VMCNT(SLOW_D - 1);
            }
            __builtin_amdgcn_sched_barrier(0);
            acc = __builtin_amdgcn_mfma_i32_32x32x32_i8((i32x4)xf[kb], (i32x4)ring[kb % RING], acc, 0, 0, 0);
            __builtin_amdgcn_sched_barrier(0);
            if (more) ring[kb % RING] = __builtin_amdgcn_raw_buffer_load_b128(rwb, voff, t * 16384 + (kb + RING) * 1024, 0);
            __builtin_amdgcn_sched_barrier(0);
        }
#pragma unroll
        for (int r = 0; r < 16; ++r)
            __hip_atomic_fetch_add(lacc + t * 1024 + ((r & 3) + 8 * (r >> 2)) * 32, (unsigned)acc[r],
                                   __ATOMIC_RELAXED, __HIP_MEMORY_SCOPE_WORKGROUP);
        LGKM0();
        if (lane == 0) __hip_atomic_fetch_add(&lds[LDS_CNT / 4 + t], 1u, __ATOMIC_RELAXED, __HIP_MEMORY_SCOPE_WORKGROUP);
    };
#pragma unroll 1
    for (unsigned t = 0; t < 7; ++t) {
        tile(t, false);
        if (t == 0) STAMP(7);
        if (t == 3) STAMP(8);
    }
    tile(7, true);
    STAMP(9);

    if (wave == 0) {
        if (lane == 0) __builtin_amdgcn_raw_buffer_store_b32(call, ctrl, 0, OFF_SEQ + b * 128, AUX_SYS);
        const float s = P.sx[0] * P.sw[0];
        const rsrc_t ro = mk_rsrc(P.out + (size_t)rb * 32 * NN + g * 256, (31 * NN + 256) * 4);
        const unsigned ooff = (lane >> 3) * (NN * 4) + (lane & 7) * 16;
        bool have_ref = false;
#pragma unroll 1
        for (unsigned t = 0; t < 8; ++t) {
            while (rfl(__hip_atomic_load(&lds[LDS_CNT / 4 + t], __ATOMIC_RELAXED, __HIP_MEMORY_SCOPE_WORKGROUP)) < 8u) {
                if (t < 7) {
                    if (!have_ref) {
                        asm volatile("" ::: "memory");
                        const unsigned pv = __builtin_amdgcn_raw_buffer_load_b32(ctrl, (lane & 15) * 4, OFF_PROG, AUX_SYS) - (unsigned)t0;
                        if (__all(pv < 10000u)) {
                            unsigned sum = 0;
#pragma unroll
                            for (int l = 0; l < 16; ++l) sum += __builtin_amdgcn_readlane(pv, l);
                            if (lane == 0) __hip_atomic_store(&lds[LDS_CNT / 4 + 12], ((unsigned)t0 + (sum >> 4)) | 1u, __ATOMIC_RELAXED, __HIP_MEMORY_SCOPE_WORKGROUP);
                            have_ref = true;
                        } else {
                            __builtin_amdgcn_s_sleep(64);
                        }
                    }
                    __builtin_amdgcn_s_sleep(4);
                }
            }
            asm volatile("" ::: "memory");
            if (t == 7) STAMP(10);
            const bool bad = __hip_atomic_load(&lds[LDS_CNT / 4 + 9], __ATOMIC_RELAXED, __HIP_MEMORY_SCOPE_WORKGROUP) != 0;
            i32x4 a[4];
            f32x4 y[4];
#pragma unroll
            for (int q = 0; q < 4; ++q) a[q] = *(const i32x4*)&lds[LDS_ACC / 4 + t * 1024 + q * 256 + lane * 4];
#pragma unroll
            for (int q = 0; q < 4; ++q)
#pragma unroll
                for (int e = 0; e < 4; ++e) y[q][e] = bad ? __builtin_nanf("") : fmaxf((float)a[q][e] * s, 0.0f);
            __builtin_amdgcn_sched_barrier(0);
#pragma unroll
            for (int q = 0; q < 4; ++q) {
                __builtin_amdgcn_raw_buffer_store_b128((u32x4)y[q], ro, ooff, q * 8 * (NN * 4) + t * 128, AUX_SYS);
            }
            __builtin_amdgcn_sched_barrier(0);
            asm volatile("s_nop 1");
        }
        STAMP(11);
    }
}

}

extern "C" void kernel_launch(int rank, int nranks, ncclComm_t comm,
                              hipStream_t stream,
                              void* const* d_in, const int* in_sizes, int n_in,
                              void* d_out, int out_size,
                              const int* in_dtypes, int out_dtype,
                              void* const* d_out_all,
                              void* const* d_ws_all, size_t ws_bytes) {
    if (nranks != NR || (unsigned)rank >= (unsigned)NR || n_in < 4 || ws_bytes < WS_NEED ||
        out_size != MS * NN || in_sizes[0] != 4096 * KS || in_sizes[1] != KS * NN) {
        fprintf(stderr, "unexpected problem shape\n");
        return;
    }
    Params P;
    P.x = (const int8_t*)d_in[0];
    P.w = (const int8_t*)d_in[1];
    P.sx = (const float*)d_in[2];
    P.sw = (const float*)d_in[3];
    P.out = (float*)d_out;
    for (int j = 0; j < NR; ++j) P.ws[j] = (char*)d_ws_all[(rank + j) % NR];
    P.rank = rank;
    gemm_rs_fused<<<NWG, 512, 0, stream>>>(P);
    HIP_CHECK(hipGetLastError());
}
```

```cpp
#include <hip/hip_runtime.h>
#include <rccl/rccl.h>
#include <cstdio>
#include <cstdint>

#define HIP_CHECK(expr) do { hipError_t _e = (expr); if (_e != hipSuccess) { \
    fprintf(stderr, "HIP error %s at %s:%d\n", hipGetErrorString(_e), __FILE__, __LINE__); \
    return; } } while (0)

namespace {

typedef int i32x4 __attribute__((ext_vector_type(4)));
typedef unsigned u32x4 __attribute__((ext_vector_type(4)));
typedef unsigned u32x2 __attribute__((ext_vector_type(2)));
typedef int i32x16 __attribute__((ext_vector_type(16)));
typedef float f32x4 __attribute__((ext_vector_type(4)));
typedef __amdgpu_buffer_rsrc_t rsrc_t;
typedef __attribute__((address_space(1))) unsigned gu32;
typedef __attribute__((address_space(1))) unsigned long long gu64;

constexpr int NR = 8;
constexpr int MS = 512, KS = 512;
constexpr int NN = 4096;
constexpr int NWG = 256;

constexpr unsigned REG_HDR = 8192, REG_DATA = 262144, REG_BYTES = REG_HDR + 2 * REG_DATA;
constexpr unsigned HDR_XFLAG = 0, HDR_WFLAG = 2048, HDR_CFLAG = 4096, HDR_WARM = 6144;
constexpr unsigned OFF_XREG = 0, OFF_WREG = 6u << 20, OFF_CREG = 12u << 20;
__host__ __device__ constexpr unsigned reg_off(unsigned i) {
    return ((i * 11u) >> 5) * (2u << 20) + (i - ((i * 11u) >> 5) * 3u) * REG_BYTES;
}
constexpr unsigned OFF_CTRL  = 0x190000;
constexpr unsigned OFF_SEQ   = 0;
constexpr unsigned OFF_WCNT  = 32768;
constexpr unsigned OFF_TMO   = 36864;
constexpr unsigned OFF_INV   = 40960;
constexpr unsigned OFF_PROG  = 45056;
constexpr unsigned CTRL_BYTES = 65536;
constexpr unsigned OFF_DBG   = 18u << 20;
constexpr size_t   WS_NEED   = 20u << 20;
static_assert(3 * REG_BYTES <= OFF_CTRL && OFF_CTRL + CTRL_BYTES <= (2u << 20), "layout");

#define PULL_D 4
#define RING 8
#define LAG_P 2
#define LAG_T 2
#define BAL_T 30
#define BAL_POS 32
#define BAL_TAU 438
#define RD_LEAD 8
#define BAL_OFS 0
#define PLAN_TILE 2
#define RELAY 1
#define RD_K 3
#define LAG_UNTIL 0
#define BOOST_D 5
#define SLOW_D 3
#define STAGE_SEQ 0
#define GATE_XR 100
#define GATE_XM 30
#define INV_SLEEP 1
#define GATE_X 200
#define GATE_W 400
#define GATE_P 0
#define REMAP 1
#define POLL_SLEEP 1
#define POLL_START 0
#define BAR_AT XWIN
#define WARM_DELAY 2
#define XWIN 10
constexpr unsigned long long DEADLINE_TICKS = 100000000ull;

constexpr int AUX_SYS = 17;

struct Params {
    const int8_t* x;
    const int8_t* w;
    const float* sx;
    const float* sw;
    float* out;
    char* ws[NR];
    int rank;
};

#define VMCNT(n) asm volatile("s_waitcnt vmcnt(%0)" :: "i"(n) : "memory")
#define LGKM0() asm volatile("s_waitcnt lgkmcnt(0)" ::: "memory")

__device__ __forceinline__ rsrc_t mk_rsrc(const void* p, unsigned bytes) {
    return __builtin_amdgcn_make_buffer_rsrc(const_cast<void*>(p), 0, bytes, 0x00020000);
}
__device__ __forceinline__ unsigned rfl(unsigned v) { return __builtin_amdgcn_readfirstlane(v); }

__device__ __forceinline__ int row_sum(int v) {
    v += __builtin_amdgcn_update_dpp(0, v, 0xB1, 0xf, 0xf, false);
    v += __builtin_amdgcn_update_dpp(0, v, 0x4E, 0xf, 0xf, false);
    v += __builtin_amdgcn_update_dpp(0, v, 0x141, 0xf, 0xf, false);
    v += __builtin_amdgcn_update_dpp(0, v, 0x140, 0xf, 0xf, false);
    return v;
}

#define STAMP(i) do {} while (0)

__device__ __forceinline__ unsigned wait_flags(rsrc_t ctrl, rsrc_t hdr, unsigned offw, unsigned offx, unsigned need,
                                               unsigned call, unsigned long long t0, unsigned wgate) {
    unsigned spins = 0;
    for (;;) {
        asm volatile("" ::: "memory");
        const unsigned wv = __builtin_amdgcn_raw_buffer_load_b32(hdr, 0, offw, AUX_SYS);
        const u32x2 xv = __builtin_amdgcn_raw_buffer_load_b64(hdr, 0, offx, AUX_SYS);
        const unsigned a = rfl(xv.x), c = rfl(xv.y);
        const bool open = GATE_W == 0 || (int)((unsigned)__builtin_amdgcn_s_memrealtime() - wgate) >= 0;
        const unsigned ready = ((rfl(wv) >= call && open) ? 1u : 0u) | ((a >= call && c >= call) ? 2u : 0u);
        if (ready & need) return ready;
        if (++spins > 256) {
            const unsigned tmo = rfl(__builtin_amdgcn_raw_buffer_load_b32(ctrl, 0, OFF_TMO, AUX_SYS));
            if (tmo == call || __builtin_amdgcn_s_memrealtime() - t0 > DEADLINE_TICKS) {
                __builtin_amdgcn_raw_buffer_store_b32(call, ctrl, 0, OFF_TMO, AUX_SYS);
                return 7u;
            }
            __builtin_amdgcn_s_sleep(8);
        }
        __builtin_amdgcn_s_sleep(POLL_SLEEP);
    }
}

constexpr int LDS_ACC = 0;
constexpr int LDS_CNT = 32768;
constexpr int LDS_BYTES = 84 * 1024;

#define LDS_INIT() do { \
    *(u32x4*)&lds[tid * 16] = u32x4{0, 0, 0, 0}; *(u32x4*)&lds[tid * 16 + 4] = u32x4{0, 0, 0, 0}; \
    *(u32x4*)&lds[tid * 16 + 8] = u32x4{0, 0, 0, 0}; *(u32x4*)&lds[tid * 16 + 12] = u32x4{0, 0, 0, 0}; } while (0)

__global__ void __launch_bounds__(512, 2) gemm_rs_fused(Params P) {
    __shared__ __attribute__((aligned(16))) unsigned lds[LDS_BYTES / 4];
    const int tid = threadIdx.x, lane = tid & 63;
    const unsigned wave = rfl(tid >> 6);
    const unsigned b = blockIdx.x;
    const unsigned idx = b >> 3;
    const unsigned gh = idx & 1;
    const unsigned rb = 15 - (idx >> 1);
    const bool invwg = idx == 8;
    const unsigned g = (b & 7) * 2 + gh;
    if (wave == 5 && invwg) { __builtin_amdgcn_fence(__ATOMIC_ACQUIRE, ""); }
    const unsigned pj = gh * 4 + (rb & 3);
    unsigned long long t0 = __builtin_amdgcn_s_memrealtime();
    char* myws = P.ws[0];
    char* wsv = P.ws[wave];
    char* wsj = P.ws[pj];
    const int8_t* px = P.x;
    const int8_t* pw = P.w;
    int prank = P.rank;
    asm volatile("" : "+s"(myws), "+s"(wsv), "+s"(wsj), "+s"(px), "+s"(pw), "+s"(prank), "+s"(t0));
    const rsrc_t ctrl = mk_rsrc(myws + OFF_CTRL, CTRL_BYTES);

    const bool pusher = (wave == 0) && (rb >= 12);
    const bool stager = (wave >= 1) && (wave <= 4) && (rb < 8);
    const bool warmer = rb >= 8 && rb < 12 && wave < 2;
    char* const xr = wsj + OFF_XREG + reg_off((8 - pj) & 7);
    unsigned call, tbar;

    if (__builtin_expect(pusher, 1)) {
        const unsigned q = b & 7;
        const unsigned dst = ((unsigned)prank + pj) & 7u;
        const rsrc_t rx = mk_rsrc(px + (size_t)dst * MS * KS, MS * KS);
        const unsigned row = lane & 7, c = lane >> 3;
        u32x4 v[32];
        unsigned seqv = 0;
#pragma unroll
        for (int r = 0; r < 4; ++r) {
#pragma unroll
            for (int e = 0; e < 8; ++e) {
                const unsigned unit = 4 * r + (q >> 1);
                const unsigned soff = unit * 16384 + ((q & 1) * 2 + (e >> 2)) * 128 + (e & 3) * 4096;
                v[r * 8 + e] = __builtin_amdgcn_raw_buffer_load_b128(rx, row * 512 + c * 16, soff, 0);
                if (r == 0 && e == 0) {
                    asm volatile("" ::: "memory");
                    seqv = __builtin_amdgcn_raw_buffer_load_b32(ctrl, 0, OFF_SEQ + b * 128, AUX_SYS);
                    asm volatile("" ::: "memory");
                }
            }
        }
        LDS_INIT();
        call = rfl(seqv) + 1;
        tbar = (unsigned)t0;
        const rsrc_t rd = mk_rsrc(xr + REG_HDR + (call & 1) * REG_DATA, REG_DATA);
        const rsrc_t rf = mk_rsrc(xr + HDR_XFLAG, 2048);
#pragma unroll
        for (int s = 0; s < 32 + XWIN; ++s) {
            if (s == BAR_AT) {
                if (lane < 32) lds[LDS_CNT / 4 + lane] = lane == 10 ? call : 0;
                LGKM0();
                __builtin_amdgcn_s_barrier();
            }
            if (s >= XWIN) {
                const int done = s - XWIN;
                int younger = (s < 32 ? s : 32) - 1 - done;
                for (int r = 0; r < 4; ++r) { const int it = 8 * r + 7 + XWIN; if (it < s && it > done) ++younger; }
                VMCNT(younger);
                if ((done & 7) == 7) {
                    const unsigned unit = 4 * (done >> 3) + (q >> 1);
                    if (lane == 0) __builtin_amdgcn_raw_buffer_store_b32(call, rf, 0, unit * 128 + (q & 1) * 4, AUX_SYS);
                }
            }
            if (s < 32) {
                const int r = s >> 3, e = s & 7;
                const unsigned unit = 4 * r + (q >> 1);
                const unsigned soff = unit * 16384 + ((q & 1) * 2 + (e >> 2)) * 4096 + (e & 3) * 128;
                __builtin_amdgcn_raw_buffer_store_b128(v[s], rd, c * 512 + row * 16, soff, AUX_SYS);
            }
        }
        STAMP(2);
    } else if (stager) {
        const unsigned h = (wave - 1) & 1, skb = rb * 2 + ((wave - 1) >> 1);
        const rsrc_t rw = mk_rsrc(pw, KS * NN);
        unsigned d[16];
#pragma unroll
        for (int r = 0; r < 16; ++r)
            d[r] = __builtin_amdgcn_raw_buffer_load_b32(rw, lane * 4, (skb * 32 + h * 16 + r) * NN + g * 256, 0);
        LDS_INIT();
        LGKM0();
        __builtin_amdgcn_s_barrier();
        asm volatile("" ::: "memory");
        call = rfl(lds[LDS_CNT / 4 + 10]);
        tbar = (unsigned)__builtin_amdgcn_s_memrealtime();
        u32x4 o[4];
#pragma unroll
        for (int a = 0; a < 4; ++a) {
            const unsigned t0_ = __builtin_amdgcn_perm(d[4 * a + 1], d[4 * a], 0x05010400u);
            const unsigned t1_ = __builtin_amdgcn_perm(d[4 * a + 1], d[4 * a], 0x07030602u);
            const unsigned t2_ = __builtin_amdgcn_perm(d[4 * a + 3], d[4 * a + 2], 0x05010400u);
            const unsigned t3_ = __builtin_amdgcn_perm(d[4 * a + 3], d[4 * a + 2], 0x07030602u);
            o[0][a] = __builtin_amdgcn_perm(t2_, t0_, 0x05040100u);
            o[1][a] = __builtin_amdgcn_perm(t2_, t0_, 0x07060302u);
            o[2][a] = __builtin_amdgcn_perm(t3_, t1_, 0x05040100u);
            o[3][a] = __builtin_amdgcn_perm(t3_, t1_, 0x07060302u);
        }
        STAMP(1);
        const rsrc_t rd = mk_rsrc(myws + OFF_WREG + reg_off(b & 7) + REG_HDR + (call & 1) * REG_DATA + gh * 131072, 131072);
#pragma unroll
        for (int j = 0; j < 4; ++j)
            __builtin_amdgcn_raw_buffer_store_b128(o[j], rd, (lane >> 3) * 16384 + (lane & 7) * 16,
                                                   skb * 1024 + h * 512 + j * 128, AUX_SYS);
        VMCNT(0);
        STAMP(2);
        unsigned tk = 0;
        if (lane == 0)
            tk = __hip_atomic_fetch_add((gu32*)(myws + OFF_CTRL + OFF_WCNT + g * 128), 1u, __ATOMIC_RELAXED, __HIP_MEMORY_SCOPE_AGENT);
        tk = rfl(tk);
        if (tk + 1 == call * 32u) {
            if (lane == 0) {
#pragma unroll
                for (int jj = 1; jj <= NR; ++jj)
                    __builtin_amdgcn_raw_buffer_store_b32(call, mk_rsrc(P.ws[jj & 7] + OFF_XREG + reg_off((8 - jj) & 7) + HDR_WFLAG, 2048),
                                                          0, g * 128, AUX_SYS);
            }
        }
        STAMP(3);
    } else {
        unsigned warm = 0;
        if (warmer) {
            const char* wp = (wave & 1) == 0 ? xr + HDR_WARM : wsj + OFF_WREG + reg_off(b & 7) + HDR_WARM;
            if (wave != 0) __builtin_amdgcn_s_sleep(WARM_DELAY);
            warm = __builtin_amdgcn_raw_buffer_load_b32(mk_rsrc(wp, 4), 0, 0, AUX_SYS);
        }
        unsigned seqv = 0;
        if (wave == 0) seqv = __builtin_amdgcn_raw_buffer_load_b32(ctrl, 0, OFF_SEQ + b * 128, AUX_SYS);
        LDS_INIT();
        if (wave == 0 && lane < 32) lds[LDS_CNT / 4 + lane] = lane == 10 ? seqv + 1 : 0;
        LGKM0();
        __builtin_amdgcn_s_barrier();
        asm volatile("" ::: "memory");
        call = rfl(lds[LDS_CNT / 4 + 10]);
        tbar = (unsigned)__builtin_amdgcn_s_memrealtime();
        asm volatile("" :: "v"(warm));
        if (POLL_START && !(wave == 5 && invwg)) __builtin_amdgcn_s_sleep(POLL_START);
        if (wave == 5) {
            const unsigned xcc = __builtin_amdgcn_s_getreg(20 | (3 << 11)) & 15u;
            bool mine = invwg;
            while (!mine) {
                asm volatile("" ::: "memory");
                if (rfl(__builtin_amdgcn_raw_buffer_load_b32(ctrl, 0, OFF_INV + xcc * 128, AUX_SYS)) >= call) break;
                if (__builtin_amdgcn_s_memrealtime() - t0 > 1000ull) mine = true;
                __builtin_amdgcn_s_sleep(INV_SLEEP);
            }
            if (mine) {
                if (!invwg) __builtin_amdgcn_fence(__ATOMIC_ACQUIRE, "");
                VMCNT(0);
                if (lane == 0) __builtin_amdgcn_raw_buffer_store_b32(call, ctrl, 0, OFF_INV + xcc * 128, AUX_SYS);
            }
            STAMP(1);
            if (lane == 0) __hip_atomic_store(&lds[LDS_CNT / 4 + 8], 1u, __ATOMIC_RELAXED, __HIP_MEMORY_SCOPE_WORKGROUP);
        }
    }

    char* const myx = myws + OFF_XREG + reg_off(wave);
    const rsrc_t hdr = mk_rsrc(myx, REG_HDR);
    const unsigned offw = HDR_WFLAG + g * 128, offx = HDR_XFLAG + rb * 128;
    {
        unsigned gx = GATE_X + (wave ? (rb >> 2) * GATE_XR : 0);
        if (gx > GATE_W - GATE_XM) gx = GATE_W - GATE_XM;
        while ((unsigned)__builtin_amdgcn_s_memrealtime() - tbar < gx) __builtin_amdgcn_s_sleep(2);
    }
    unsigned st = wait_flags(ctrl, hdr, offw, offx, 3u, call, t0, tbar + (wave ? GATE_W : 0));
    STAMP(4);
    while (rfl(__hip_atomic_load(&lds[LDS_CNT / 4 + 8], __ATOMIC_RELAXED, __HIP_MEMORY_SCOPE_WORKGROUP)) == 0)
        __builtin_amdgcn_s_sleep(1);
    asm volatile("" ::: "memory");
    const rsrc_t rwb = mk_rsrc(wsv + OFF_WREG + reg_off(b & 7) + REG_HDR + (call & 1) * REG_DATA + gh * 131072, 131072);
    const rsrc_t rxb = mk_rsrc(myx + REG_HDR + (call & 1) * REG_DATA + rb * 16384, 16384);
    const unsigned voff = lane * 16;
    const bool remote = wave != 0;
    const unsigned hg = 1 + (g >= 14 ? g - 14 : g >= 7 ? g - 7 : g);
    auto tile_id = [&](unsigned t) -> unsigned { return ((unsigned)prank + (t == 7 ? hg : t < hg ? t : t + 1)) & 7u; };
    const unsigned o0 = tile_id(0) * 16384;
    u32x4 ring[RING];
    u32x4 xf[16];
    if (st & 2u) {
        if (st & 1u) {
            STAMP(13);
#pragma unroll
            for (int f = 0; f < PULL_D; ++f) ring[f] = __builtin_amdgcn_raw_buffer_load_b128(rwb, voff, o0 + f * 1024, 0);
        }
        asm volatile("" ::: "memory");
#pragma unroll
        for (int kb = 0; kb < 16; ++kb) xf[kb] = __builtin_amdgcn_raw_buffer_load_b128(rxb, voff, kb * 1024, 0);
        if (!(st & 1u)) {
            st = wait_flags(ctrl, hdr, offw, offx, 1u, call, t0, tbar + (wave ? GATE_W : 0));
            asm volatile("" ::: "memory");
            STAMP(13);
#pragma unroll
            for (int f = 0; f < PULL_D; ++f) ring[f] = __builtin_amdgcn_raw_buffer_load_b128(rwb, voff, o0 + f * 1024, 0);
        }
        STAMP(5);
#pragma unroll
        for (int f = PULL_D; f < RING; ++f) {
            VMCNT(PULL_D - 1);
            ring[f] = __builtin_amdgcn_raw_buffer_load_b128(rwb, voff, o0 + f * 1024, 0);
        }
    } else {
        STAMP(13);
#pragma unroll
        for (int f = 0; f < RING; ++f) {
            if (f >= PULL_D) VMCNT(PULL_D - 1);
            ring[f] = __builtin_amdgcn_raw_buffer_load_b128(rwb, voff, o0 + f * 1024, 0);
        }
        st = wait_flags(ctrl, hdr, offw, offx, 2u, call, t0, tbar);
        STAMP(5);
        asm volatile("" ::: "memory");
#pragma unroll
        for (int kb = 0; kb < 16; ++kb) xf[kb] = __builtin_amdgcn_raw_buffer_load_b128(rxb, voff, kb * 1024, 0);
    }
    STAMP(6);
    if ((st & 4u) && lane == 0) __hip_atomic_store(&lds[LDS_CNT / 4 + 9], 1u, __ATOMIC_RELAXED, __HIP_MEMORY_SCOPE_WORKGROUP);

    const unsigned lcol = 4 * (lane & 7) + ((lane >> 3) & 3);
    unsigned* const lacc = &lds[LDS_ACC / 4 + (lane >> 5) * 128 + lcol];

    unsigned lvl = remote ? 1u : 3u;
    rsrc_t rl = rwb;
    unsigned ol = tile_id(7) * 16384;
    int nmoved = 0;
    auto tile = [&](unsigned t, bool last) __attribute__((always_inline)) {
        i32x16 acc = {0, 0, 0, 0, 0, 0, 0, 0, 0, 0, 0, 0, 0, 0, 0, 0};
        const unsigned ocur = tile_id(t) * 16384;
        unsigned onext = tile_id(t + 1) * 16384;
        rsrc_t rnext = rwb;
#pragma unroll
        for (int kb = 0; kb < 16; ++kb) {
            const bool more = !last || kb + RING < 16;
            if (!last && kb == 16 - RING && t == 6) {
                const unsigned plan = rfl(__hip_atomic_load(&lds[LDS_CNT / 4 + 13], __ATOMIC_RELAXED, __HIP_MEMORY_SCOPE_WORKGROUP));
                if ((plan >> wave) & 1u) {
                    rl = mk_rsrc(P.ws[hg] + OFF_CREG + reg_off(b & 7) + REG_HDR + (call & 1) * REG_DATA + gh * 131072, 131072);
                    ol = ((wave - hg) & 7u) * 16384;
                }
                nmoved = (int)rfl(__hip_atomic_load(&lds[LDS_CNT / 4 + 24 + wave], __ATOMIC_RELAXED, __HIP_MEMORY_SCOPE_WORKGROUP));
                rnext = rl;
                onext = ol;
            }
            if (more && remote && (kb & 1) == 0) {
                const unsigned pos = t * 16 + kb;
                const unsigned tref = rfl(__hip_atomic_load(&lds[LDS_CNT / 4 + 16 + wave], __ATOMIC_RELAXED, __HIP_MEMORY_SCOPE_WORKGROUP));
                const unsigned now = (unsigned)__builtin_amdgcn_s_memrealtime();
                if (kb == (BAL_POS & 15) && t == (BAL_POS >> 4) && rb < 4 && lane == 0)
                    __builtin_amdgcn_raw_buffer_store_b32(now, ctrl, 0, OFF_PROG + wave * 256 + g * 16 + rb * 4, AUX_SYS);
                int behind = (int)(now - tref) - (((int)pos - BAL_POS) * BAL_TAU >> 4);
                if (last || kb >= 16 - RING) behind += nmoved * ((last ? kb + RING : kb - (16 - RING)) + RD_LEAD) * BAL_TAU >> 8;
                lvl = tref == 0 ? 1u : behind >= 3 * BAL_T ? 3u : behind >= BAL_T ? 2u : behind <= -BAL_T ? 0u : 1u;
            }
            if (more) {
                if (lvl <= 2) VMCNT(BOOST_D - 1);
                if (lvl <= 1) VMCNT(PULL_D - 1);
                if (lvl == 0) VMCNT(SLOW_D - 1);
            }
            __builtin_amdgcn_sched_barrier(0);
            acc = __builtin_amdgcn_mfma_i32_32x32x32_i8((i32x4)xf[kb], (i32x4)ring[kb % RING], acc, 0, 0, 0);
            __builtin_amdgcn_sched_barrier(0);
            if (more) {
                if (kb + RING < 16) ring[kb % RING] = __builtin_amdgcn_raw_buffer_load_b128(last ? rl : rwb, voff, (last ? ol : ocur) + (kb + RING) * 1024, 0);
                else ring[kb % RING] = __builtin_amdgcn_raw_buffer_load_b128(rnext, voff, onext + (kb + RING - 16) * 1024, 0);
            }
            __builtin_amdgcn_sched_barrier(0);
        }
#pragma unroll
        for (int r = 0; r < 16; ++r)
            __hip_atomic_fetch_add(lacc + t * 1024 + ((r & 3) + 8 * (r >> 2)) * 32, (unsigned)acc[r],
                                   __ATOMIC_RELAXED, __HIP_MEMORY_SCOPE_WORKGROUP);
        LGKM0();
        if (lane == 0) __hip_atomic_fetch_add(&lds[LDS_CNT / 4 + t], 1u, __ATOMIC_RELAXED, __HIP_MEMORY_SCOPE_WORKGROUP);
    };

    bool need_copy = RELAY && wave == 0 && rb >= 1 && rb <= 7 && rb != 8 - hg;
    bool need_plan = wave == 0;
    auto side_jobs = [&]() __attribute__((always_inline)) {
        if (need_copy && rfl(__hip_atomic_load(&lds[LDS_CNT / 4 + 0], __ATOMIC_RELAXED, __HIP_MEMORY_SCOPE_WORKGROUP)) >= 8u) {
            need_copy = false;
            if (rfl(__hip_atomic_load(&lds[LDS_CNT / 4 + 9], __ATOMIC_RELAXED, __HIP_MEMORY_SCOPE_WORKGROUP)) == 0) {
                asm volatile("" ::: "memory");
                const unsigned ro_ = reg_off(b & 7) + REG_HDR + (call & 1) * REG_DATA + gh * 131072;
                const rsrc_t rs = mk_rsrc(P.ws[rb & 7] + OFF_WREG + ro_ + o0, 16384);
                const rsrc_t rc = mk_rsrc(myws + OFF_CREG + ro_ + (rb & 7) * 16384, 16384);
                u32x4 c[16];
#pragma unroll
                for (int i = 0; i < 16; ++i) c[i] = __builtin_amdgcn_raw_buffer_load_b128(rs, voff, i * 1024, 0);
#pragma unroll
                for (int i = 0; i < 16; ++i) __builtin_amdgcn_raw_buffer_store_b128(c[i], rc, voff, i * 1024, AUX_SYS);
                VMCNT(0);
                if (lane == 0)
                    __builtin_amdgcn_raw_buffer_store_b32(call, mk_rsrc(P.ws[8 - hg] + OFF_XREG + reg_off(hg) + HDR_CFLAG, 2048),
                                                          0, g * 128 + ((rb + hg) & 7u) * 4, AUX_SYS);
            }
        }
        if (need_plan && rfl(__hip_atomic_load(&lds[LDS_CNT / 4 + PLAN_TILE], __ATOMIC_RELAXED, __HIP_MEMORY_SCOPE_WORKGROUP)) >= 8u) {
            asm volatile("" ::: "memory");
            const u32x4 qa = __builtin_amdgcn_raw_buffer_load_b128(ctrl, voff, OFF_PROG, AUX_SYS) - (unsigned)t0;
            const u32x4 qb = __builtin_amdgcn_raw_buffer_load_b128(ctrl, voff, OFF_PROG + 1024, AUX_SYS) - (unsigned)t0;
            const unsigned cf = __builtin_amdgcn_raw_buffer_load_b32(mk_rsrc(myws + OFF_XREG + reg_off(hg) + HDR_CFLAG + g * 128, 32), lane * 4, 0, AUX_SYS);
            const unsigned xa = max(max(qa.x, qa.y), max(qa.z, qa.w)), xb = max(max(qb.x, qb.y), max(qb.z, qb.w));
            if (__all((lane < 16 || xa < 10000u) && xb < 10000u)) {
                need_plan = false;
                int sa = (int)min(min(qa.x, qa.y), min(qa.z, qa.w)), sb = (int)min(min(qb.x, qb.y), min(qb.z, qb.w));
                sa = row_sum(sa); sb = row_sum(sb);
                int S = 0;
#pragma unroll
                for (int v = 1; v < 8; ++v) {
                    const int sv = __builtin_amdgcn_readlane(v < 4 ? sa : sb, (v & 3) * 16);
                    S = lane == v ? sv : S;
                }
                if (lane >= 1 && lane < 8) __hip_atomic_store(&lds[LDS_CNT / 4 + 16 + lane], ((unsigned)t0 + ((unsigned)S >> 4) + BAL_OFS) | 1u, __ATOMIC_RELAXED, __HIP_MEMORY_SCOPE_WORKGROUP);
                unsigned used = 0;
                unsigned nv = 0;
#pragma unroll 1
                for (int pass = 0; pass < 3; ++pass) {
                    int mx = __builtin_amdgcn_readlane(S, 1), mn = mx;
#pragma unroll
                    for (int v = 2; v < 8; ++v) { const int sv = __builtin_amdgcn_readlane(S, v); mx = max(mx, sv); mn = min(mn, sv); }
                    if (mx - mn < BAL_TAU * RD_K / 2) break;
                    bool moved = false;
                    unsigned hh = 1;
#pragma unroll 1
                    for (unsigned gg = 0; gg < 16; ++gg) {
                        const unsigned um = __builtin_amdgcn_readlane(used, gg) | (1u << hh) | 1u;
                        int key = ((um >> (lane & 7)) & 1u) ? -1 : (S * 8 + lane);
                        key = max(key, __builtin_amdgcn_update_dpp(key, key, 0xB1, 0xf, 0xf, false));
                        key = max(key, __builtin_amdgcn_update_dpp(key, key, 0x4E, 0xf, 0xf, false));
                        const int top = max((int)__builtin_amdgcn_readlane(key, 0), (int)__builtin_amdgcn_readlane(key, 4));
                        const int sh = __builtin_amdgcn_readlane(S, hh);
                        if (top >= 0 && (top >> 3) - sh >= BAL_TAU * RD_K / 2) {
                            const unsigned vv = top & 7;
                            if ((unsigned)lane == vv) { S -= BAL_TAU; ++nv; }
                            if ((unsigned)lane == hh) S += BAL_TAU;
                            if ((unsigned)lane == gg) used |= 1u << vv;
                            moved = true;
                        }
                        hh = hh == 7 ? 1 : hh + 1;
                    }
                    if (!moved) break;
                }
                const unsigned have = (unsigned)__ballot(lane < 8 && cf >= call);
                const unsigned plan = __builtin_amdgcn_readlane(used, g) & have;
                if (lane < 8) __hip_atomic_store(&lds[LDS_CNT / 4 + 24 + lane], nv, __ATOMIC_RELAXED, __HIP_MEMORY_SCOPE_WORKGROUP);
                if (lane == 0) __hip_atomic_store(&lds[LDS_CNT / 4 + 13], plan, __ATOMIC_RELAXED, __HIP_MEMORY_SCOPE_WORKGROUP);
                STAMP(2);
            }
        }
    };
#pragma unroll 1
    for (unsigned t = 0; t < 7; ++t) {
        tile(t, false);
        if (t == 0) STAMP(7);
        if (t == 3) STAMP(8);
        if (wave == 0) side_jobs();
    }
    tile(7, true);
    STAMP(9);

    if (wave == 0) {
        if (lane == 0) __builtin_amdgcn_raw_buffer_store_b32(call, ctrl, 0, OFF_SEQ + b * 128, AUX_SYS);
        const float s = P.sx[0] * P.sw[0];
        const rsrc_t ro = mk_rsrc(P.out + (size_t)rb * 32 * NN + g * 256, (31 * NN + 256) * 4);
        const unsigned ooff = (lane >> 3) * (NN * 4) + (lane & 7) * 16;
#pragma unroll 1
        for (unsigned t = 0; t < 8; ++t) {
            while (rfl(__hip_atomic_load(&lds[LDS_CNT / 4 + t], __ATOMIC_RELAXED, __HIP_MEMORY_SCOPE_WORKGROUP)) < 8u) {
                if (t < 7) {
                    side_jobs();
                    if (need_plan && t > PLAN_TILE) __builtin_amdgcn_s_sleep(64);
                    __builtin_amdgcn_s_sleep(4);
                }
            }
            asm volatile("" ::: "memory");
            if (t == 7) STAMP(10);
            const bool bad = __hip_atomic_load(&lds[LDS_CNT / 4 + 9], __ATOMIC_RELAXED, __HIP_MEMORY_SCOPE_WORKGROUP) != 0;
            i32x4 a[4];
            f32x4 y[4];
#pragma unroll
            for (int q = 0; q < 4; ++q) a[q] = *(const i32x4*)&lds[LDS_ACC / 4 + t * 1024 + q * 256 + lane * 4];
#pragma unroll
            for (int q = 0; q < 4; ++q)
#pragma unroll
                for (int e = 0; e < 4; ++e) y[q][e] = bad ? __builtin_nanf("") : fmaxf((float)a[q][e] * s, 0.0f);
            __builtin_amdgcn_sched_barrier(0);
#pragma unroll
            for (int q = 0; q < 4; ++q) {
                __builtin_amdgcn_raw_buffer_store_b128((u32x4)y[q], ro, ooff, q * 8 * (NN * 4) + tile_id(t) * 128, AUX_SYS);
            }
            __builtin_amdgcn_sched_barrier(0);
            asm volatile("s_nop 1");
        }
        STAMP(11);
    }
}

}

extern "C" void kernel_launch(int rank, int nranks, ncclComm_t comm,
                              hipStream_t stream,
                              void* const* d_in, const int* in_sizes, int n_in,
                              void* d_out, int out_size,
                              const int* in_dtypes, int out_dtype,
                              void* const* d_out_all,
                              void* const* d_ws_all, size_t ws_bytes) {
    if (nranks != NR || (unsigned)rank >= (unsigned)NR || n_in < 4 || ws_bytes < WS_NEED ||
        out_size != MS * NN || in_sizes[0] != 4096 * KS || in_sizes[1] != KS * NN) {
        fprintf(stderr, "unexpected problem shape\n");
        return;
    }
    Params P;
    P.x = (const int8_t*)d_in[0];
    P.w = (const int8_t*)d_in[1];
    P.sx = (const float*)d_in[2];
    P.sw = (const float*)d_in[3];
    P.out = (float*)d_out;
    for (int j = 0; j < NR; ++j) P.ws[j] = (char*)d_ws_all[(rank + j) % NR];
    P.rank = rank;
    gemm_rs_fused<<<NWG, 512, 0, stream>>>(P);
    HIP_CHECK(hipGetLastError());
}
```

```cpp
#include <hip/hip_runtime.h>
#include <rccl/rccl.h>
#include <cstdio>
#include <cstdint>

#define HIP_CHECK(expr) do { hipError_t _e = (expr); if (_e != hipSuccess) { \
    fprintf(stderr, "HIP error %s at %s:%d\n", hipGetErrorString(_e), __FILE__, __LINE__); \
    return; } } while (0)

namespace {

typedef int i32x4 __attribute__((ext_vector_type(4)));
typedef unsigned u32x4 __attribute__((ext_vector_type(4)));
typedef unsigned u32x2 __attribute__((ext_vector_type(2)));
typedef int i32x16 __attribute__((ext_vector_type(16)));
typedef float f32x4 __attribute__((ext_vector_type(4)));
typedef __amdgpu_buffer_rsrc_t rsrc_t;
typedef __attribute__((address_space(1))) unsigned gu32;

constexpr int NR = 8;
constexpr int MS = 512, KS = 512;
constexpr int NN = 4096;
constexpr int NWG = 256;

constexpr unsigned REG_HDR = 8192, REG_DATA = 262144, REG_BYTES = REG_HDR + 2 * REG_DATA;
constexpr unsigned HDR_XFLAG = 0, HDR_WFLAG = 2048, HDR_CFLAG = 4096, HDR_WARM = 6144;
constexpr unsigned OFF_XREG = 0, OFF_WREG = 6u << 20, OFF_CREG = 12u << 20;
__host__ __device__ constexpr unsigned reg_off(unsigned i) {
    return ((i * 11u) >> 5) * (2u << 20) + (i - ((i * 11u) >> 5) * 3u) * REG_BYTES;
}
constexpr unsigned OFF_CTRL  = 0x190000;
constexpr unsigned OFF_SEQ   = 0;
constexpr unsigned OFF_WCNT  = 32768;
constexpr unsigned OFF_TMO   = 36864;
constexpr unsigned OFF_INV   = 40960;
constexpr unsigned OFF_PROG  = 45056;
constexpr unsigned CTRL_BYTES = 65536;
constexpr size_t   WS_NEED   = 18u << 20;
static_assert(3 * REG_BYTES <= OFF_CTRL && OFF_CTRL + CTRL_BYTES <= (2u << 20), "layout");
static_assert(OFF_CREG + reg_off(7) + REG_BYTES <= WS_NEED, "layout");

constexpr int RING = 8;
constexpr int PULL_D = 4, BOOST_D = 5, SLOW_D = 3;
constexpr int XWIN = 10;
constexpr unsigned GATE_W = 400;
constexpr unsigned GATE_X = 200, GATE_XR = 100;
constexpr int BAL_POS = 24;
constexpr int BAL_TAU = 438;
constexpr int BAL_T = 30;
constexpr int MOVE_MIN = BAL_TAU * 3 / 2;
constexpr unsigned long long DEADLINE_TICKS = 100000000ull;

constexpr int AUX_SYS = 17;

struct Params {
    const int8_t* x;
    const int8_t* w;
    const float* sx;
    const float* sw;
    float* out;
    char* ws[NR];
    int rank;
};

#define VMCNT(n) asm volatile("s_waitcnt vmcnt(%0)" :: "i"(n) : "memory")
#define LGKM0() asm volatile("s_waitcnt lgkmcnt(0)" ::: "memory")
#define ORDER() asm volatile("" ::: "memory")
#define LDS_LD(i) __hip_atomic_load(&lds[LDS_CNT / 4 + (i)], __ATOMIC_RELAXED, __HIP_MEMORY_SCOPE_WORKGROUP)
#define LDS_ST(i, v) __hip_atomic_store(&lds[LDS_CNT / 4 + (i)], (v), __ATOMIC_RELAXED, __HIP_MEMORY_SCOPE_WORKGROUP)

__device__ __forceinline__ rsrc_t mk_rsrc(const void* p, unsigned bytes) {
    return __builtin_amdgcn_make_buffer_rsrc(const_cast<void*>(p), 0, bytes, 0x00020000);
}
__device__ __forceinline__ unsigned rfl(unsigned v) { return __builtin_amdgcn_readfirstlane(v); }
__device__ __forceinline__ unsigned clk() { return (unsigned)__builtin_amdgcn_s_memrealtime(); }

__device__ __forceinline__ int row_sum(int v) {
    v += __builtin_amdgcn_update_dpp(0, v, 0xB1, 0xf, 0xf, false);
    v += __builtin_amdgcn_update_dpp(0, v, 0x4E, 0xf, 0xf, false);
    v += __builtin_amdgcn_update_dpp(0, v, 0x141, 0xf, 0xf, false);
    v += __builtin_amdgcn_update_dpp(0, v, 0x140, 0xf, 0xf, false);
    return v;
}

__device__ __forceinline__ unsigned wait_flags(rsrc_t ctrl, rsrc_t hdr, unsigned offw, unsigned offx, unsigned need,
                                               unsigned call, unsigned long long t0, unsigned wgate) {
    unsigned spins = 0;
    for (;;) {
        ORDER();
        const unsigned wv = __builtin_amdgcn_raw_buffer_load_b32(hdr, 0, offw, AUX_SYS);
        const u32x2 xv = __builtin_amdgcn_raw_buffer_load_b64(hdr, 0, offx, AUX_SYS);
        const unsigned a = rfl(xv.x), c = rfl(xv.y);
        const bool open = (int)(clk() - wgate) >= 0;
        const unsigned ready = ((rfl(wv) >= call && open) ? 1u : 0u) | ((a >= call && c >= call) ? 2u : 0u);
        if (ready & need) return ready;
        if (++spins > 256) {
            const unsigned tmo = rfl(__builtin_amdgcn_raw_buffer_load_b32(ctrl, 0, OFF_TMO, AUX_SYS));
            if (tmo == call || __builtin_amdgcn_s_memrealtime() - t0 > DEADLINE_TICKS) {
                __builtin_amdgcn_raw_buffer_store_b32(call, ctrl, 0, OFF_TMO, AUX_SYS);
                return 7u;
            }
            __builtin_amdgcn_s_sleep(8);
        }
        __builtin_amdgcn_s_sleep(1);
    }
}

constexpr int LDS_ACC = 0;
constexpr int LDS_CNT = 32768;
constexpr int W_INV = 8, W_BAD = 9, W_CALL = 10, W_PLAN = 13, W_TREF = 16, W_MOVED = 24;
constexpr int LDS_BYTES = 84 * 1024;

#define LDS_INIT() do { \
    *(u32x4*)&lds[tid * 16] = u32x4{0, 0, 0, 0}; *(u32x4*)&lds[tid * 16 + 4] = u32x4{0, 0, 0, 0}; \
    *(u32x4*)&lds[tid * 16 + 8] = u32x4{0, 0, 0, 0}; *(u32x4*)&lds[tid * 16 + 12] = u32x4{0, 0, 0, 0}; } while (0)

__global__ void __launch_bounds__(512, 2) gemm_rs_fused(Params P) {
    __shared__ __attribute__((aligned(16))) unsigned lds[LDS_BYTES / 4];
    const int tid = threadIdx.x, lane = tid & 63;
    const unsigned wave = rfl(tid >> 6) & 7u;
    const unsigned b = blockIdx.x & (NWG - 1);
    const unsigned idx = b >> 3;
    const unsigned gh = idx & 1;
    const unsigned rb = 15 - (idx >> 1);
    const unsigned g = (b & 7) * 2 + gh;
    const bool invwg = idx == 8;
    if (wave == 5 && invwg) __builtin_amdgcn_fence(__ATOMIC_ACQUIRE, "");
    const unsigned pj = gh * 4 + (rb & 3);
    unsigned long long t0 = __builtin_amdgcn_s_memrealtime();
    char* myws = P.ws[0];
    char* wsv = P.ws[wave];
    char* wsj = P.ws[pj];
    const int8_t* px = P.x;
    const int8_t* pw = P.w;
    int prank = P.rank;
    asm volatile("" : "+s"(myws), "+s"(wsv), "+s"(wsj), "+s"(px), "+s"(pw), "+s"(prank), "+s"(t0));
    const rsrc_t ctrl = mk_rsrc(myws + OFF_CTRL, CTRL_BYTES);

    const bool pusher = (wave == 0) && (rb >= 12);
    const bool stager = (wave >= 1) && (wave <= 4) && (rb < 8);
    char* const xr = wsj + OFF_XREG + reg_off((8 - pj) & 7);
    unsigned call, tbar;

    if (__builtin_expect(pusher, 1)) {
        const unsigned q = b & 7;
        const unsigned dst = ((unsigned)prank + pj) & 7u;
        const rsrc_t rx = mk_rsrc(px + (size_t)dst * MS * KS, MS * KS);
        const unsigned row = lane & 7, c = lane >> 3;
        u32x4 v[32];
        unsigned seqv = 0;
#pragma unroll
        for (int r = 0; r < 4; ++r) {
#pragma unroll
            for (int e = 0; e < 8; ++e) {
                const unsigned unit = 4 * r + (q >> 1);
                const unsigned soff = unit * 16384 + ((q & 1) * 2 + (e >> 2)) * 128 + (e & 3) * 4096;
                v[r * 8 + e] = __builtin_amdgcn_raw_buffer_load_b128(rx, row * 512 + c * 16, soff, 0);
                if (r == 0 && e == 0) {
                    ORDER();
                    seqv = __builtin_amdgcn_raw_buffer_load_b32(ctrl, 0, OFF_SEQ + b * 128, AUX_SYS);
                    ORDER();
                }
            }
        }
        LDS_INIT();
        call = rfl(seqv) + 1;
        tbar = (unsigned)t0;
        const rsrc_t rd = mk_rsrc(xr + REG_HDR + (call & 1) * REG_DATA, REG_DATA);
        const rsrc_t rf = mk_rsrc(xr + HDR_XFLAG, 2048);
#pragma unroll
        for (int s = 0; s < 32 + XWIN; ++s) {
            if (s == XWIN) {
                if (lane < 32) lds[LDS_CNT / 4 + lane] = lane == W_CALL ? call : 0;
                LGKM0();
                __builtin_amdgcn_s_barrier();
            }
            if (s >= XWIN) {
                const int done = s - XWIN;
                int younger = (s < 32 ? s : 32) - 1 - done;
                for (int r = 0; r < 4; ++r) { const int it = 8 * r + 7 + XWIN; if (it < s && it > done) ++younger; }
                VMCNT(younger);
                if ((done & 7) == 7) {
                    const unsigned unit = 4 * (done >> 3) + (q >> 1);
                    if (lane == 0) __builtin_amdgcn_raw_buffer_store_b32(call, rf, 0, unit * 128 + (q & 1) * 4, AUX_SYS);
                }
            }
            if (s < 32) {
                const int r = s >> 3, e = s & 7;
                const unsigned unit = 4 * r + (q >> 1);
                const unsigned soff = unit * 16384 + ((q & 1) * 2 + (e >> 2)) * 4096 + (e & 3) * 128;
                __builtin_amdgcn_raw_buffer_store_b128(v[s], rd, c * 512 + row * 16, soff, AUX_SYS);
            }
        }
    } else if (stager) {
        const unsigned h = (wave - 1) & 1, skb = rb * 2 + ((wave - 1) >> 1);
        const rsrc_t rw = mk_rsrc(pw, KS * NN);
        unsigned d[16];
#pragma unroll
        for (int r = 0; r < 16; ++r)
            d[r] = __builtin_amdgcn_raw_buffer_load_b32(rw, lane * 4, (skb * 32 + h * 16 + r) * NN + g * 256, 0);
        LDS_INIT();
        LGKM0();
        __builtin_amdgcn_s_barrier();
        ORDER();
        call = rfl(lds[LDS_CNT / 4 + W_CALL]);
        tbar = clk();
        u32x4 o[4];
#pragma unroll
        for (int a = 0; a < 4; ++a) {
            const unsigned p0 = __builtin_amdgcn_perm(d[4 * a + 1], d[4 * a], 0x05010400u);
            const unsigned p1 = __builtin_amdgcn_perm(d[4 * a + 1], d[4 * a], 0x07030602u);
            const unsigned p2 = __builtin_amdgcn_perm(d[4 * a + 3], d[4 * a + 2], 0x05010400u);
            const unsigned p3 = __builtin_amdgcn_perm(d[4 * a + 3], d[4 * a + 2], 0x07030602u);
            o[0][a] = __builtin_amdgcn_perm(p2, p0, 0x05040100u);
            o[1][a] = __builtin_amdgcn_perm(p2, p0, 0x07060302u);
            o[2][a] = __builtin_amdgcn_perm(p3, p1, 0x05040100u);
            o[3][a] = __builtin_amdgcn_perm(p3, p1, 0x07060302u);
        }
        const rsrc_t rd = mk_rsrc(myws + OFF_WREG + reg_off(b & 7) + REG_HDR + (call & 1) * REG_DATA + gh * 131072, 131072);
#pragma unroll
        for (int j = 0; j < 4; ++j)
            __builtin_amdgcn_raw_buffer_store_b128(o[j], rd, (lane >> 3) * 16384 + (lane & 7) * 16,
                                                   skb * 1024 + h * 512 + j * 128, AUX_SYS);
        VMCNT(0);
        unsigned tk = 0;
        if (lane == 0)
            tk = __hip_atomic_fetch_add((gu32*)(myws + OFF_CTRL + OFF_WCNT + g * 128), 1u, __ATOMIC_RELAXED, __HIP_MEMORY_SCOPE_AGENT);
        if (rfl(tk) + 1 == call * 32u && lane == 0) {
#pragma unroll
            for (int jj = 1; jj <= NR; ++jj)
                __builtin_amdgcn_raw_buffer_store_b32(call, mk_rsrc(P.ws[jj & 7] + OFF_XREG + reg_off((8 - jj) & 7) + HDR_WFLAG, 2048),
                                                      0, g * 128, AUX_SYS);
        }
    } else {
        unsigned warm = 0;
        if (rb >= 8 && rb < 12 && wave < 2) {
            const char* wp = wave == 0 ? xr + HDR_WARM : wsj + OFF_WREG + reg_off(b & 7) + HDR_WARM;
            if (wave != 0) __builtin_amdgcn_s_sleep(2);
            warm = __builtin_amdgcn_raw_buffer_load_b32(mk_rsrc(wp, 4), 0, 0, AUX_SYS);
        }
        unsigned seqv = 0;
        if (wave == 0) seqv = __builtin_amdgcn_raw_buffer_load_b32(ctrl, 0, OFF_SEQ + b * 128, AUX_SYS);
        LDS_INIT();
        if (wave == 0 && lane < 32) lds[LDS_CNT / 4 + lane] = lane == W_CALL ? seqv + 1 : 0;
        LGKM0();
        __builtin_amdgcn_s_barrier();
        ORDER();
        call = rfl(lds[LDS_CNT / 4 + W_CALL]);
        tbar = clk();
        asm volatile("" :: "v"(warm));
        if (wave == 5) {
            const unsigned xcc = __builtin_amdgcn_s_getreg(20 | (3 << 11)) & 15u;
            bool mine = invwg;
            while (!mine) {
                ORDER();
                if (rfl(__builtin_amdgcn_raw_buffer_load_b32(ctrl, 0, OFF_INV + xcc * 128, AUX_SYS)) >= call) break;
                if (__builtin_amdgcn_s_memrealtime() - t0 > 1000ull) mine = true;
                __builtin_amdgcn_s_sleep(1);
            }
            if (mine) {
                if (!invwg) __builtin_amdgcn_fence(__ATOMIC_ACQUIRE, "");
                VMCNT(0);
                if (lane == 0) __builtin_amdgcn_raw_buffer_store_b32(call, ctrl, 0, OFF_INV + xcc * 128, AUX_SYS);
            }
            if (lane == 0) LDS_ST(W_INV, 1u);
        }
    }

    char* const myx = myws + OFF_XREG + reg_off(wave);
    const rsrc_t hdr = mk_rsrc(myx, REG_HDR);
    const unsigned offw = HDR_WFLAG + g * 128, offx = HDR_XFLAG + rb * 128;
    const bool remote = wave != 0;
    {
        unsigned gx = GATE_X + (remote ? (rb >> 2) * GATE_XR : 0);
        if (gx > GATE_W - 30) gx = GATE_W - 30;
        while (clk() - tbar < gx) __builtin_amdgcn_s_sleep(2);
    }
    const unsigned wgate = tbar + (remote ? GATE_W : 0);
    unsigned st = wait_flags(ctrl, hdr, offw, offx, 3u, call, t0, wgate);
    while (rfl(LDS_LD(W_INV)) == 0) __builtin_amdgcn_s_sleep(1);
    ORDER();
    const unsigned doff = reg_off(b & 7) + REG_HDR + (call & 1) * REG_DATA + gh * 131072;
    const rsrc_t rwb = mk_rsrc(wsv + OFF_WREG + doff, 131072);
    const rsrc_t rxb = mk_rsrc(myx + REG_HDR + (call & 1) * REG_DATA + rb * 16384, 16384);
    const unsigned voff = lane * 16;
    const unsigned hg = 1 + (g >= 14 ? g - 14 : g >= 7 ? g - 7 : g);
    auto tile_id = [&](unsigned t) -> unsigned { return ((unsigned)prank + (t == 7 ? hg : t < hg ? t : t + 1)) & 7u; };
    const unsigned o0 = tile_id(0) * 16384;
    u32x4 ring[RING];
    u32x4 xf[16];
    if (st & 2u) {
        if (st & 1u) {
#pragma unroll
            for (int f = 0; f < PULL_D; ++f) ring[f] = __builtin_amdgcn_raw_buffer_load_b128(rwb, voff, o0 + f * 1024, 0);
        }
        ORDER();
#pragma unroll
        for (int kb = 0; kb < 16; ++kb) xf[kb] = __builtin_amdgcn_raw_buffer_load_b128(rxb, voff, kb * 1024, 0);
        if (!(st & 1u)) {
            st = wait_flags(ctrl, hdr, offw, offx, 1u, call, t0, wgate);
            ORDER();
#pragma unroll
            for (int f = 0; f < PULL_D; ++f) ring[f] = __builtin_amdgcn_raw_buffer_load_b128(rwb, voff, o0 + f * 1024, 0);
        }
#pragma unroll
        for (int f = PULL_D; f < RING; ++f) {
            VMCNT(PULL_D - 1);
            ring[f] = __builtin_amdgcn_raw_buffer_load_b128(rwb, voff, o0 + f * 1024, 0);
        }
    } else {
#pragma unroll
        for (int f = 0; f < RING; ++f) {
            if (f >= PULL_D) VMCNT(PULL_D - 1);
            ring[f] = __builtin_amdgcn_raw_buffer_load_b128(rwb, voff, o0 + f * 1024, 0);
        }
        st = wait_flags(ctrl, hdr, offw, offx, 2u, call, t0, tbar);
        ORDER();
#pragma unroll
        for (int kb = 0; kb < 16; ++kb) xf[kb] = __builtin_amdgcn_raw_buffer_load_b128(rxb, voff, kb * 1024, 0);
    }
    if ((st & 4u) && lane == 0) LDS_ST(W_BAD, 1u);

    const unsigned lcol = 4 * (lane & 7) + ((lane >> 3) & 3);
    unsigned* const lacc = &lds[LDS_ACC / 4 + (lane >> 5) * 128 + lcol];

    unsigned lvl = remote ? 1u : 3u;
    rsrc_t rl = rwb;
    unsigned ol = tile_id(7) * 16384;
    int nmoved = 0;
    auto tile = [&](unsigned t, bool last) __attribute__((always_inline)) {
        i32x16 acc = {0, 0, 0, 0, 0, 0, 0, 0, 0, 0, 0, 0, 0, 0, 0, 0};
        const unsigned ocur = tile_id(t) * 16384;
        unsigned onext = tile_id(t + 1) * 16384;
        rsrc_t rnext = rwb;
#pragma unroll
        for (int kb = 0; kb < 16; ++kb) {
            const bool more = !last || kb + RING < 16;
            if (!last && kb == 16 - RING && t == 6) {
                if ((rfl(LDS_LD(W_PLAN)) >> wave) & 1u) {
                    rl = mk_rsrc(P.ws[hg] + OFF_CREG + doff, 131072);
                    ol = ((wave - hg) & 7u) * 16384;
                }
                nmoved = (int)rfl(LDS_LD(W_MOVED + wave));
                rnext = rl;
                onext = ol;
            }
            if (more && remote && (kb & 1) == 0) {
                const unsigned pos = t * 16 + kb;
                const unsigned tref = rfl(LDS_LD(W_TREF + wave));
                const unsigned now = clk();
                if (kb == (BAL_POS & 15) && t == (BAL_POS >> 4) && rb < 4 && lane == 0)
                    __builtin_amdgcn_raw_buffer_store_b32(now, ctrl, 0, OFF_PROG + wave * 256 + g * 16 + rb * 4, AUX_SYS);
                int behind = (int)(now - tref) - (((int)pos - BAL_POS) * BAL_TAU >> 4);
                if (last || kb >= 16 - RING) behind += nmoved * ((last ? kb + RING : kb - (16 - RING)) + 8) * BAL_TAU >> 8;
                lvl = tref == 0 ? 1u : behind >= 3 * BAL_T ? 3u : behind >= BAL_T ? 2u : behind <= -BAL_T ? 0u : 1u;
            }
            if (more) {
                if (lvl <= 2) VMCNT(BOOST_D - 1);
                if (lvl <= 1) VMCNT(PULL_D - 1);
                if (lvl == 0) VMCNT(SLOW_D - 1);
            }
            __builtin_amdgcn_sched_barrier(0);
            acc = __builtin_amdgcn_mfma_i32_32x32x32_i8((i32x4)xf[kb], (i32x4)ring[kb % RING], acc, 0, 0, 0);
            __builtin_amdgcn_sched_barrier(0);
            if (more) {
                if (kb + RING < 16) ring[kb % RING] = __builtin_amdgcn_raw_buffer_load_b128(last ? rl : rwb, voff, (last ? ol : ocur) + (kb + RING) * 1024, 0);
                else ring[kb % RING] = __builtin_amdgcn_raw_buffer_load_b128(rnext, voff, onext + (kb + RING - 16) * 1024, 0);
            }
            __builtin_amdgcn_sched_barrier(0);
        }
#pragma unroll
        for (int r = 0; r < 16; ++r)
            __hip_atomic_fetch_add(lacc + t * 1024 + ((r & 3) + 8 * (r >> 2)) * 32, (unsigned)acc[r],
                                   __ATOMIC_RELAXED, __HIP_MEMORY_SCOPE_WORKGROUP);
        LGKM0();
        if (lane == 0) __hip_atomic_fetch_add(&lds[LDS_CNT / 4 + t], 1u, __ATOMIC_RELAXED, __HIP_MEMORY_SCOPE_WORKGROUP);
    };

    bool need_copy = wave == 0 && rb >= 1 && rb <= 7 && rb != 8 - hg;
    bool need_plan = wave == 0;
    unsigned want = 0;
    auto side_jobs = [&]() __attribute__((always_inline)) {
        if (need_copy && rfl(LDS_LD(0)) >= 8u) {
            need_copy = false;
            if (rfl(LDS_LD(W_BAD)) == 0) {
                ORDER();
                const rsrc_t rs = mk_rsrc(P.ws[rb & 7] + OFF_WREG + doff + o0, 16384);
                const rsrc_t rc = mk_rsrc(myws + OFF_CREG + doff + (rb & 7) * 16384, 16384);
                u32x4 c[16];
#pragma unroll
                for (int i = 0; i < 16; ++i) c[i] = __builtin_amdgcn_raw_buffer_load_b128(rs, voff, i * 1024, 0);
#pragma unroll
                for (int i = 0; i < 16; ++i) __builtin_amdgcn_raw_buffer_store_b128(c[i], rc, voff, i * 1024, AUX_SYS);
                VMCNT(0);
                if (lane == 0)
                    __builtin_amdgcn_raw_buffer_store_b32(call, mk_rsrc(P.ws[8 - hg] + OFF_XREG + reg_off(hg) + HDR_CFLAG, 2048),
                                                          0, g * 128 + ((rb + hg) & 7u) * 4, AUX_SYS);
            }
        }
        if (need_plan && rfl(LDS_LD(1)) >= 8u) {
            ORDER();
            const u32x4 qa = __builtin_amdgcn_raw_buffer_load_b128(ctrl, voff, OFF_PROG, AUX_SYS) - (unsigned)t0;
            const u32x4 qb = __builtin_amdgcn_raw_buffer_load_b128(ctrl, voff, OFF_PROG + 1024, AUX_SYS) - (unsigned)t0;
            int sa = (int)min(min(qa.x, qa.y), min(qa.z, qa.w)), sb = (int)min(min(qb.x, qb.y), min(qb.z, qb.w));
            if (__all((lane < 16 || (unsigned)sa < 10000u) && (unsigned)sb < 10000u)) {
                need_plan = false;
                sa = row_sum(sa); sb = row_sum(sb);
                int S = 0;
#pragma unroll
                for (int v = 1; v < 8; ++v) {
                    const int sv = __builtin_amdgcn_readlane(v < 4 ? sa : sb, (v & 3) * 16);
                    S = lane == v ? sv : S;
                }
                if (lane >= 1 && lane < 8) LDS_ST(W_TREF + lane, ((unsigned)t0 + ((unsigned)S >> 4)) | 1u);
                unsigned used = 0;
                unsigned nv = 0;
#pragma unroll 1
                for (int pass = 0; pass < 5; ++pass) {
                    int mx = __builtin_amdgcn_readlane(S, 1), mn = mx;
#pragma unroll
                    for (int v = 2; v < 8; ++v) { const int sv = __builtin_amdgcn_readlane(S, v); mx = max(mx, sv); mn = min(mn, sv); }
                    if (mx - mn < MOVE_MIN) break;
                    bool moved = false;
                    unsigned hh = 1;
#pragma unroll 1
                    for (unsigned gg = 0; gg < 16; ++gg) {
                        const unsigned um = __builtin_amdgcn_readlane(used, gg) | (1u << hh) | 1u;
                        int key = ((um >> (lane & 7)) & 1u) ? -1 : (S * 8 + lane);
                        key = max(key, __builtin_amdgcn_update_dpp(key, key, 0xB1, 0xf, 0xf, false));
                        key = max(key, __builtin_amdgcn_update_dpp(key, key, 0x4E, 0xf, 0xf, false));
                        const int top = max((int)__builtin_amdgcn_readlane(key, 0), (int)__builtin_amdgcn_readlane(key, 4));
                        const int sh = __builtin_amdgcn_readlane(S, hh);
                        if (top >= 0 && (top >> 3) - sh >= MOVE_MIN) {
                            const unsigned vv = top & 7;
                            if ((unsigned)lane == vv) { S -= BAL_TAU; ++nv; }
                            if ((unsigned)lane == hh) S += BAL_TAU;
                            if ((unsigned)lane == gg) used |= 1u << vv;
                            moved = true;
                        }
                        hh = hh == 7 ? 1 : hh + 1;
                    }
                    if (!moved) break;
                }
                want = __builtin_amdgcn_readlane(used, g);
                if (lane < 8) LDS_ST(W_MOVED + lane, nv);
                if (want) {
                    const unsigned wv = __builtin_amdgcn_raw_buffer_load_b32(mk_rsrc(P.ws[hg] + OFF_CREG + reg_off(b & 7) + HDR_WARM, 4), 0, 0, 0);
                    asm volatile("" :: "v"(wv));
                }
            }
        }
        if (want) {
            ORDER();
            const unsigned cf = __builtin_amdgcn_raw_buffer_load_b32(mk_rsrc(myws + OFF_XREG + reg_off(hg) + HDR_CFLAG + g * 128, 32), lane * 4, 0, AUX_SYS);
            const unsigned got = want & (unsigned)__ballot(lane < 8 && cf >= call);
            want &= ~got;
            if (lane == 0) __hip_atomic_fetch_or(&lds[LDS_CNT / 4 + W_PLAN], got, __ATOMIC_RELAXED, __HIP_MEMORY_SCOPE_WORKGROUP);
        }
    };
#pragma unroll 1
    for (unsigned t = 0; t < 7; ++t) {
        tile(t, false);
        if (wave == 0) side_jobs();
    }
    tile(7, true);

    if (wave == 0) {
        if (lane == 0) __builtin_amdgcn_raw_buffer_store_b32(call, ctrl, 0, OFF_SEQ + b * 128, AUX_SYS);
        const float s = P.sx[0] * P.sw[0];
        const rsrc_t ro = mk_rsrc(P.out + (size_t)rb * 32 * NN + g * 256, (31 * NN + 256) * 4);
        const unsigned ooff = (lane >> 3) * (NN * 4) + (lane & 7) * 16;
#pragma unroll 1
        for (unsigned t = 0; t < 8; ++t) {
            while (rfl(LDS_LD(t)) < 8u) {
                if (t < 7) {
                    side_jobs();
                    if ((need_plan && t > 1) || want) __builtin_amdgcn_s_sleep(64);
                    __builtin_amdgcn_s_sleep(4);
                }
            }
            ORDER();
            const bool bad = LDS_LD(W_BAD) != 0;
            i32x4 a[4];
            f32x4 y[4];
#pragma unroll
            for (int q = 0; q < 4; ++q) a[q] = *(const i32x4*)&lds[LDS_ACC / 4 + t * 1024 + q * 256 + lane * 4];
#pragma unroll
            for (int q = 0; q < 4; ++q)
#pragma unroll
                for (int e = 0; e < 4; ++e) y[q][e] = bad ? __builtin_nanf("") : fmaxf((float)a[q][e] * s, 0.0f);
            __builtin_amdgcn_sched_barrier(0);
#pragma unroll
            for (int q = 0; q < 4; ++q)
                __builtin_amdgcn_raw_buffer_store_b128((u32x4)y[q], ro, ooff, q * 8 * (NN * 4) + tile_id(t) * 128, AUX_SYS);
            __builtin_amdgcn_sched_barrier(0);
            asm volatile("s_nop 1");
        }
    }
}

}

extern "C" void kernel_launch(int rank, int nranks, ncclComm_t comm,
                              hipStream_t stream,
                              void* const* d_in, const int* in_sizes, int n_in,
                              void* d_out, int out_size,
                              const int* in_dtypes, int out_dtype,
                              void* const* d_out_all,
                              void* const* d_ws_all, size_t ws_bytes) {
    if (nranks != NR || (unsigned)rank >= (unsigned)NR || n_in < 4 || ws_bytes < WS_NEED ||
        out_size != MS * NN || in_sizes[0] != 4096 * KS || in_sizes[1] != KS * NN) {
        fprintf(stderr, "unexpected problem shape\n");
        return;
    }
    Params P;
    P.x = (const int8_t*)d_in[0];
    P.w = (const int8_t*)d_in[1];
    P.sx = (const float*)d_in[2];
    P.sw = (const float*)d_in[3];
    P.out = (float*)d_out;
    for (int j = 0; j < NR; ++j) P.ws[j] = (char*)d_ws_all[(rank + j) % NR];
    P.rank = rank;
    gemm_rs_fused<<<NWG, 512, 0, stream>>>(P);
    HIP_CHECK(hipGetLastError());
}
```

```cpp
#include <hip/hip_runtime.h>
#include <rccl/rccl.h>
#include <cstdio>
#include <cstdint>

#define HIP_CHECK(expr) do { hipError_t _e = (expr); if (_e != hipSuccess) { \
    fprintf(stderr, "HIP error %s at %s:%d\n", hipGetErrorString(_e), __FILE__, __LINE__); \
    return; } } while (0)

namespace {

typedef int i32x4 __attribute__((ext_vector_type(4)));
typedef unsigned u32x4 __attribute__((ext_vector_type(4)));
typedef unsigned u32x2 __attribute__((ext_vector_type(2)));
typedef int i32x16 __attribute__((ext_vector_type(16)));
typedef float f32x4 __attribute__((ext_vector_type(4)));
typedef __amdgpu_buffer_rsrc_t rsrc_t;
typedef __attribute__((address_space(1))) unsigned gu32;

constexpr int NR = 8;
constexpr int MS = 512, KS = 512;
constexpr int NN = 4096;
constexpr int NWG = 256;

constexpr unsigned REG_HDR = 8192, REG_DATA = 262144, REG_BYTES = REG_HDR + 2 * REG_DATA;
constexpr unsigned HDR_XFLAG = 0, HDR_WFLAG = 2048, HDR_CFLAG = 4096, HDR_WARM = 6144;
constexpr unsigned OFF_XREG = 0, OFF_WREG = 6u << 20, OFF_CREG = 12u << 20;
__host__ __device__ constexpr unsigned reg_off(unsigned i) {
    return ((i * 11u) >> 5) * (2u << 20) + (i - ((i * 11u) >> 5) * 3u) * REG_BYTES;
}
constexpr unsigned OFF_CTRL  = 0x190000;
constexpr unsigned OFF_SEQ   = 0;
constexpr unsigned OFF_WCNT  = 32768;
constexpr unsigned OFF_TMO   = 36864;
constexpr unsigned OFF_INV   = 40960;
constexpr unsigned OFF_PROG  = 45056;
constexpr unsigned CTRL_BYTES = 65536;
constexpr size_t   WS_NEED   = 18u << 20;
static_assert(3 * REG_BYTES <= OFF_CTRL && OFF_CTRL + CTRL_BYTES <= (2u << 20), "layout");
static_assert(OFF_CREG + reg_off(7) + REG_BYTES <= WS_NEED, "layout");

constexpr int RING = 8;
constexpr int PULL_D = 4, BOOST_D = 5, SLOW_D = 3;
constexpr int XWIN = 10;
constexpr unsigned GATE_W = 400;
constexpr unsigned GATE_X = 200, GATE_XR = 100;
constexpr int BAL_POS = 24;
constexpr int BAL_TAU = 438;
constexpr int BAL_T = 30;
constexpr unsigned long long DEADLINE_TICKS = 100000000ull;

constexpr int AUX_SYS = 17;

struct Params {
    const int8_t* x;
    const int8_t* w;
    const float* sx;
    const float* sw;
    float* out;
    char* ws[NR];
    int rank;
};

#define VMCNT(n) asm volatile("s_waitcnt vmcnt(%0)" :: "i"(n) : "memory")
#define LGKM0() asm volatile("s_waitcnt lgkmcnt(0)" ::: "memory")
#define ORDER() asm volatile("" ::: "memory")
#define LDS_LD(i) __hip_atomic_load(&lds[LDS_CNT / 4 + (i)], __ATOMIC_RELAXED, __HIP_MEMORY_SCOPE_WORKGROUP)
#define LDS_ST(i, v) __hip_atomic_store(&lds[LDS_CNT / 4 + (i)], (v), __ATOMIC_RELAXED, __HIP_MEMORY_SCOPE_WORKGROUP)

__device__ __forceinline__ rsrc_t mk_rsrc(const void* p, unsigned bytes) {
    return __builtin_amdgcn_make_buffer_rsrc(const_cast<void*>(p), 0, bytes, 0x00020000);
}
__device__ __forceinline__ unsigned rfl(unsigned v) { return __builtin_amdgcn_readfirstlane(v); }
__device__ __forceinline__ unsigned clk() { return (unsigned)__builtin_amdgcn_s_memrealtime(); }

__device__ __forceinline__ int row_sum(int v) {
    v += __builtin_amdgcn_update_dpp(0, v, 0xB1, 0xf, 0xf, false);
    v += __builtin_amdgcn_update_dpp(0, v, 0x4E, 0xf, 0xf, false);
    v += __builtin_amdgcn_update_dpp(0, v, 0x141, 0xf, 0xf, false);
    v += __builtin_amdgcn_update_dpp(0, v, 0x140, 0xf, 0xf, false);
    return v;
}

__device__ __forceinline__ unsigned wait_flags(rsrc_t ctrl, rsrc_t hdr, unsigned offw, unsigned offx, unsigned need,
                                               unsigned call, unsigned long long t0, unsigned wgate) {
    unsigned spins = 0;
    for (;;) {
        ORDER();
        const unsigned wv = __builtin_amdgcn_raw_buffer_load_b32(hdr, 0, offw, AUX_SYS);
        const u32x2 xv = __builtin_amdgcn_raw_buffer_load_b64(hdr, 0, offx, AUX_SYS);
        const unsigned a = rfl(xv.x), c = rfl(xv.y);
        const bool open = (int)(clk() - wgate) >= 0;
        const unsigned ready = ((rfl(wv) >= call && open) ? 1u : 0u) | ((a >= call && c >= call) ? 2u : 0u);
        if (ready & need) return ready;
        if (++spins > 256) {
            const unsigned tmo = rfl(__builtin_amdgcn_raw_buffer_load_b32(ctrl, 0, OFF_TMO, AUX_SYS));
            if (tmo == call || __builtin_amdgcn_s_memrealtime() - t0 > DEADLINE_TICKS) {
                __builtin_amdgcn_raw_buffer_store_b32(call, ctrl, 0, OFF_TMO, AUX_SYS);
                return 7u;
            }
            __builtin_amdgcn_s_sleep(8);
        }
        __builtin_amdgcn_s_sleep(1);
    }
}

constexpr int LDS_ACC = 0;
constexpr int LDS_CNT = 32768;
constexpr int W_INV = 8, W_CALL = 10, W_PLAN = 13, W_SCALE = 14, W_BAD = 15, W_TREF = 16, W_MOVED = 24;
constexpr int LDS_BYTES = 84 * 1024;

#define LDS_INIT() do { \
    *(u32x4*)&lds[tid * 16] = u32x4{0, 0, 0, 0}; *(u32x4*)&lds[tid * 16 + 4] = u32x4{0, 0, 0, 0}; \
    *(u32x4*)&lds[tid * 16 + 8] = u32x4{0, 0, 0, 0}; *(u32x4*)&lds[tid * 16 + 12] = u32x4{0, 0, 0, 0}; } while (0)

__global__ void __launch_bounds__(512, 2) gemm_rs_fused(Params P) {
    __shared__ __attribute__((aligned(16))) unsigned lds[LDS_BYTES / 4];
    const int tid = threadIdx.x, lane = tid & 63;
    const unsigned wave = rfl(tid >> 6) & 7u;
    const unsigned b = blockIdx.x & (NWG - 1);
    const unsigned idx = b >> 3;
    const unsigned gh = idx & 1;
    const unsigned rb = 15 - (idx >> 1);
    const unsigned g = (b & 7) * 2 + gh;
    const bool invwg = idx == 8;
    if (wave == 5 && invwg) __builtin_amdgcn_fence(__ATOMIC_ACQUIRE, "");
    const unsigned pj = gh * 4 + (rb & 3);
    unsigned long long t0 = __builtin_amdgcn_s_memrealtime();
    char* myws = P.ws[0];
    char* wsv = P.ws[wave];
    char* wsj = P.ws[pj];
    const int8_t* px = P.x;
    const int8_t* pw = P.w;
    int prank = P.rank;
    asm volatile("" : "+s"(myws), "+s"(wsv), "+s"(wsj), "+s"(px), "+s"(pw), "+s"(prank), "+s"(t0));
    const rsrc_t ctrl = mk_rsrc(myws + OFF_CTRL, CTRL_BYTES);

    const bool pusher = (wave == 0) && (rb >= 12);
    const bool stager = (wave >= 1) && (wave <= 4) && (rb < 8);
    char* const xr = wsj + OFF_XREG + reg_off((8 - pj) & 7);
    unsigned call, tbar;

    if (__builtin_expect(pusher, 1)) {
        const unsigned q = b & 7;
        const unsigned dst = ((unsigned)prank + pj) & 7u;
        const rsrc_t rx = mk_rsrc(px + (size_t)dst * MS * KS, MS * KS);
        const unsigned row = lane & 7, c = lane >> 3;
        u32x4 v[32];
        unsigned seqv = 0;
#pragma unroll
        for (int r = 0; r < 4; ++r) {
#pragma unroll
            for (int e = 0; e < 8; ++e) {
                const unsigned unit = 4 * r + (q >> 1);
                const unsigned soff = unit * 16384 + ((q & 1) * 2 + (e >> 2)) * 128 + (e & 3) * 4096;
                v[r * 8 + e] = __builtin_amdgcn_raw_buffer_load_b128(rx, row * 512 + c * 16, soff, 0);
                if (r == 0 && e == 0) {
                    ORDER();
                    seqv = __builtin_amdgcn_raw_buffer_load_b32(ctrl, 0, OFF_SEQ + b * 128, AUX_SYS);
                    ORDER();
                }
            }
        }
        LDS_INIT();
        call = rfl(seqv) + 1;
        tbar = (unsigned)t0;
        const rsrc_t rd = mk_rsrc(xr + REG_HDR + (call & 1) * REG_DATA, REG_DATA);
        const rsrc_t rf = mk_rsrc(xr + HDR_XFLAG, 2048);
#pragma unroll
        for (int s = 0; s < 32 + XWIN; ++s) {
            if (s == XWIN) {
                if (lane < 32) lds[LDS_CNT / 4 + lane] = lane == W_CALL ? call : 0;
                LGKM0();
                __builtin_amdgcn_s_barrier();
            }
            if (s >= XWIN) {
                const int done = s - XWIN;
                int younger = (s < 32 ? s : 32) - 1 - done;
                for (int r = 0; r < 4; ++r) { const int it = 8 * r + 7 + XWIN; if (it < s && it > done) ++younger; }
                VMCNT(younger);
                if ((done & 7) == 7) {
                    const unsigned unit = 4 * (done >> 3) + (q >> 1);
                    if (lane == 0) __builtin_amdgcn_raw_buffer_store_b32(call, rf, 0, unit * 128 + (q & 1) * 4, AUX_SYS);
                }
            }
            if (s < 32) {
                const int r = s >> 3, e = s & 7;
                const unsigned unit = 4 * r + (q >> 1);
                const unsigned soff = unit * 16384 + ((q & 1) * 2 + (e >> 2)) * 4096 + (e & 3) * 128;
                __builtin_amdgcn_raw_buffer_store_b128(v[s], rd, c * 512 + row * 16, soff, AUX_SYS);
            }
        }
    } else if (stager) {
        const unsigned h = (wave - 1) & 1, skb = rb * 2 + ((wave - 1) >> 1);
        const rsrc_t rw = mk_rsrc(pw, KS * NN);
        unsigned d[16];
#pragma unroll
        for (int r = 0; r < 16; ++r)
            d[r] = __builtin_amdgcn_raw_buffer_load_b32(rw, lane * 4, (skb * 32 + h * 16 + r) * NN + g * 256, 0);
        LDS_INIT();
        LGKM0();
        __builtin_amdgcn_s_barrier();
        ORDER();
        call = rfl(lds[LDS_CNT / 4 + W_CALL]);
        tbar = clk();
        u32x4 o[4];
#pragma unroll
        for (int a = 0; a < 4; ++a) {
            const unsigned p0 = __builtin_amdgcn_perm(d[4 * a + 1], d[4 * a], 0x05010400u);
            const unsigned p1 = __builtin_amdgcn_perm(d[4 * a + 1], d[4 * a], 0x07030602u);
            const unsigned p2 = __builtin_amdgcn_perm(d[4 * a + 3], d[4 * a + 2], 0x05010400u);
            const unsigned p3 = __builtin_amdgcn_perm(d[4 * a + 3], d[4 * a + 2], 0x07030602u);
            o[0][a] = __builtin_amdgcn_perm(p2, p0, 0x05040100u);
            o[1][a] = __builtin_amdgcn_perm(p2, p0, 0x07060302u);
            o[2][a] = __builtin_amdgcn_perm(p3, p1, 0x05040100u);
            o[3][a] = __builtin_amdgcn_perm(p3, p1, 0x07060302u);
        }
        const rsrc_t rd = mk_rsrc(myws + OFF_WREG + reg_off(b & 7) + REG_HDR + (call & 1) * REG_DATA + gh * 131072, 131072);
#pragma unroll
        for (int j = 0; j < 4; ++j)
            __builtin_amdgcn_raw_buffer_store_b128(o[j], rd, (lane >> 3) * 16384 + (lane & 7) * 16,
                                                   skb * 1024 + h * 512 + j * 128, AUX_SYS);
        VMCNT(0);
        unsigned tk = 0;
        if (lane == 0)
            tk = __hip_atomic_fetch_add((gu32*)(myws + OFF_CTRL + OFF_WCNT + g * 128), 1u, __ATOMIC_RELAXED, __HIP_MEMORY_SCOPE_AGENT);
        if (rfl(tk) + 1 == call * 32u && lane == 0) {
#pragma unroll
            for (int jj = 1; jj <= NR; ++jj)
                __builtin_amdgcn_raw_buffer_store_b32(call, mk_rsrc(P.ws[jj & 7] + OFF_XREG + reg_off((8 - jj) & 7) + HDR_WFLAG, 2048),
                                                      0, g * 128, AUX_SYS);
        }
    } else {
        unsigned warm = 0;
        if (rb >= 8 && rb < 12 && wave < 2) {
            const char* wp = wave == 0 ? xr + HDR_WARM : wsj + OFF_WREG + reg_off(b & 7) + HDR_WARM;
            if (wave != 0) __builtin_amdgcn_s_sleep(2);
            warm = __builtin_amdgcn_raw_buffer_load_b32(mk_rsrc(wp, 4), 0, 0, AUX_SYS);
        }
        unsigned seqv = 0;
        if (wave == 0) seqv = __builtin_amdgcn_raw_buffer_load_b32(ctrl, 0, OFF_SEQ + b * 128, AUX_SYS);
        LDS_INIT();
        if (wave == 0 && lane < 32) lds[LDS_CNT / 4 + lane] = lane == W_CALL ? seqv + 1 : 0;
        LGKM0();
        __builtin_amdgcn_s_barrier();
        ORDER();
        call = rfl(lds[LDS_CNT / 4 + W_CALL]);
        tbar = clk();
        asm volatile("" :: "v"(warm));
        if (wave == 5) {
            const unsigned xcc = __builtin_amdgcn_s_getreg(20 | (3 << 11)) & 15u;
            bool mine = invwg;
            while (!mine) {
                ORDER();
                if (rfl(__builtin_amdgcn_raw_buffer_load_b32(ctrl, 0, OFF_INV + xcc * 128, AUX_SYS)) >= call) break;
                if (__builtin_amdgcn_s_memrealtime() - t0 > 1000ull) mine = true;
                __builtin_amdgcn_s_sleep(1);
            }
            if (mine) {
                if (!invwg) __builtin_amdgcn_fence(__ATOMIC_ACQUIRE, "");
                VMCNT(0);
                if (lane == 0) __builtin_amdgcn_raw_buffer_store_b32(call, ctrl, 0, OFF_INV + xcc * 128, AUX_SYS);
            }
            if (lane == 0) LDS_ST(W_INV, 1u);
        }
    }

    char* const myx = myws + OFF_XREG + reg_off(wave);
    const rsrc_t hdr = mk_rsrc(myx, REG_HDR);
    const unsigned offw = HDR_WFLAG + g * 128, offx = HDR_XFLAG + rb * 128;
    const bool remote = wave != 0;
    {
        unsigned gx = GATE_X + (remote ? (rb >> 2) * GATE_XR : 0);
        if (gx > GATE_W - 30) gx = GATE_W - 30;
        while (clk() - tbar < gx) __builtin_amdgcn_s_sleep(2);
    }
    const unsigned wgate = tbar + (remote ? GATE_W : 0);
    unsigned st = wait_flags(ctrl, hdr, offw, offx, 3u, call, t0, wgate);
    while (rfl(LDS_LD(W_INV)) == 0) __builtin_amdgcn_s_sleep(1);
    ORDER();
    const unsigned doff = reg_off(b & 7) + REG_HDR + (call & 1) * REG_DATA + gh * 131072;
    const rsrc_t rwb = mk_rsrc(wsv + OFF_WREG + doff, 131072);
    const rsrc_t rxb = mk_rsrc(myx + REG_HDR + (call & 1) * REG_DATA + rb * 16384, 16384);
    const unsigned voff = lane * 16;
    const unsigned hg = 1 + (g >= 14 ? g - 14 : g >= 7 ? g - 7 : g);
    auto tile_id = [&](unsigned t) -> unsigned { return ((unsigned)prank + (t == 7 ? hg : t < hg ? t : t + 1)) & 7u; };
    const unsigned o0 = tile_id(0) * 16384;
    u32x4 ring[RING];
    u32x4 xf[16];
    if (st & 2u) {
        if (st & 1u) {
#pragma unroll
            for (int f = 0; f < PULL_D; ++f) ring[f] = __builtin_amdgcn_raw_buffer_load_b128(rwb, voff, o0 + f * 1024, 0);
        }
        ORDER();
#pragma unroll
        for (int kb = 0; kb < 16; ++kb) xf[kb] = __builtin_amdgcn_raw_buffer_load_b128(rxb, voff, kb * 1024, 0);
        if (!(st & 1u)) {
            st = wait_flags(ctrl, hdr, offw, offx, 1u, call, t0, wgate);
            ORDER();
#pragma unroll
            for (int f = 0; f < PULL_D; ++f) ring[f] = __builtin_amdgcn_raw_buffer_load_b128(rwb, voff, o0 + f * 1024, 0);
        }
#pragma unroll
        for (int f = PULL_D; f < RING; ++f) {
            VMCNT(PULL_D - 1);
            ring[f] = __builtin_amdgcn_raw_buffer_load_b128(rwb, voff, o0 + f * 1024, 0);
        }
    } else {
#pragma unroll
        for (int f = 0; f < RING; ++f) {
            if (f >= PULL_D) VMCNT(PULL_D - 1);
            ring[f] = __builtin_amdgcn_raw_buffer_load_b128(rwb, voff, o0 + f * 1024, 0);
        }
        st = wait_flags(ctrl, hdr, offw, offx, 2u, call, t0, tbar);
        ORDER();
#pragma unroll
        for (int kb = 0; kb < 16; ++kb) xf[kb] = __builtin_amdgcn_raw_buffer_load_b128(rxb, voff, kb * 1024, 0);
    }
    if ((st & 4u) && lane == 0) LDS_ST(W_BAD, 1u);

    const unsigned lcol = 4 * (lane & 7) + ((lane >> 3) & 3);
    unsigned* const lacc = &lds[LDS_ACC / 4 + (lane >> 5) * 128 + lcol];

    unsigned lvl = remote ? 1u : 3u;
    rsrc_t rl = rwb;
    unsigned ol = tile_id(7) * 16384;
    int nmoved = 0;
    auto tile = [&](unsigned t, bool last) __attribute__((always_inline)) {
        i32x16 acc = {0, 0, 0, 0, 0, 0, 0, 0, 0, 0, 0, 0, 0, 0, 0, 0};
        const unsigned ocur = tile_id(t) * 16384;
        unsigned onext = tile_id(t + 1) * 16384;
        rsrc_t rnext = rwb;
#pragma unroll
        for (int kb = 0; kb < 16; ++kb) {
            const bool more = !last || kb + RING < 16;
            if (!last && kb == 16 - RING && t == 6) {
                if ((rfl(LDS_LD(W_PLAN)) >> wave) & 1u) {
                    rl = mk_rsrc(P.ws[hg] + OFF_CREG + doff, 131072);
                    ol = ((wave - hg) & 7u) * 16384;
                }
                nmoved = (int)rfl(LDS_LD(W_MOVED + wave));
                rnext = rl;
                onext = ol;
            }
            if (more && remote && (kb & 1) == 0) {
                const unsigned pos = t * 16 + kb;
                const unsigned tref = rfl(LDS_LD(W_TREF + wave));
                const unsigned now = clk();
                if (kb == (BAL_POS & 15) && t == (BAL_POS >> 4) && rb < 4 && lane == 0)
                    __builtin_amdgcn_raw_buffer_store_b32(now, ctrl, 0, OFF_PROG + wave * 256 + g * 16 + rb * 4, AUX_SYS);
                int behind = (int)(now - tref) - (((int)pos - BAL_POS) * BAL_TAU >> 4);
                if (last || kb >= 16 - RING) behind += nmoved * ((last ? kb + RING : kb - (16 - RING)) + 8) * BAL_TAU >> 8;
                lvl = tref == 0 ? 1u : behind >= 3 * BAL_T ? 3u : behind >= BAL_T ? 2u : behind <= -BAL_T ? 0u : 1u;
            }
            if (more) {
                if (lvl <= 2) VMCNT(BOOST_D - 1);
                if (lvl <= 1) VMCNT(PULL_D - 1);
                if (lvl == 0) VMCNT(SLOW_D - 1);
            }
            __builtin_amdgcn_sched_barrier(0);
            acc = __builtin_amdgcn_mfma_i32_32x32x32_i8((i32x4)xf[kb], (i32x4)ring[kb % RING], acc, 0, 0, 0);
            __builtin_amdgcn_sched_barrier(0);
            if (more) {
                if (kb + RING < 16) ring[kb % RING] = __builtin_amdgcn_raw_buffer_load_b128(last ? rl : rwb, voff, (last ? ol : ocur) + (kb + RING) * 1024, 0);
                else ring[kb % RING] = __builtin_amdgcn_raw_buffer_load_b128(rnext, voff, onext + (kb + RING - 16) * 1024, 0);
            }
            __builtin_amdgcn_sched_barrier(0);
        }
#pragma unroll
        for (int r = 0; r < 16; ++r)
            __hip_atomic_fetch_add(lacc + t * 1024 + ((r & 3) + 8 * (r >> 2)) * 32, (unsigned)acc[r],
                                   __ATOMIC_RELAXED, __HIP_MEMORY_SCOPE_WORKGROUP);
        LGKM0();
        if (!last && lane == 0) __hip_atomic_fetch_add(&lds[LDS_CNT / 4 + t], 1u, __ATOMIC_RELAXED, __HIP_MEMORY_SCOPE_WORKGROUP);
    };

    bool need_copy = wave == 0 && rb >= 1 && rb <= 7 && rb != 8 - hg;
    bool need_plan = wave == 0;
    unsigned want = 0;
    auto side_jobs = [&]() __attribute__((always_inline)) {
        if (need_copy && rfl(LDS_LD(0)) >= 8u) {
            need_copy = false;
            if (rfl(LDS_LD(W_BAD)) == 0) {
                ORDER();
                const rsrc_t rs = mk_rsrc(P.ws[rb & 7] + OFF_WREG + doff + o0, 16384);
                const rsrc_t rc = mk_rsrc(myws + OFF_CREG + doff + (rb & 7) * 16384, 16384);
                u32x4 c[16];
#pragma unroll
                for (int i = 0; i < 16; ++i) c[i] = __builtin_amdgcn_raw_buffer_load_b128(rs, voff, i * 1024, 0);
#pragma unroll
                for (int i = 0; i < 16; ++i) __builtin_amdgcn_raw_buffer_store_b128(c[i], rc, voff, i * 1024, AUX_SYS);
                VMCNT(0);
                if (lane == 0)
                    __builtin_amdgcn_raw_buffer_store_b32(call, mk_rsrc(P.ws[8 - hg] + OFF_XREG + reg_off(hg) + HDR_CFLAG, 2048),
                                                          0, g * 128 + ((rb + hg) & 7u) * 4, AUX_SYS);
            }
        }
        if (need_plan && rfl(LDS_LD(1)) >= 8u) {
            ORDER();
            const u32x4 qa = __builtin_amdgcn_raw_buffer_load_b128(ctrl, voff, OFF_PROG, AUX_SYS) - (unsigned)t0;
            const u32x4 qb = __builtin_amdgcn_raw_buffer_load_b128(ctrl, voff, OFF_PROG + 1024, AUX_SYS) - (unsigned)t0;
            int sa = (int)min(min(qa.x, qa.y), min(qa.z, qa.w)), sb = (int)min(min(qb.x, qb.y), min(qb.z, qb.w));
            if (__all((lane < 16 || (unsigned)sa < 10000u) && (unsigned)sb < 10000u)) {
                need_plan = false;
                sa = row_sum(sa); sb = row_sum(sb);
                int S = 0;
#pragma unroll
                for (int v = 1; v < 8; ++v) {
                    const int sv = __builtin_amdgcn_readlane(v < 4 ? sa : sb, (v & 3) * 16);
                    S = lane == v ? sv : S;
                }
                if (lane >= 1 && lane < 8) LDS_ST(W_TREF + lane, ((unsigned)t0 + ((unsigned)S >> 4)) | 1u);
                int tot = 0;
#pragma unroll
                for (int v = 1; v < 8; ++v) tot += __builtin_amdgcn_readlane(S, v);
                const int ex = S * 7 - tot;
                const int shed = (lane >= 1 && lane < 8 && ex > 0) ? (ex + 2 * BAL_TAU) / (7 * BAL_TAU) : 0;
                const int lead = ex < 0 ? -ex : 0;
                int cap = 0;
#pragma unroll
                for (int v = 1; v < 8; ++v) cap += __builtin_amdgcn_readlane(lead, v);
                const float sc = (float)shed / (float)(cap + 1), ph = (float)((lane * 3) & 7) * 0.125f + 0.0625f;
                unsigned nv = 0;
                int cum = 0, prev = 0;
                bool mine = false;
#pragma unroll
                for (int hh = 1; hh < 8; ++hh) {
                    cum += __builtin_amdgcn_readlane(lead, hh);
                    const int up = (int)(sc * (float)cum + ph);
                    const int f = min(up - prev, hh <= 2 ? 3 : 2);
                    prev = up;
                    nv += f;
                    if ((unsigned)hh == hg) mine = (int)(g >= 14 ? 2 : g >= 7 ? 1 : 0) < f;
                }
                want = (unsigned)__ballot(mine) & 0xfeu;
                if (lane < 8) LDS_ST(W_MOVED + lane, nv);
                if (want) {
                    const unsigned wv = __builtin_amdgcn_raw_buffer_load_b32(mk_rsrc(P.ws[hg] + OFF_CREG + reg_off(b & 7) + HDR_WARM, 4), 0, 0, 0);
                    asm volatile("" :: "v"(wv));
                }
            }
        }
        if (want) {
            ORDER();
            const unsigned cf = __builtin_amdgcn_raw_buffer_load_b32(mk_rsrc(myws + OFF_XREG + reg_off(hg) + HDR_CFLAG + g * 128, 32), lane * 4, 0, AUX_SYS);
            const unsigned got = want & (unsigned)__ballot(lane < 8 && cf >= call);
            want &= ~got;
            if (lane == 0) __hip_atomic_fetch_or(&lds[LDS_CNT / 4 + W_PLAN], got, __ATOMIC_RELAXED, __HIP_MEMORY_SCOPE_WORKGROUP);
        }
    };
#pragma unroll 1
    for (unsigned t = 0; t < 7; ++t) {
        tile(t, false);
        if (wave == 0) side_jobs();
    }
    float* pout = P.out + (size_t)(rb * 32 + wave * 4) * NN + g * 256 + tile_id(7) * 32;
    asm volatile("" : "+s"(pout));
    tile(7, true);

    if (wave == 0) {
        if (lane == 0) __builtin_amdgcn_raw_buffer_store_b32(call, ctrl, 0, OFF_SEQ + b * 128, AUX_SYS);
        const float s = P.sx[0] * P.sw[0];
        if (lane == 0) LDS_ST(W_SCALE, __float_as_uint(s));
        const rsrc_t ro = mk_rsrc(P.out + (size_t)rb * 32 * NN + g * 256, (31 * NN + 256) * 4);
        const unsigned ooff = (lane >> 3) * (NN * 4) + (lane & 7) * 16;
#pragma unroll 1
        for (unsigned t = 0; t < 7; ++t) {
            while (rfl(LDS_LD(t)) < 8u) {
                side_jobs();
                if ((need_plan && t > 1) || want) __builtin_amdgcn_s_sleep(64);
                __builtin_amdgcn_s_sleep(4);
            }
            ORDER();
            const bool bad = LDS_LD(W_BAD) != 0;
            i32x4 a[4];
            f32x4 y[4];
#pragma unroll
            for (int q = 0; q < 4; ++q) a[q] = *(const i32x4*)&lds[LDS_ACC / 4 + t * 1024 + q * 256 + lane * 4];
#pragma unroll
            for (int q = 0; q < 4; ++q)
#pragma unroll
                for (int e = 0; e < 4; ++e) y[q][e] = bad ? __builtin_nanf("") : fmaxf((float)a[q][e] * s, 0.0f);
            __builtin_amdgcn_sched_barrier(0);
#pragma unroll
            for (int q = 0; q < 4; ++q)
                __builtin_amdgcn_raw_buffer_store_b128((u32x4)y[q], ro, ooff, q * 8 * (NN * 4) + tile_id(t) * 128, AUX_SYS);
            __builtin_amdgcn_sched_barrier(0);
            asm volatile("s_nop 1");
        }
        LGKM0();
    }
    __builtin_amdgcn_s_barrier();
    ORDER();
    const u32x2 sb = *(const u32x2*)&lds[LDS_CNT / 4 + W_SCALE];
    const u32x2 a = *(const u32x2*)&lds[LDS_ACC / 4 + 7 * 1024 + wave * 128 + lane * 2];
    const float s = __uint_as_float(sb.x);
    u32x2 y;
    y.x = sb.y ? 0x7fc00000u : __float_as_uint(fmaxf((float)(int)a.x * s, 0.0f));
    y.y = sb.y ? 0x7fc00000u : __float_as_uint(fmaxf((float)(int)a.y * s, 0.0f));
    __builtin_amdgcn_raw_buffer_store_b64(y, mk_rsrc(pout, (3 * NN + 32) * 4), (lane >> 4) * (NN * 4) + (lane & 15) * 8, 0, AUX_SYS);
}

}

extern "C" void kernel_launch(int rank, int nranks, ncclComm_t comm,
                              hipStream_t stream,
                              void* const* d_in, const int* in_sizes, int n_in,
                              void* d_out, int out_size,
                              const int* in_dtypes, int out_dtype,
                              void* const* d_out_all,
                              void* const* d_ws_all, size_t ws_bytes) {
    if (nranks != NR || (unsigned)rank >= (unsigned)NR || n_in < 4 || ws_bytes < WS_NEED ||
        out_size != MS * NN || in_sizes[0] != 4096 * KS || in_sizes[1] != KS * NN) {
        fprintf(stderr, "unexpected problem shape\n");
        return;
    }
    Params P;
    P.x = (const int8_t*)d_in[0];
    P.w = (const int8_t*)d_in[1];
    P.sx = (const float*)d_in[2];
    P.sw = (const float*)d_in[3];
    P.out = (float*)d_out;
    for (int j = 0; j < NR; ++j) P.ws[j] = (char*)d_ws_all[(rank + j) % NR];
    P.rank = rank;
    gemm_rs_fused<<<NWG, 512, 0, stream>>>(P);
    HIP_CHECK(hipGetLastError());
}
```

```cpp
#include <hip/hip_runtime.h>
#include <rccl/rccl.h>
#include <cstdio>
#include <cstdint>

#define HIP_CHECK(expr) do { hipError_t _e = (expr); if (_e != hipSuccess) { \
    fprintf(stderr, "HIP error %s at %s:%d\n", hipGetErrorString(_e), __FILE__, __LINE__); \
    return; } } while (0)

namespace {

typedef int i32x4 __attribute__((ext_vector_type(4)));
typedef unsigned u32x4 __attribute__((ext_vector_type(4)));
typedef unsigned u32x2 __attribute__((ext_vector_type(2)));
typedef int i32x16 __attribute__((ext_vector_type(16)));
typedef float f32x4 __attribute__((ext_vector_type(4)));
typedef __amdgpu_buffer_rsrc_t rsrc_t;
typedef __attribute__((address_space(1))) unsigned gu32;

constexpr int NR = 8;
constexpr int MS = 512, KS = 512;
constexpr int NN = 4096;
constexpr int NWG = 256;

constexpr unsigned REG_HDR = 8192, REG_DATA = 262144, REG_BYTES = REG_HDR + 2 * REG_DATA;
constexpr unsigned HDR_XFLAG = 0, HDR_WFLAG = 2048, HDR_CFLAG = 4096, HDR_WARM = 6144;
constexpr unsigned OFF_XREG = 0, OFF_WREG = 6u << 20, OFF_CREG = 12u << 20;
__host__ __device__ constexpr unsigned reg_off(unsigned i) {
    return ((i * 11u) >> 5) * (2u << 20) + (i - ((i * 11u) >> 5) * 3u) * REG_BYTES;
}
constexpr unsigned OFF_CTRL  = 0x190000;
constexpr unsigned OFF_SEQ   = 0;
constexpr unsigned OFF_WCNT  = 32768;
constexpr unsigned OFF_TMO   = 36864;
constexpr unsigned OFF_INV   = 40960;
constexpr unsigned OFF_PROG  = 45056;
constexpr unsigned CTRL_BYTES = 65536;
constexpr size_t   WS_NEED   = 18u << 20;
static_assert(3 * REG_BYTES <= OFF_CTRL && OFF_CTRL + CTRL_BYTES <= (2u << 20), "layout");
static_assert(OFF_CREG + reg_off(7) + REG_BYTES <= WS_NEED, "layout");

constexpr int RING = 8;
constexpr int PULL_D = 4, BOOST_D = 5, SLOW_D = 3;
constexpr int XWIN = 10;
constexpr unsigned GATE_W = 400;
constexpr unsigned GATE_X = 200, GATE_XR = 100;
constexpr int BAL_POS = 24;
constexpr int BAL_TAU = 438;
constexpr int BAL_T = 30;
constexpr unsigned long long DEADLINE_TICKS = 100000000ull;

constexpr int AUX_SYS = 17;

struct Params {
    const int8_t* x;
    const int8_t* w;
    const float* sx;
    const float* sw;
    float* out;
    char* ws[NR];
    int rank;
    int pad;
    const int8_t* xsrc[NR];
    char* xdst[NR];
};

#define VMCNT(n) asm volatile("s_waitcnt vmcnt(%0)" :: "i"(n) : "memory")
#define LGKM0() asm volatile("s_waitcnt lgkmcnt(0)" ::: "memory")
#define ORDER() asm volatile("" ::: "memory")
#define LDS_LD(i) __hip_atomic_load(&lds[LDS_CNT / 4 + (i)], __ATOMIC_RELAXED, __HIP_MEMORY_SCOPE_WORKGROUP)
#define LDS_ST(i, v) __hip_atomic_store(&lds[LDS_CNT / 4 + (i)], (v), __ATOMIC_RELAXED, __HIP_MEMORY_SCOPE_WORKGROUP)

__device__ __forceinline__ rsrc_t mk_rsrc(const void* p, unsigned bytes) {
    return __builtin_amdgcn_make_buffer_rsrc(const_cast<void*>(p), 0, bytes, 0x00020000);
}
__device__ __forceinline__ unsigned rfl(unsigned v) { return __builtin_amdgcn_readfirstlane(v); }
__device__ __forceinline__ unsigned clk() { return (unsigned)__builtin_amdgcn_s_memrealtime(); }

__device__ __forceinline__ int row_sum(int v) {
    v += __builtin_amdgcn_update_dpp(0, v, 0xB1, 0xf, 0xf, false);
    v += __builtin_amdgcn_update_dpp(0, v, 0x4E, 0xf, 0xf, false);
    v += __builtin_amdgcn_update_dpp(0, v, 0x141, 0xf, 0xf, false);
    v += __builtin_amdgcn_update_dpp(0, v, 0x140, 0xf, 0xf, false);
    return v;
}

__device__ __forceinline__ unsigned wait_flags(rsrc_t ctrl, rsrc_t hdr, unsigned offw, unsigned offx, unsigned need,
                                               unsigned call, unsigned long long t0, unsigned wgate) {
    unsigned spins = 0;
    for (;;) {
        ORDER();
        const unsigned wv = __builtin_amdgcn_raw_buffer_load_b32(hdr, 0, offw, AUX_SYS);
        const u32x2 xv = __builtin_amdgcn_raw_buffer_load_b64(hdr, 0, offx, AUX_SYS);
        const unsigned a = rfl(xv.x), c = rfl(xv.y);
        const bool open = (int)(clk() - wgate) >= 0;
        const unsigned ready = ((rfl(wv) >= call && open) ? 1u : 0u) | ((a >= call && c >= call) ? 2u : 0u);
        if (ready & need) return ready;
        if (++spins > 256) {
            const unsigned tmo = rfl(__builtin_amdgcn_raw_buffer_load_b32(ctrl, 0, OFF_TMO, AUX_SYS));
            if (tmo == call || __builtin_amdgcn_s_memrealtime() - t0 > DEADLINE_TICKS) {
                __builtin_amdgcn_raw_buffer_store_b32(call, ctrl, 0, OFF_TMO, AUX_SYS);
                return 7u;
            }
            __builtin_amdgcn_s_sleep(8);
        }
        __builtin_amdgcn_s_sleep(1);
    }
}

constexpr int LDS_ACC = 0;
constexpr int LDS_CNT = 32768;
constexpr int W_INV = 8, W_CALL = 10, W_PLAN = 13, W_SCALE = 14, W_BAD = 15, W_TREF = 16, W_MOVED = 24;
constexpr int LDS_BYTES = 84 * 1024;

#define LDS_INIT() do { \
    *(u32x4*)&lds[tid * 16] = u32x4{0, 0, 0, 0}; *(u32x4*)&lds[tid * 16 + 4] = u32x4{0, 0, 0, 0}; \
    *(u32x4*)&lds[tid * 16 + 8] = u32x4{0, 0, 0, 0}; *(u32x4*)&lds[tid * 16 + 12] = u32x4{0, 0, 0, 0}; } while (0)

__global__ void __launch_bounds__(512, 2) gemm_rs_fused(Params P) {
    __shared__ __attribute__((aligned(16))) unsigned lds[LDS_BYTES / 4];
    const unsigned b = blockIdx.x & (NWG - 1);
    const unsigned pj = (b >> 3) & 7;
    const int8_t* px = P.xsrc[pj];
    char* xr = P.xdst[pj];
    char* myws = P.ws[0];
    const int8_t* pw = P.w;
    int prank = P.rank;
    unsigned long long t0 = __builtin_amdgcn_s_memrealtime();
    __builtin_amdgcn_sched_barrier(0);
    const int tid = threadIdx.x, lane = tid & 63;
    const unsigned wave = rfl(tid >> 6) & 7u;
    const unsigned idx = b >> 3;
    const unsigned gh = idx & 1;
    const unsigned rb = 15 - (idx >> 1);
    const unsigned g = (b & 7) * 2 + gh;
    const bool invwg = idx == 8;
    const bool pusher = (wave == 0) && (rb >= 12);
    const bool stager = (wave >= 1) && (wave <= 4) && (rb < 8);
    unsigned call, tbar;
#define ARGS_HERE() asm volatile("" : "+s"(px), "+s"(xr), "+s"(myws), "+s"(pw), "+s"(prank), "+s"(t0))
#define CTRL() mk_rsrc(myws + OFF_CTRL, CTRL_BYTES)

    if (__builtin_expect(pusher, 1)) {
        const unsigned q = b & 7;
        unsigned xs0 = (q >> 1) * 16384 + (q & 1) * 256, xvo = (lane & 7) * 512 + (lane >> 3) * 16;
        asm volatile("" : "+s"(xs0));
        asm volatile("" : "+v"(xvo));
        ARGS_HERE();
        const rsrc_t rx = mk_rsrc(px, MS * KS);
        const unsigned row = lane & 7, c = lane >> 3;
        u32x4 v[32];
        unsigned seqv = 0;
#pragma unroll
        for (int r = 0; r < 4; ++r) {
#pragma unroll
            for (int e = 0; e < 8; ++e) {
                v[r * 8 + e] = __builtin_amdgcn_raw_buffer_load_b128(rx, xvo + (e >> 2) * 128, xs0 + r * 65536 + (e & 3) * 4096, 0);
                if (r == 0 && e == 0) {
                    ORDER();
                    seqv = __builtin_amdgcn_raw_buffer_load_b32(CTRL(), 0, OFF_SEQ + b * 128, AUX_SYS);
                    ORDER();
                }
            }
        }
        ORDER();
        LDS_INIT();
        tbar = (unsigned)t0;
        char* xd = xr + REG_HDR;
        unsigned ds0 = (q >> 1) * 16384 + (q & 1) * 8192, dvo = c * 512 + row * 16 + REG_DATA;
        asm volatile("" : "+s"(xd), "+s"(ds0));
        asm volatile("" : "+v"(dvo));
        const rsrc_t rd = mk_rsrc(xd, 2 * REG_DATA);
        __builtin_amdgcn_sched_barrier(0);
        dvo ^= (seqv & 1) * REG_DATA;
        call = 0;
        const rsrc_t rf = mk_rsrc(xr + HDR_XFLAG, 2048);
#pragma unroll
        for (int s = 0; s < 32 + XWIN; ++s) {
            if (s == XWIN) {
                call = rfl(seqv) + 1;
                if (lane < 32) lds[LDS_CNT / 4 + lane] = lane == W_CALL ? call : 0;
                LGKM0();
                __builtin_amdgcn_s_barrier();
            }
            if (s >= XWIN) {
                const int done = s - XWIN;
                int younger = (s < 32 ? s : 32) - 1 - done;
                for (int r = 0; r < 4; ++r) { const int it = 8 * r + 7 + XWIN; if (it < s && it > done) ++younger; }
                VMCNT(younger);
                if ((done & 7) == 7) {
                    const unsigned unit = 4 * (done >> 3) + (q >> 1);
                    if (lane == 0) __builtin_amdgcn_raw_buffer_store_b32(call, rf, 0, unit * 128 + (q & 1) * 4, AUX_SYS);
                    ORDER();
                }
            }
            if (s < 32) {
                const int r = s >> 3, e = s & 7;
                __builtin_amdgcn_raw_buffer_store_b128(v[s], rd, dvo + (e & 3) * 128, ds0 + r * 65536 + (e >> 2) * 4096, AUX_SYS);
                ORDER();
            }
        }
    } else if (stager) {
        const unsigned h = (wave - 1) & 1, skb = rb * 2 + ((wave - 1) >> 1);
        ARGS_HERE();
        const rsrc_t rw = mk_rsrc(pw, KS * NN);
        unsigned d[16];
#pragma unroll
        for (int r = 0; r < 16; ++r)
            d[r] = __builtin_amdgcn_raw_buffer_load_b32(rw, lane * 4, (skb * 32 + h * 16 + r) * NN + g * 256, 0);
        LDS_INIT();
        LGKM0();
        __builtin_amdgcn_s_barrier();
        ORDER();
        call = rfl(lds[LDS_CNT / 4 + W_CALL]);
        tbar = clk();
        u32x4 o[4];
#pragma unroll
        for (int a = 0; a < 4; ++a) {
            const unsigned p0 = __builtin_amdgcn_perm(d[4 * a + 1], d[4 * a], 0x05010400u);
            const unsigned p1 = __builtin_amdgcn_perm(d[4 * a + 1], d[4 * a], 0x07030602u);
            const unsigned p2 = __builtin_amdgcn_perm(d[4 * a + 3], d[4 * a + 2], 0x05010400u);
            const unsigned p3 = __builtin_amdgcn_perm(d[4 * a + 3], d[4 * a + 2], 0x07030602u);
            o[0][a] = __builtin_amdgcn_perm(p2, p0, 0x05040100u);
            o[1][a] = __builtin_amdgcn_perm(p2, p0, 0x07060302u);
            o[2][a] = __builtin_amdgcn_perm(p3, p1, 0x05040100u);
            o[3][a] = __builtin_amdgcn_perm(p3, p1, 0x07060302u);
        }
        const rsrc_t rd = mk_rsrc(myws + OFF_WREG + reg_off(b & 7) + REG_HDR + (call & 1) * REG_DATA + gh * 131072, 131072);
#pragma unroll
        for (int j = 0; j < 4; ++j)
            __builtin_amdgcn_raw_buffer_store_b128(o[j], rd, (lane >> 3) * 16384 + (lane & 7) * 16,
                                                   skb * 1024 + h * 512 + j * 128, AUX_SYS);
        VMCNT(0);
        unsigned tk = 0;
        if (lane == 0)
            tk = __hip_atomic_fetch_add((gu32*)(myws + OFF_CTRL + OFF_WCNT + g * 128), 1u, __ATOMIC_RELAXED, __HIP_MEMORY_SCOPE_AGENT);
        if (rfl(tk) + 1 == call * 32u && lane == 0) {
#pragma unroll
            for (int jj = 1; jj <= NR; ++jj)
                __builtin_amdgcn_raw_buffer_store_b32(call, mk_rsrc(P.ws[jj & 7] + OFF_XREG + reg_off((8 - jj) & 7) + HDR_WFLAG, 2048),
                                                      0, g * 128, AUX_SYS);
        }
    } else {
        if (wave == 5 && invwg) asm volatile("buffer_inv sc0 sc1" ::: "memory");
        ARGS_HERE();
        const rsrc_t ctrl = CTRL();
        unsigned warm = 0;
        if (rb >= 8 && rb < 12 && wave < 2) {
            const char* wp = wave == 0 ? xr + HDR_WARM : P.ws[pj] + OFF_WREG + reg_off(b & 7) + HDR_WARM;
            if (wave != 0) __builtin_amdgcn_s_sleep(2);
            warm = __builtin_amdgcn_raw_buffer_load_b32(mk_rsrc(wp, 4), 0, 0, AUX_SYS);
        }
        unsigned seqv = 0;
        if (wave == 0) seqv = __builtin_amdgcn_raw_buffer_load_b32(ctrl, 0, OFF_SEQ + b * 128, AUX_SYS);
        LDS_INIT();
        if (wave == 0 && lane < 32) lds[LDS_CNT / 4 + lane] = lane == W_CALL ? seqv + 1 : 0;
        LGKM0();
        __builtin_amdgcn_s_barrier();
        ORDER();
        call = rfl(lds[LDS_CNT / 4 + W_CALL]);
        tbar = clk();
        asm volatile("" :: "v"(warm));
        if (wave == 5) {
            const unsigned xcc = __builtin_amdgcn_s_getreg(20 | (3 << 11)) & 15u;
            bool mine = invwg;
            while (!mine) {
                ORDER();
                if (rfl(__builtin_amdgcn_raw_buffer_load_b32(ctrl, 0, OFF_INV + xcc * 128, AUX_SYS)) >= call) break;
                if (__builtin_amdgcn_s_memrealtime() - t0 > 1000ull) mine = true;
                __builtin_amdgcn_s_sleep(1);
            }
            if (mine) {
                if (!invwg) __builtin_amdgcn_fence(__ATOMIC_ACQUIRE, "");
                VMCNT(0);
                if (lane == 0) __builtin_amdgcn_raw_buffer_store_b32(call, ctrl, 0, OFF_INV + xcc * 128, AUX_SYS);
            }
            if (lane == 0) LDS_ST(W_INV, 1u);
        }
    }

    const rsrc_t ctrl = CTRL();
    char* const wsv = P.ws[wave];
    char* const myx = myws + OFF_XREG + reg_off(wave);
    const rsrc_t hdr = mk_rsrc(myx, REG_HDR);
    const unsigned offw = HDR_WFLAG + g * 128, offx = HDR_XFLAG + rb * 128;
    const bool remote = wave != 0;
    {
        unsigned gx = GATE_X + (remote ? (rb >> 2) * GATE_XR : 0);
        if (gx > GATE_W - 30) gx = GATE_W - 30;
        while (clk() - tbar < gx) __builtin_amdgcn_s_sleep(2);
    }
    const unsigned wgate = tbar + (remote ? GATE_W : 0);
    unsigned st = wait_flags(ctrl, hdr, offw, offx, 3u, call, t0, wgate);
    while (rfl(LDS_LD(W_INV)) == 0) __builtin_amdgcn_s_sleep(1);
    ORDER();
    const unsigned doff = reg_off(b & 7) + REG_HDR + (call & 1) * REG_DATA + gh * 131072;
    const rsrc_t rwb = mk_rsrc(wsv + OFF_WREG + doff, 131072);
    const rsrc_t rxb = mk_rsrc(myx + REG_HDR + (call & 1) * REG_DATA + rb * 16384, 16384);
    const unsigned voff = lane * 16;
    const unsigned hg = 1 + (g >= 14 ? g - 14 : g >= 7 ? g - 7 : g);
    auto tile_id = [&](unsigned t) -> unsigned { return ((unsigned)prank + (t == 7 ? hg : t < hg ? t : t + 1)) & 7u; };
    const unsigned o0 = tile_id(0) * 16384;
    u32x4 ring[RING];
    u32x4 xf[16];
    if (st & 2u) {
        if (st & 1u) {
#pragma unroll
            for (int f = 0; f < PULL_D; ++f) ring[f] = __builtin_amdgcn_raw_buffer_load_b128(rwb, voff, o0 + f * 1024, 0);
        }
        ORDER();
#pragma unroll
        for (int kb = 0; kb < 16; ++kb) xf[kb] = __builtin_amdgcn_raw_buffer_load_b128(rxb, voff, kb * 1024, 0);
        if (!(st & 1u)) {
            st = wait_flags(ctrl, hdr, offw, offx, 1u, call, t0, wgate);
            ORDER();
#pragma unroll
            for (int f = 0; f < PULL_D; ++f) ring[f] = __builtin_amdgcn_raw_buffer_load_b128(rwb, voff, o0 + f * 1024, 0);
        }
#pragma unroll
        for (int f = PULL_D; f < RING; ++f) {
            VMCNT(PULL_D - 1);
            ring[f] = __builtin_amdgcn_raw_buffer_load_b128(rwb, voff, o0 + f * 1024, 0);
        }
    } else {
#pragma unroll
        for (int f = 0; f < RING; ++f) {
            if (f >= PULL_D) VMCNT(PULL_D - 1);
            ring[f] = __builtin_amdgcn_raw_buffer_load_b128(rwb, voff, o0 + f * 1024, 0);
        }
        st = wait_flags(ctrl, hdr, offw, offx, 2u, call, t0, tbar);
        ORDER();
#pragma unroll
        for (int kb = 0; kb < 16; ++kb) xf[kb] = __builtin_amdgcn_raw_buffer_load_b128(rxb, voff, kb * 1024, 0);
    }
    if ((st & 4u) && lane == 0) LDS_ST(W_BAD, 1u);

    const unsigned lcol = 4 * (lane & 7) + ((lane >> 3) & 3);
    unsigned* const lacc = &lds[LDS_ACC / 4 + (lane >> 5) * 128 + lcol];

    unsigned lvl = remote ? 1u : 3u;
    rsrc_t rl = rwb;
    unsigned ol = tile_id(7) * 16384;
    int nmoved = 0;
    auto tile = [&](unsigned t, bool last) __attribute__((always_inline)) {
        i32x16 acc = {0, 0, 0, 0, 0, 0, 0, 0, 0, 0, 0, 0, 0, 0, 0, 0};
        const unsigned ocur = tile_id(t) * 16384;
        unsigned onext = tile_id(t + 1) * 16384;
        rsrc_t rnext = rwb;
#pragma unroll
        for (int kb = 0; kb < 16; ++kb) {
            const bool more = !last || kb + RING < 16;
            if (!last && kb == 16 - RING && t == 6) {
                if ((rfl(LDS_LD(W_PLAN)) >> wave) & 1u) {
                    rl = mk_rsrc(P.ws[hg] + OFF_CREG + doff, 131072);
                    ol = ((wave - hg) & 7u) * 16384;
                }
                nmoved = (int)rfl(LDS_LD(W_MOVED + wave));
                rnext = rl;
                onext = ol;
            }
            if (more && remote && (kb & 1) == 0) {
                const unsigned pos = t * 16 + kb;
                const unsigned tref = rfl(LDS_LD(W_TREF + wave));
                const unsigned now = clk();
                if (kb == (BAL_POS & 15) && t == (BAL_POS >> 4) && rb < 4 && lane == 0)
                    __builtin_amdgcn_raw_buffer_store_b32(now, ctrl, 0, OFF_PROG + wave * 256 + g * 16 + rb * 4, AUX_SYS);
                int behind = (int)(now - tref) - (((int)pos - BAL_POS) * BAL_TAU >> 4);
                if (last || kb >= 16 - RING) behind += nmoved * ((last ? kb + RING : kb - (16 - RING)) + 16) * BAL_TAU >> 8;
                lvl = tref == 0 ? 1u : behind >= 3 * BAL_T ? 3u : behind >= BAL_T ? 2u : behind <= -BAL_T ? 0u : 1u;
            }
            if (more) {
                if (lvl <= 2) VMCNT(BOOST_D - 1);
                if (lvl <= 1) VMCNT(PULL_D - 1);
                if (lvl == 0) VMCNT(SLOW_D - 1);
            }
            __builtin_amdgcn_sched_barrier(0);
            acc = __builtin_amdgcn_mfma_i32_32x32x32_i8((i32x4)xf[kb], (i32x4)ring[kb % RING], acc, 0, 0, 0);
            __builtin_amdgcn_sched_barrier(0);
            if (more) {
                if (kb + RING < 16) ring[kb % RING] = __builtin_amdgcn_raw_buffer_load_b128(last ? rl : rwb, voff, (last ? ol : ocur) + (kb + RING) * 1024, 0);
                else ring[kb % RING] = __builtin_amdgcn_raw_buffer_load_b128(rnext, voff, onext + (kb + RING - 16) * 1024, 0);
            }
            __builtin_amdgcn_sched_barrier(0);
        }
#pragma unroll
        for (int r = 0; r < 16; ++r)
            __hip_atomic_fetch_add(lacc + t * 1024 + ((r & 3) + 8 * (r >> 2)) * 32, (unsigned)acc[r],
                                   __ATOMIC_RELAXED, __HIP_MEMORY_SCOPE_WORKGROUP);
        LGKM0();
        if (!last && lane == 0) __hip_atomic_fetch_add(&lds[LDS_CNT / 4 + t], 1u, __ATOMIC_RELAXED, __HIP_MEMORY_SCOPE_WORKGROUP);
    };

    bool need_copy = wave == 0 && rb >= 1 && rb <= 7 && rb != 8 - hg;
    bool need_plan = wave == 0;
    unsigned want = 0;
    auto side_jobs = [&]() __attribute__((always_inline)) {
        if (need_copy && rfl(LDS_LD(0)) >= 8u) {
            need_copy = false;
            if (rfl(LDS_LD(W_BAD)) == 0) {
                ORDER();
                const rsrc_t rs = mk_rsrc(P.ws[rb & 7] + OFF_WREG + doff + o0, 16384);
                const rsrc_t rc = mk_rsrc(myws + OFF_CREG + doff + (rb & 7) * 16384, 16384);
                u32x4 c[16];
#pragma unroll
                for (int i = 0; i < 16; ++i) c[i] = __builtin_amdgcn_raw_buffer_load_b128(rs, voff, i * 1024, 0);
#pragma unroll
                for (int i = 0; i < 16; ++i) __builtin_amdgcn_raw_buffer_store_b128(c[i], rc, voff, i * 1024, AUX_SYS);
                VMCNT(0);
                if (lane == 0)
                    __builtin_amdgcn_raw_buffer_store_b32(call, mk_rsrc(P.ws[8 - hg] + OFF_XREG + reg_off(hg) + HDR_CFLAG, 2048),
                                                          0, g * 128 + ((rb + hg) & 7u) * 4, AUX_SYS);
            }
        }
        if (need_plan && rfl(LDS_LD(1)) >= 8u) {
            ORDER();
            const u32x4 qa = __builtin_amdgcn_raw_buffer_load_b128(ctrl, voff, OFF_PROG, AUX_SYS) - (unsigned)t0;
            const u32x4 qb = __builtin_amdgcn_raw_buffer_load_b128(ctrl, voff, OFF_PROG + 1024, AUX_SYS) - (unsigned)t0;
            int sa = (int)min(min(qa.x, qa.y), min(qa.z, qa.w)), sb = (int)min(min(qb.x, qb.y), min(qb.z, qb.w));
            if (__all((lane < 16 || (unsigned)sa < 10000u) && (unsigned)sb < 10000u)) {
                need_plan = false;
                sa = row_sum(sa); sb = row_sum(sb);
                int S = 0;
#pragma unroll
                for (int v = 1; v < 8; ++v) {
                    const int sv = __builtin_amdgcn_readlane(v < 4 ? sa : sb, (v & 3) * 16);
                    S = lane == v ? sv : S;
                }
                if (lane >= 1 && lane < 8) LDS_ST(W_TREF + lane, ((unsigned)t0 + ((unsigned)S >> 4)) | 1u);
                int tot = 0;
#pragma unroll
                for (int v = 1; v < 8; ++v) tot += __builtin_amdgcn_readlane(S, v);
                const int ex = S * 7 - tot;
                const int shed = (lane >= 1 && lane < 8 && ex > 0) ? (ex + 2 * BAL_TAU) / (7 * BAL_TAU) : 0;
                const int lead = ex < 0 ? -ex : 0;
                int cap = 0;
#pragma unroll
                for (int v = 1; v < 8; ++v) cap += __builtin_amdgcn_readlane(lead, v);
                const float sc = (float)shed / (float)(cap + 1), ph = (float)((lane * 3) & 7) * 0.125f + 0.0625f;
                int nv = 0, cum = 0;
                bool mine = false;
#pragma unroll
                for (int hh = 1; hh < 8; ++hh) {
                    const int lh = __builtin_amdgcn_readlane(lead, hh);
                    cum += lh;
                    const int f = lh > 0 ? min((int)(sc * (float)cum + ph) - nv, hh <= 2 ? 3 : 2) : 0;
                    nv += f;
                    if ((unsigned)hh == hg) mine = (int)(g >= 14 ? 2 : g >= 7 ? 1 : 0) < f;
                }
                want = (unsigned)__ballot(mine) & 0xfeu;
                if (lane < 8) LDS_ST(W_MOVED + lane, (unsigned)nv);
                if (want) {
                    const unsigned wv = __builtin_amdgcn_raw_buffer_load_b32(mk_rsrc(P.ws[hg] + OFF_CREG + reg_off(b & 7) + HDR_WARM, 4), 0, 0, 0);
                    asm volatile("" :: "v"(wv));
                }
            }
        }
        if (want) {
            ORDER();
            const unsigned cf = __builtin_amdgcn_raw_buffer_load_b32(mk_rsrc(myws + OFF_XREG + reg_off(hg) + HDR_CFLAG + g * 128, 32), lane * 4, 0, AUX_SYS);
            const unsigned got = want & (unsigned)__ballot(lane < 8 && cf >= call);
            want &= ~got;
            if (lane == 0) __hip_atomic_fetch_or(&lds[LDS_CNT / 4 + W_PLAN], got, __ATOMIC_RELAXED, __HIP_MEMORY_SCOPE_WORKGROUP);
        }
    };
#pragma unroll 1
    for (unsigned t = 0; t < 7; ++t) {
        tile(t, false);
        if (wave == 0) side_jobs();
    }
    float* pout = P.out + (size_t)(rb * 32 + wave * 4) * NN + g * 256 + tile_id(7) * 32;
    asm volatile("" : "+s"(pout));
    tile(7, true);

    if (wave == 0) {
        if (lane == 0) __builtin_amdgcn_raw_buffer_store_b32(call, ctrl, 0, OFF_SEQ + b * 128, AUX_SYS);
        const float s = P.sx[0] * P.sw[0];
        if (lane == 0) LDS_ST(W_SCALE, __float_as_uint(s));
        const rsrc_t ro = mk_rsrc(P.out + (size_t)rb * 32 * NN + g * 256, (31 * NN + 256) * 4);
        const unsigned ooff = (lane >> 3) * (NN * 4) + (lane & 7) * 16;
#pragma unroll 1
        for (unsigned t = 0; t < 7; ++t) {
            while (rfl(LDS_LD(t)) < 8u) {
                if (t == 6) need_plan = false;
                side_jobs();
                if (need_plan && t > 1) __builtin_amdgcn_s_sleep(64);
                if (want) __builtin_amdgcn_s_sleep(16);
                __builtin_amdgcn_s_sleep(4);
            }
            ORDER();
            const bool bad = LDS_LD(W_BAD) != 0;
            i32x4 a[4];
            f32x4 y[4];
#pragma unroll
            for (int q = 0; q < 4; ++q) a[q] = *(const i32x4*)&lds[LDS_ACC / 4 + t * 1024 + q * 256 + lane * 4];
#pragma unroll
            for (int q = 0; q < 4; ++q)
#pragma unroll
                for (int e = 0; e < 4; ++e) y[q][e] = bad ? __builtin_nanf("") : fmaxf((float)a[q][e] * s, 0.0f);
            __builtin_amdgcn_sched_barrier(0);
#pragma unroll
            for (int q = 0; q < 4; ++q)
                __builtin_amdgcn_raw_buffer_store_b128((u32x4)y[q], ro, ooff, q * 8 * (NN * 4) + tile_id(t) * 128, AUX_SYS);
            __builtin_amdgcn_sched_barrier(0);
            asm volatile("s_nop 1");
        }
        LGKM0();
    }
    __builtin_amdgcn_s_barrier();
    ORDER();
    const u32x2 sb = *(const u32x2*)&lds[LDS_CNT / 4 + W_SCALE];
    const u32x2 a = *(const u32x2*)&lds[LDS_ACC / 4 + 7 * 1024 + wave * 128 + lane * 2];
    const float s = __uint_as_float(sb.x);
    u32x2 y;
    y.x = sb.y ? 0x7fc00000u : __float_as_uint(fmaxf((float)(int)a.x * s, 0.0f));
    y.y = sb.y ? 0x7fc00000u : __float_as_uint(fmaxf((float)(int)a.y * s, 0.0f));
    __builtin_amdgcn_raw_buffer_store_b64(y, mk_rsrc(pout, (3 * NN + 32) * 4), (lane >> 4) * (NN * 4) + (lane & 15) * 8, 0, AUX_SYS);
}

}

extern "C" void kernel_launch(int rank, int nranks, ncclComm_t comm,
                              hipStream_t stream,
                              void* const* d_in, const int* in_sizes, int n_in,
                              void* d_out, int out_size,
                              const int* in_dtypes, int out_dtype,
                              void* const* d_out_all,
                              void* const* d_ws_all, size_t ws_bytes) {
    if (nranks != NR || (unsigned)rank >= (unsigned)NR || n_in < 4 || ws_bytes < WS_NEED ||
        out_size != MS * NN || in_sizes[0] != 4096 * KS || in_sizes[1] != KS * NN) {
        fprintf(stderr, "unexpected problem shape\n");
        return;
    }
    Params P;
    P.x = (const int8_t*)d_in[0];
    P.w = (const int8_t*)d_in[1];
    P.sx = (const float*)d_in[2];
    P.sw = (const float*)d_in[3];
    P.out = (float*)d_out;
    for (int j = 0; j < NR; ++j) P.ws[j] = (char*)d_ws_all[(rank + j) % NR];
    P.rank = rank;
    P.pad = 0;
    for (int j = 0; j < NR; ++j) {
        P.xsrc[j] = P.x + (size_t)((rank + j) % NR) * MS * KS;
        P.xdst[j] = P.ws[j] + OFF_XREG + reg_off((8 - j) & 7);
    }
    gemm_rs_fused<<<NWG, 512, 0, stream>>>(P);
    HIP_CHECK(hipGetLastError());
}
```
